```python
import numpy as np

kernel_name = "gpu_gemm_a2a_m4096_k4096_n4096_bf16_reluqe4m3"


ABSMAX_RTOL = 0.09

COMPUTE_DTYPE = "bfloat16"
EPILOGUE = "relu_quant_e4m3"
IN_QUANT = None
OUT_QUANT = "e4m3"
IN_DTYPES = {"x": "bfloat16", "w_mat": "bfloat16"}
WORLD_SIZE = 8

def _to_bf16(x):
    u = np.ascontiguousarray(x, dtype=np.float32).view(np.uint32)
    return ((u + 0x7FFF + ((u >> 16) & 1)) & 0xFFFF0000).view(np.float32)

def setup_inputs():

    x = _to_bf16(np.random.randn(4096, 4096).astype(np.float32))
    w_mat = _to_bf16((np.random.randn(4096, 4096) / np.sqrt(4096)).astype(np.float32))
    return {"x": x, "w_mat": w_mat}

def _snap_fp8(x, mant_bits, max_abs, min_normal):
    a = np.abs(x.astype(np.float64))
    _m, _e = np.frexp(a)
    q = np.where(a >= min_normal, np.ldexp(1.0, _e - 1 - mant_bits),
                 min_normal * 2.0 ** -mant_bits)
    snapped = np.minimum(np.round(a / q) * q, max_abs)
    return (np.sign(x) * snapped).astype(np.float32)
def _quant_dequant(y, amax):
    scale = amax / 448.0
    q = _snap_fp8(y / scale, 3, 448.0, 2.0 ** -6)
    return (q * scale).astype(np.float32)
def _epilogue(y):
    y = np.maximum(y, 0.0)
    return _quant_dequant(y, np.abs(y).max())

def reference(x, w_mat):
    y = (x @ w_mat).astype(np.float32)
    return _epilogue(y)
```

```cpp
#include <hip/hip_runtime.h>
#include <rccl/rccl.h>
#include <cstdio>
#include <cstdint>

#define HIP_CHECK(expr) do { hipError_t _e = (expr); if (_e != hipSuccess) { \
    fprintf(stderr, "HIP error %s at %s:%d\n", hipGetErrorString(_e), __FILE__, __LINE__); \
    return; } } while (0)

namespace {

typedef short          s16x4 __attribute__((ext_vector_type(4)));
typedef short          s16x8 __attribute__((ext_vector_type(8)));
typedef __bf16         bf16x8 __attribute__((ext_vector_type(8)));
typedef float          f32x4 __attribute__((ext_vector_type(4)));
typedef unsigned       u32x4 __attribute__((ext_vector_type(4)));
typedef unsigned       u32x2 __attribute__((ext_vector_type(2)));
#define LAS __attribute__((address_space(3)))
#define GAS __attribute__((address_space(1)))

constexpr int NR = 8;
constexpr int MLOC = 512, KD = 4096, ND = 4096, NSH = 512;
constexpr int BM = 64, BN = 128, BK = 64;
constexpr int NKT = KD / BK;
constexpr int XBYTES = BM * BK * 2;
constexpr int WBYTES = BK * BN * 2;
constexpr int SLOT = XBYTES + WBYTES;
constexpr int NS = 6;
constexpr int CTRL = NS * SLOT;
constexpr int LDS_BYTES = CTRL + 256;
constexpr int NWG = 256;
constexpr int TROW = BN * 4 + 16;

constexpr size_t WS_CNT  = 0;
constexpr size_t WS_TMO  = 4096;
constexpr size_t WS_AMAX = 8192;
constexpr size_t WS_DONE = WS_AMAX + 2 * 16384;
constexpr size_t WS_END  = WS_DONE + 256 * 128;
constexpr unsigned long long DEADLINE = 100000000ull;

#define BARRIER() do { asm volatile("" ::: "memory"); __builtin_amdgcn_s_barrier(); asm volatile("" ::: "memory"); } while (0)
#define WAIT_VM(n) asm volatile("s_waitcnt vmcnt(" #n ")" ::: "memory")
#define WAIT_LGKM0() asm volatile("s_waitcnt lgkmcnt(0)" ::: "memory")

__device__ __forceinline__ unsigned rfl(unsigned v) { return __builtin_amdgcn_readfirstlane(v); }

__device__ __forceinline__ unsigned wave_max_u32(unsigned v) {
    v = max(v, (unsigned)__builtin_amdgcn_update_dpp(0, (int)v, 0xB1, 0xF, 0xF, true));
    v = max(v, (unsigned)__builtin_amdgcn_update_dpp(0, (int)v, 0x4E, 0xF, 0xF, true));
    v = max(v, (unsigned)__builtin_amdgcn_update_dpp(0, (int)v, 0x141, 0xF, 0xF, true));
    v = max(v, (unsigned)__builtin_amdgcn_update_dpp(0, (int)v, 0x140, 0xF, 0xF, true));
    unsigned a = __builtin_amdgcn_readlane(v, 0), b = __builtin_amdgcn_readlane(v, 16);
    unsigned c = __builtin_amdgcn_readlane(v, 32), d = __builtin_amdgcn_readlane(v, 48);
    return max(max(a, b), max(c, d));
}

#define GLDS(rsrc, ldsaddr, voff, soff) \
    asm volatile("s_mov_b32 m0, %0\n\ts_nop 0\n\tbuffer_load_dwordx4 %1, %2, %3 offen lds" \
                 :: "s"(ldsaddr), "v"(voff), "s"(rsrc), "s"(soff) : "memory")

#define STAMP(i) do {} while (0)

__global__ void __launch_bounds__(512, 2)
gemm_a2a_kernel(const void* __restrict__ x, const void* __restrict__ w,
                void* const* __restrict__ out_tab, void* const* __restrict__ ws_tab, int rank)
{
    extern __shared__ __attribute__((aligned(16))) char smem[];
    LAS char* lds = (LAS char*)smem;
    const unsigned ldsb = (unsigned)(uintptr_t)lds;
    const int tid = threadIdx.x, lane = tid & 63;
    const int wid = rfl(tid >> 6);
    const int b = blockIdx.x;
    const int dst = b & 7;
    const int j = b >> 3;
    const int m0 = (j & 7) * BM;
    const int nc0 = (j >> 3) * BN;
    char* myws = (char*)ws_tab[rank];
    LAS unsigned* ctrl = (LAS unsigned*)(lds + CTRL);
    STAMP(0);

    f32x4 v[4];
    const int orow = wid * 8 + (lane >> 3), ocol = (lane & 7) * 16;

    if (wid >= 4) {
        const int lw = wid - 4;
        __amdgpu_buffer_rsrc_t rx = __builtin_amdgcn_make_buffer_rsrc((void*)x, 0, MLOC * KD * 2, 0x00020000);
        __amdgpu_buffer_rsrc_t rw = __builtin_amdgcn_make_buffer_rsrc((void*)w, 0, KD * ND * 2, 0x00020000);
        unsigned vx[2], vw[4];
#pragma unroll
        for (int i = 0; i < 2; ++i) {
            const int row = 16 * lw + 8 * i + (lane >> 3);
            vx[i] = (unsigned)((m0 + row) * (KD * 2) + (((lane & 7) ^ ((row >> 1) & 7)) << 4));
        }
#pragma unroll
        for (int i = 0; i < 4; ++i) {
            const int k = 16 * lw + 4 * i + (lane >> 4);
            const int f = ((k & 3) << 1) | (k & 8);
            vw[i] = (unsigned)(k * (ND * 2) + (dst * NSH + nc0) * 2 + (((lane & 15) ^ f) << 4));
        }
        const unsigned lx = ldsb + lw * 2048, lwb = ldsb + XBYTES + lw * 4096;
#define ISSUE(kt, slotoff) do { \
            const unsigned sx_ = (unsigned)(kt) * (BK * 2), sw_ = (unsigned)(kt) * (BK * ND * 2); \
            GLDS(rw, lwb + (slotoff), vw[0], sw_); GLDS(rw, lwb + (slotoff) + 1024, vw[1], sw_); \
            GLDS(rx, lx + (slotoff), vx[0], sx_); \
            GLDS(rw, lwb + (slotoff) + 2048, vw[2], sw_); GLDS(rw, lwb + (slotoff) + 3072, vw[3], sw_); \
            GLDS(rx, lx + (slotoff) + 1024, vx[1], sx_); } while (0)
        asm volatile("s_nop 4" ::: "memory");
#pragma unroll
        for (int s = 0; s < NS; ++s) ISSUE(s, s * SLOT);
        WAIT_VM(30); BARRIER();
        STAMP(1);
        WAIT_VM(24); BARRIER();
        unsigned slot = 0;
        for (int t = 0; t < NKT - NS; ++t) {
            ISSUE(t + NS, slot);
            slot += SLOT; if (slot == NS * SLOT) slot = 0;
            WAIT_VM(24); BARRIER();
        }
        WAIT_VM(18); BARRIER();
        WAIT_VM(12); BARRIER();
        WAIT_VM(6);  BARRIER();
        WAIT_VM(0);  BARRIER();
        BARRIER();
        STAMP(2);
        BARRIER();
        BARRIER();
        const unsigned call = rfl(ctrl[4]);
#pragma unroll
        for (int i = 0; i < 4; ++i) v[i] = *(LAS f32x4*)(lds + orow * TROW + (ocol + 4 * i) * 4);
        const char* g = myws + WS_AMAX + (call & 1) * 16384 + lw * 4096 + lane * 16;
        GAS unsigned* tmo = (GAS unsigned*)(myws + WS_TMO);
        unsigned mx = 0, bad = 0;
        const unsigned long long t0 = __builtin_amdgcn_s_memrealtime();
        for (;;) {
            u32x4 a, c, d, e;
            asm volatile("global_load_dwordx4 %0, %4, off sc0 sc1\n\t"
                         "global_load_dwordx4 %1, %4, off offset:1024 sc0 sc1\n\t"
                         "global_load_dwordx4 %2, %4, off offset:2048 sc0 sc1\n\t"
                         "global_load_dwordx4 %3, %4, off offset:3072 sc0 sc1\n\t"
                         "s_waitcnt vmcnt(0)"
                         : "=&v"(a), "=&v"(c), "=&v"(d), "=&v"(e) : "v"(g) : "memory");
            const bool ok = a.y == call && a.w == call && c.y == call && c.w == call &&
                            d.y == call && d.w == call && e.y == call && e.w == call;
            mx = max(max(max(a.x, a.z), max(c.x, c.z)), max(max(d.x, d.z), max(e.x, e.z)));
            if (__all(ok)) break;
            if (rfl(__hip_atomic_load(tmo, __ATOMIC_RELAXED, __HIP_MEMORY_SCOPE_AGENT)) == call ||
                __builtin_amdgcn_s_memrealtime() - t0 > DEADLINE) {
                __hip_atomic_store(tmo, call, __ATOMIC_RELAXED, __HIP_MEMORY_SCOPE_AGENT);
                bad = 1;
                break;
            }
            __builtin_amdgcn_s_sleep(1);
        }
        mx = wave_max_u32(mx);
        if (lane == 0) { ctrl[8 + lw] = mx; ctrl[12 + lw] = bad; }
        STAMP(3);
    } else {
        const int wm = wid >> 1, wn = wid & 1;
        const int fr = lane & 15, fq = lane >> 4, q = (lane & 15) >> 2, p = lane & 3;
        unsigned callv = __hip_atomic_load((GAS unsigned*)(myws + WS_CNT) + b, __ATOMIC_RELAXED, __HIP_MEMORY_SCOPE_AGENT);
        char* peerws = (char*)ws_tab[lane & 7];
        unsigned ax[2], aw[4];
#pragma unroll
        for (int ks = 0; ks < 2; ++ks) {
            const int row = wm * 32 + fr;
            ax[ks] = row * 128 + (((ks * 4 + fq) ^ ((row >> 1) & 7)) << 4);
        }
#pragma unroll
        for (int c = 0; c < 4; ++c) {
            const int k = 8 * fq + q;
            const int f = ((k & 3) << 1) | (k & 8);
            aw[c] = XBYTES + k * 256 + (((wn * 8 + 2 * c + (p >> 1)) ^ f) << 4) + (p & 1) * 8;
        }
        f32x4 acc[2][4];
#pragma unroll
        for (int mi = 0; mi < 2; ++mi)
#pragma unroll
            for (int c = 0; c < 4; ++c) acc[mi][c] = (f32x4){0.f, 0.f, 0.f, 0.f};
        s16x8 xa[2][2][2];
        s16x4 wa[2][4][2][2];
#define READ(buf, slotoff) do { \
            _Pragma("unroll") for (int ks = 0; ks < 2; ++ks) { \
                _Pragma("unroll") for (int mi = 0; mi < 2; ++mi) \
                    xa[buf][mi][ks] = *(LAS s16x8*)(lds + (slotoff) + ax[ks] + mi * 2048); \
                _Pragma("unroll") for (int c = 0; c < 4; ++c) \
                    _Pragma("unroll") for (int t = 0; t < 2; ++t) \
                        wa[buf][c][ks][t] = __builtin_amdgcn_ds_read_tr16_b64_v4i16((LAS s16x4*)(lds + (slotoff) + aw[c] + ks * 8192 + t * 1024)); \
            } } while (0)
#define MMA(buf) do { \
            _Pragma("unroll") for (int ks = 0; ks < 2; ++ks) \
                _Pragma("unroll") for (int c = 0; c < 4; ++c) { \
                    const s16x8 wf = __builtin_shufflevector(wa[buf][c][ks][0], wa[buf][c][ks][1], 0, 1, 2, 3, 4, 5, 6, 7); \
                    _Pragma("unroll") for (int mi = 0; mi < 2; ++mi) \
                        acc[mi][c] = __builtin_amdgcn_mfma_f32_16x16x32_bf16(__builtin_bit_cast(bf16x8, wf), \
                                        __builtin_bit_cast(bf16x8, xa[buf][mi][ks]), acc[mi][c], 0, 0, 0); \
                } } while (0)
#define INTERLEAVE() do { \
            _Pragma("unroll") for (int i = 0; i < 10; ++i) { \
                __builtin_amdgcn_sched_group_barrier(0x008, 1, 0); \
                __builtin_amdgcn_sched_group_barrier(0x100, 2, 0); } \
            __builtin_amdgcn_sched_group_barrier(0x008, 6, 0); \
            __builtin_amdgcn_sched_barrier(0); } while (0)
        BARRIER();
        READ(0, 0);
        WAIT_LGKM0(); BARRIER();
        unsigned slot = SLOT;
        for (int t = 0; t < NKT - 2; t += 2) {
            READ(1, slot); MMA(0); INTERLEAVE();
            slot += SLOT; if (slot == NS * SLOT) slot = 0;
            WAIT_LGKM0(); BARRIER();
            READ(0, slot); MMA(1); INTERLEAVE();
            slot += SLOT; if (slot == NS * SLOT) slot = 0;
            WAIT_LGKM0(); BARRIER();
        }
        READ(1, slot); MMA(0); INTERLEAVE();
        WAIT_LGKM0(); BARRIER();
        MMA(1);
        STAMP(2);
        float mxf = 0.f;
#pragma unroll
        for (int mi = 0; mi < 2; ++mi)
#pragma unroll
            for (int c = 0; c < 4; ++c)
                mxf = fmaxf(fmaxf(fmaxf(acc[mi][c][0], acc[mi][c][1]), fmaxf(acc[mi][c][2], acc[mi][c][3])), mxf);
        const unsigned wmx = wave_max_u32(__float_as_uint(mxf) & 0x7fffffffu);
        const unsigned call = rfl(callv) + 1;
        if (lane == 0) { ctrl[wid] = wmx; ctrl[4] = call; }
        WAIT_LGKM0(); BARRIER();
        if (wid == 0) {
            const u32x4 m4 = *(LAS u32x4*)ctrl;
            const unsigned tmx = max(max(m4.x, m4.y), max(m4.z, m4.w));
            if (lane < NR) {
                char* gp = peerws + WS_AMAX + (call & 1) * 16384 + (rank * NWG + b) * 8;
                u32x2 gr = {tmx, call};
                asm volatile("global_store_dwordx2 %0, %1, off sc0 sc1\n\ts_nop 1" :: "v"(gp), "v"(gr) : "memory");
            }
            if (lane == 0)
                __hip_atomic_store((GAS unsigned*)(myws + WS_CNT) + b, call, __ATOMIC_RELAXED, __HIP_MEMORY_SCOPE_AGENT);
        }
        STAMP(3);
#pragma unroll
        for (int mi = 0; mi < 2; ++mi)
#pragma unroll
            for (int c = 0; c < 4; ++c)
                *(LAS f32x4*)(lds + (wm * 32 + mi * 16 + fr) * TROW + (wn * 64 + c * 16 + fq * 4) * 4) = acc[mi][c];
        WAIT_LGKM0(); BARRIER();
#pragma unroll
        for (int i = 0; i < 4; ++i) v[i] = *(LAS f32x4*)(lds + orow * TROW + (ocol + 4 * i) * 4);
    }
    char* optr = (char*)out_tab[dst] + (size_t)(rank * MLOC + m0 + orow) * NSH + nc0 + ocol;
    WAIT_LGKM0(); BARRIER();
    STAMP(4);
    const u32x4 g4 = *(LAS u32x4*)(ctrl + 8), b4 = *(LAS u32x4*)(ctrl + 12);
    const unsigned call = rfl(ctrl[4]);
    const float amax = __uint_as_float(max(max(g4.x, g4.y), max(g4.z, g4.w)));
    const bool timed_out = (b4.x | b4.y | b4.z | b4.w) != 0;
    const float scale = amax / 448.0f;
    const float inv = amax > 0.f ? 1.0f / scale : 0.f;
    u32x4 o;
#pragma unroll
    for (int i = 0; i < 4; ++i) {
        int r = __builtin_amdgcn_cvt_pk_fp8_f32(__builtin_amdgcn_fmed3f(v[i][0] * inv, 0.f, 448.f),
                                                __builtin_amdgcn_fmed3f(v[i][1] * inv, 0.f, 448.f), 0, false);
        r = __builtin_amdgcn_cvt_pk_fp8_f32(__builtin_amdgcn_fmed3f(v[i][2] * inv, 0.f, 448.f),
                                            __builtin_amdgcn_fmed3f(v[i][3] * inv, 0.f, 448.f), r, true);
        o[i] = timed_out ? 0x7f7f7f7fu : (unsigned)r;
    }
    asm volatile("global_store_dwordx4 %0, %1, off sc0 sc1\n\ts_nop 1" :: "v"(optr), "v"(o) : "memory");
    WAIT_VM(0);
    BARRIER();
    STAMP(5);
    if (wid == 0) {
        if (dst != rank && lane == 0) {
            GAS unsigned* df = (GAS unsigned*)((char*)ws_tab[dst] + WS_DONE + (rank * 32 + j) * 128);
            __hip_atomic_store(df, call, __ATOMIC_RELAXED, __HIP_MEMORY_SCOPE_SYSTEM);
        }
        if ((b >> 5) != rank) {
            GAS unsigned* f = (GAS unsigned*)(myws + WS_DONE + b * 128);
            GAS unsigned* tmo = (GAS unsigned*)(myws + WS_TMO);
            const unsigned long long t0 = __builtin_amdgcn_s_memrealtime();
            while (rfl(__hip_atomic_load(f, __ATOMIC_RELAXED, __HIP_MEMORY_SCOPE_SYSTEM)) < call) {
                if (rfl(__hip_atomic_load(tmo, __ATOMIC_RELAXED, __HIP_MEMORY_SCOPE_AGENT)) == call ||
                    __builtin_amdgcn_s_memrealtime() - t0 > DEADLINE) {
                    __hip_atomic_store(tmo, call, __ATOMIC_RELAXED, __HIP_MEMORY_SCOPE_AGENT);
                    if (lane == 0) {
                        const int sj = b & 31;
                        GAS unsigned char* po = (GAS unsigned char*)out_tab[rank] +
                            (size_t)((b >> 5) * MLOC + (sj & 7) * BM) * NSH + (sj >> 3) * BN;
                        __hip_atomic_store(po, (unsigned char)0x7f, __ATOMIC_RELAXED, __HIP_MEMORY_SCOPE_SYSTEM);
                    }
                    break;
                }
                __builtin_amdgcn_s_sleep(1);
            }
        }
        STAMP(6);
    }
}

}

extern "C" void kernel_launch(int rank, int nranks, ncclComm_t comm,
                              hipStream_t stream,
                              void* const* d_in, const int* in_sizes, int n_in,
                              void* d_out, int out_size,
                              const int* in_dtypes, int out_dtype,
                              void* const* d_out_all,
                              void* const* d_ws_all, size_t ws_bytes) {
    if (nranks != NR || n_in < 2 || in_sizes[0] != MLOC * KD || in_sizes[1] != KD * ND ||
        out_size != ND * NSH || ws_bytes < WS_END || (unsigned)rank >= (unsigned)NR) {
        fprintf(stderr, "kernel_launch: unexpected problem shape\n");
        return;
    }
    HIP_CHECK(hipFuncSetAttribute((const void*)gemm_a2a_kernel, hipFuncAttributeMaxDynamicSharedMemorySize, LDS_BYTES));
    hipLaunchKernelGGL(gemm_a2a_kernel, dim3(NWG), dim3(512), LDS_BYTES, stream,
                       (const void*)d_in[0], (const void*)d_in[1], d_out_all, d_ws_all, rank);
    HIP_CHECK(hipGetLastError());
}
```

```cpp
#include <hip/hip_runtime.h>
#include <rccl/rccl.h>
#include <cstdio>
#include <cstdint>

#define HIP_CHECK(expr) do { hipError_t _e = (expr); if (_e != hipSuccess) { \
    fprintf(stderr, "HIP error %s at %s:%d\n", hipGetErrorString(_e), __FILE__, __LINE__); \
    return; } } while (0)

namespace {

typedef short          s16x4 __attribute__((ext_vector_type(4)));
typedef short          s16x8 __attribute__((ext_vector_type(8)));
typedef __bf16         bf16x8 __attribute__((ext_vector_type(8)));
typedef float          f32x4 __attribute__((ext_vector_type(4)));
typedef unsigned       u32x4 __attribute__((ext_vector_type(4)));
typedef unsigned       u32x2 __attribute__((ext_vector_type(2)));
#define LAS __attribute__((address_space(3)))
#define GAS __attribute__((address_space(1)))

constexpr int NR = 8;
constexpr int MLOC = 512, KD = 4096, ND = 4096, NSH = 512;
constexpr int BM = 64, BN = 128, BK = 64;
constexpr int NKT = KD / BK;
constexpr int XBYTES = BM * BK * 2;
constexpr int WBYTES = BK * BN * 2;
constexpr int SLOT = XBYTES + WBYTES;
#define NSLOT 4
constexpr int NS = NSLOT;
constexpr int CTRL = NS * SLOT;
constexpr int LDS_BYTES = CTRL + 256;
static_assert(NS >= 4 && NS <= 6 && LDS_BYTES > 81920, "");
constexpr int NWG = 256;
#define DEPHASE 1
constexpr int TROW = BN * 4 + 16;

constexpr size_t WS_CNT  = 0;
constexpr size_t WS_TMO  = 4096;
constexpr size_t WS_AMAX = 8192;
constexpr size_t WS_DONE = WS_AMAX + 2 * 16384;
constexpr size_t WS_END  = WS_DONE + 256 * 128;
constexpr unsigned long long DEADLINE = 100000000ull;

#define BARRIER() do { asm volatile("" ::: "memory"); __builtin_amdgcn_s_barrier(); asm volatile("" ::: "memory"); } while (0)
#define WAIT_VM(n) asm volatile("s_waitcnt vmcnt(%0)" :: "n"(n) : "memory")
#define WAIT_LGKM0() asm volatile("s_waitcnt lgkmcnt(0)" ::: "memory")

__device__ __forceinline__ unsigned rfl(unsigned v) { return __builtin_amdgcn_readfirstlane(v); }

__device__ __forceinline__ unsigned wave_max_u32(unsigned v) {
    v = max(v, (unsigned)__builtin_amdgcn_update_dpp(0, (int)v, 0xB1, 0xF, 0xF, true));
    v = max(v, (unsigned)__builtin_amdgcn_update_dpp(0, (int)v, 0x4E, 0xF, 0xF, true));
    v = max(v, (unsigned)__builtin_amdgcn_update_dpp(0, (int)v, 0x141, 0xF, 0xF, true));
    v = max(v, (unsigned)__builtin_amdgcn_update_dpp(0, (int)v, 0x140, 0xF, 0xF, true));
    unsigned a = __builtin_amdgcn_readlane(v, 0), b = __builtin_amdgcn_readlane(v, 16);
    unsigned c = __builtin_amdgcn_readlane(v, 32), d = __builtin_amdgcn_readlane(v, 48);
    return max(max(a, b), max(c, d));
}

#define GLDS(rsrc, ldsaddr, voff, soff) \
    asm volatile("s_mov_b32 m0, %0\n\ts_nop 0\n\tbuffer_load_dwordx4 %1, %2, %3 offen lds" \
                 :: "s"(ldsaddr), "v"(voff), "s"(rsrc), "s"(soff) : "memory")

#define STAMP(i) do {} while (0)

__global__ void __launch_bounds__(512, 2)
gemm_a2a_kernel(const void* __restrict__ x, const void* __restrict__ w,
                void* const* __restrict__ out_tab, void* const* __restrict__ ws_tab, int rank)
{
    extern __shared__ __attribute__((aligned(16))) char smem[];
    LAS char* lds = (LAS char*)smem;
    const unsigned ldsb = (unsigned)(uintptr_t)lds;
    const int tid = threadIdx.x, lane = tid & 63;
    const int wid = rfl(tid >> 6);
    const int b = blockIdx.x;
    const int dst = b & 7;
    const int j = b >> 3;
    const int m0 = (j & 7) * BM;
    const int nc0 = (j >> 3) * BN;
    char* myws = (char*)ws_tab[rank];
    LAS unsigned* ctrl = (LAS unsigned*)(lds + CTRL);
    STAMP(0);

    f32x4 v[4];
    const int orow = wid * 8 + (lane >> 3), ocol = (lane & 7) * 16;

    if (wid >= 4) {
        const int lw = wid - 4;
        __amdgpu_buffer_rsrc_t rx = __builtin_amdgcn_make_buffer_rsrc((void*)x, 0, MLOC * KD * 2, 0x00020000);
        __amdgpu_buffer_rsrc_t rw = __builtin_amdgcn_make_buffer_rsrc((void*)w, 0, KD * ND * 2, 0x00020000);
        unsigned vx[2], vw[4];
#pragma unroll
        for (int i = 0; i < 2; ++i) {
            const int row = 16 * lw + 8 * i + (lane >> 3);
            vx[i] = (unsigned)((m0 + row) * (KD * 2) + (((lane & 7) ^ ((row >> 1) & 7)) << 4));
        }
#pragma unroll
        for (int i = 0; i < 4; ++i) {
            const int k = 16 * lw + 4 * i + (lane >> 4);
            const int f = ((k & 3) << 1) | (k & 8);
            vw[i] = (unsigned)(k * (ND * 2) + (dst * NSH + nc0) * 2 + (((lane & 15) ^ f) << 4));
        }
        const unsigned lx = ldsb + lw * 2048, lwb = ldsb + XBYTES + lw * 4096;
        int ktab;
        {
            const int ph = DEPHASE ? dst : 0, rot = ((j & 7) + 2 * (j >> 3)) & 7;
            const int bs = lane < ph ? 0 : ph + ((lane - ph) & ~7);
            const int sz = lane < ph ? ph : min(8, NKT - bs);
            ktab = bs + (lane - bs + rot) % sz;
        }
#define ISSUE(t, slotoff) do { \
            const unsigned kt_ = __builtin_amdgcn_readlane(ktab, t); \
            const unsigned sx_ = kt_ * (BK * 2), sw_ = kt_ * (BK * ND * 2); \
            GLDS(rw, lwb + (slotoff), vw[0], sw_); GLDS(rw, lwb + (slotoff) + 1024, vw[1], sw_); \
            GLDS(rx, lx + (slotoff), vx[0], sx_); \
            GLDS(rw, lwb + (slotoff) + 2048, vw[2], sw_); GLDS(rw, lwb + (slotoff) + 3072, vw[3], sw_); \
            GLDS(rx, lx + (slotoff) + 1024, vx[1], sx_); } while (0)
        asm volatile("s_nop 4" ::: "memory");
        ISSUE(0, 0); ISSUE(1, SLOT);
        WAIT_VM(6); BARRIER();
        STAMP(1);
        ISSUE(2, 2 * SLOT);
        WAIT_VM(6); BARRIER();
#pragma unroll
        for (int s = 3; s < NS; ++s) ISSUE(s, s * SLOT);
        unsigned slot = 0;
        for (int t = 0; t < NKT - NS; ++t) {
            ISSUE(t + NS, slot);
            slot += SLOT; if (slot == NS * SLOT) slot = 0;
            WAIT_VM(6 * (NS - 2)); BARRIER();
        }
        if (NS >= 6) { WAIT_VM(18); BARRIER(); }
        if (NS >= 5) { WAIT_VM(12); BARRIER(); }
        WAIT_VM(6);  BARRIER();
        WAIT_VM(0);  BARRIER();
        BARRIER();
        STAMP(2);
        BARRIER();
        BARRIER();
        const unsigned call = rfl(ctrl[4]);
#pragma unroll
        for (int i = 0; i < 4; ++i) v[i] = *(LAS f32x4*)(lds + orow * TROW + (ocol + 4 * i) * 4);
        const char* g = myws + WS_AMAX + (call & 1) * 16384 + lw * 4096 + lane * 16;
        GAS unsigned* tmo = (GAS unsigned*)(myws + WS_TMO);
        unsigned mx = 0, bad = 0;
        const unsigned long long t0 = __builtin_amdgcn_s_memrealtime();
        for (;;) {
            u32x4 a, c, d, e;
            asm volatile("global_load_dwordx4 %0, %4, off sc0 sc1\n\t"
                         "global_load_dwordx4 %1, %4, off offset:1024 sc0 sc1\n\t"
                         "global_load_dwordx4 %2, %4, off offset:2048 sc0 sc1\n\t"
                         "global_load_dwordx4 %3, %4, off offset:3072 sc0 sc1\n\t"
                         "s_waitcnt vmcnt(0)"
                         : "=&v"(a), "=&v"(c), "=&v"(d), "=&v"(e) : "v"(g) : "memory");
            const bool ok = a.y == call && a.w == call && c.y == call && c.w == call &&
                            d.y == call && d.w == call && e.y == call && e.w == call;
            mx = max(max(max(a.x, a.z), max(c.x, c.z)), max(max(d.x, d.z), max(e.x, e.z)));
            if (__all(ok)) break;
            if (rfl(__hip_atomic_load(tmo, __ATOMIC_RELAXED, __HIP_MEMORY_SCOPE_AGENT)) == call ||
                __builtin_amdgcn_s_memrealtime() - t0 > DEADLINE) {
                __hip_atomic_store(tmo, call, __ATOMIC_RELAXED, __HIP_MEMORY_SCOPE_AGENT);
                bad = 1;
                break;
            }
            __builtin_amdgcn_s_sleep(1);
        }
        mx = wave_max_u32(mx);
        if (lane == 0) { ctrl[8 + lw] = mx; ctrl[12 + lw] = bad; }
        STAMP(3);
    } else {
        const int wm = wid >> 1, wn = wid & 1;
        const int fr = lane & 15, fq = lane >> 4, q = (lane & 15) >> 2, p = lane & 3;
        unsigned callv = __hip_atomic_load((GAS unsigned*)(myws + WS_CNT) + b, __ATOMIC_RELAXED, __HIP_MEMORY_SCOPE_AGENT);
        char* peerws = (char*)ws_tab[lane & 7];
        unsigned ax[2], aw[4];
#pragma unroll
        for (int ks = 0; ks < 2; ++ks) {
            const int row = wm * 32 + fr;
            ax[ks] = row * 128 + (((ks * 4 + fq) ^ ((row >> 1) & 7)) << 4);
        }
#pragma unroll
        for (int c = 0; c < 4; ++c) {
            const int k = 8 * fq + q;
            const int f = ((k & 3) << 1) | (k & 8);
            aw[c] = XBYTES + k * 256 + (((wn * 8 + 2 * c + (p >> 1)) ^ f) << 4) + (p & 1) * 8;
        }
        f32x4 acc[2][4];
#pragma unroll
        for (int mi = 0; mi < 2; ++mi)
#pragma unroll
            for (int c = 0; c < 4; ++c) acc[mi][c] = (f32x4){0.f, 0.f, 0.f, 0.f};
        s16x8 xa[2][2][2];
        s16x4 wa[2][4][2][2];
#define READ(buf, slotoff) do { \
            _Pragma("unroll") for (int ks = 0; ks < 2; ++ks) { \
                _Pragma("unroll") for (int mi = 0; mi < 2; ++mi) \
                    xa[buf][mi][ks] = *(LAS s16x8*)(lds + (slotoff) + ax[ks] + mi * 2048); \
                _Pragma("unroll") for (int c = 0; c < 4; ++c) \
                    _Pragma("unroll") for (int t = 0; t < 2; ++t) \
                        wa[buf][c][ks][t] = __builtin_amdgcn_ds_read_tr16_b64_v4i16((LAS s16x4*)(lds + (slotoff) + aw[c] + ks * 8192 + t * 1024)); \
            } } while (0)
#define MMA(buf) do { \
            _Pragma("unroll") for (int ks = 0; ks < 2; ++ks) \
                _Pragma("unroll") for (int c = 0; c < 4; ++c) { \
                    const s16x8 wf = __builtin_shufflevector(wa[buf][c][ks][0], wa[buf][c][ks][1], 0, 1, 2, 3, 4, 5, 6, 7); \
                    _Pragma("unroll") for (int mi = 0; mi < 2; ++mi) \
                        acc[mi][c] = __builtin_amdgcn_mfma_f32_16x16x32_bf16(__builtin_bit_cast(bf16x8, wf), \
                                        __builtin_bit_cast(bf16x8, xa[buf][mi][ks]), acc[mi][c], 0, 0, 0); \
                } } while (0)
#define INTERLEAVE() do { \
            _Pragma("unroll") for (int i = 0; i < 10; ++i) { \
                __builtin_amdgcn_sched_group_barrier(0x008, 1, 0); \
                __builtin_amdgcn_sched_group_barrier(0x100, 2, 0); } \
            __builtin_amdgcn_sched_group_barrier(0x008, 6, 0); \
            __builtin_amdgcn_sched_barrier(0); } while (0)
        BARRIER();
        READ(0, 0);
        WAIT_LGKM0(); BARRIER();
        unsigned slot = SLOT;
        for (int t = 0; t < NKT - 2; t += 2) {
            READ(1, slot); MMA(0); INTERLEAVE();
            slot += SLOT; if (slot == NS * SLOT) slot = 0;
            WAIT_LGKM0(); BARRIER();
            READ(0, slot); MMA(1); INTERLEAVE();
            slot += SLOT; if (slot == NS * SLOT) slot = 0;
            WAIT_LGKM0(); BARRIER();
        }
        READ(1, slot); MMA(0); INTERLEAVE();
        WAIT_LGKM0(); BARRIER();
        MMA(1);
        STAMP(2);
        float mxf = 0.f;
#pragma unroll
        for (int mi = 0; mi < 2; ++mi)
#pragma unroll
            for (int c = 0; c < 4; ++c)
                mxf = fmaxf(fmaxf(fmaxf(acc[mi][c][0], acc[mi][c][1]), fmaxf(acc[mi][c][2], acc[mi][c][3])), mxf);
        const unsigned wmx = wave_max_u32(__float_as_uint(mxf) & 0x7fffffffu);
        const unsigned call = rfl(callv) + 1;
        if (lane == 0) { ctrl[wid] = wmx; ctrl[4] = call; }
        WAIT_LGKM0(); BARRIER();
        if (wid == 0) {
            const u32x4 m4 = *(LAS u32x4*)ctrl;
            const unsigned tmx = max(max(m4.x, m4.y), max(m4.z, m4.w));
            if (lane < NR) {
                char* gp = peerws + WS_AMAX + (call & 1) * 16384 + (rank * NWG + b) * 8;
                u32x2 gr = {tmx, call};
                asm volatile("global_store_dwordx2 %0, %1, off sc0 sc1\n\ts_nop 1" :: "v"(gp), "v"(gr) : "memory");
            }
            if (lane == 0)
                __hip_atomic_store((GAS unsigned*)(myws + WS_CNT) + b, call, __ATOMIC_RELAXED, __HIP_MEMORY_SCOPE_AGENT);
        }
        STAMP(3);
#pragma unroll
        for (int mi = 0; mi < 2; ++mi)
#pragma unroll
            for (int c = 0; c < 4; ++c)
                *(LAS f32x4*)(lds + (wm * 32 + mi * 16 + fr) * TROW + (wn * 64 + c * 16 + fq * 4) * 4) = acc[mi][c];
        WAIT_LGKM0(); BARRIER();
#pragma unroll
        for (int i = 0; i < 4; ++i) v[i] = *(LAS f32x4*)(lds + orow * TROW + (ocol + 4 * i) * 4);
    }
    char* optr = (char*)out_tab[dst] + (size_t)(rank * MLOC + m0 + orow) * NSH + nc0 + ocol;
    WAIT_LGKM0(); BARRIER();
    STAMP(4);
    const u32x4 g4 = *(LAS u32x4*)(ctrl + 8), b4 = *(LAS u32x4*)(ctrl + 12);
    const unsigned call = rfl(ctrl[4]);
    const float amax = __uint_as_float(max(max(g4.x, g4.y), max(g4.z, g4.w)));
    const bool timed_out = (b4.x | b4.y | b4.z | b4.w) != 0;
    const float scale = amax / 448.0f;
    const float inv = amax > 0.f ? 1.0f / scale : 0.f;
    u32x4 o;
#pragma unroll
    for (int i = 0; i < 4; ++i) {
        int r = __builtin_amdgcn_cvt_pk_fp8_f32(__builtin_amdgcn_fmed3f(v[i][0] * inv, 0.f, 448.f),
                                                __builtin_amdgcn_fmed3f(v[i][1] * inv, 0.f, 448.f), 0, false);
        r = __builtin_amdgcn_cvt_pk_fp8_f32(__builtin_amdgcn_fmed3f(v[i][2] * inv, 0.f, 448.f),
                                            __builtin_amdgcn_fmed3f(v[i][3] * inv, 0.f, 448.f), r, true);
        o[i] = timed_out ? 0x7f7f7f7fu : (unsigned)r;
    }
    asm volatile("global_store_dwordx4 %0, %1, off sc0 sc1\n\ts_nop 1" :: "v"(optr), "v"(o) : "memory");
    WAIT_VM(0);
    BARRIER();
    STAMP(5);
    if (wid == 0) {
        if (dst != rank && lane == 0) {
            GAS unsigned* df = (GAS unsigned*)((char*)ws_tab[dst] + WS_DONE + (rank * 32 + j) * 128);
            __hip_atomic_store(df, call, __ATOMIC_RELAXED, __HIP_MEMORY_SCOPE_SYSTEM);
        }
        if ((b >> 5) != rank) {
            GAS unsigned* f = (GAS unsigned*)(myws + WS_DONE + b * 128);
            GAS unsigned* tmo = (GAS unsigned*)(myws + WS_TMO);
            const unsigned long long t0 = __builtin_amdgcn_s_memrealtime();
            while (rfl(__hip_atomic_load(f, __ATOMIC_RELAXED, __HIP_MEMORY_SCOPE_SYSTEM)) < call) {
                if (rfl(__hip_atomic_load(tmo, __ATOMIC_RELAXED, __HIP_MEMORY_SCOPE_AGENT)) == call ||
                    __builtin_amdgcn_s_memrealtime() - t0 > DEADLINE) {
                    __hip_atomic_store(tmo, call, __ATOMIC_RELAXED, __HIP_MEMORY_SCOPE_AGENT);
                    if (lane == 0) {
                        const int sj = b & 31;
                        GAS unsigned char* po = (GAS unsigned char*)out_tab[rank] +
                            (size_t)((b >> 5) * MLOC + (sj & 7) * BM) * NSH + (sj >> 3) * BN;
                        __hip_atomic_store(po, (unsigned char)0x7f, __ATOMIC_RELAXED, __HIP_MEMORY_SCOPE_SYSTEM);
                    }
                    break;
                }
                __builtin_amdgcn_s_sleep(1);
            }
        }
        STAMP(6);
    }
}

}

extern "C" void kernel_launch(int rank, int nranks, ncclComm_t comm,
                              hipStream_t stream,
                              void* const* d_in, const int* in_sizes, int n_in,
                              void* d_out, int out_size,
                              const int* in_dtypes, int out_dtype,
                              void* const* d_out_all,
                              void* const* d_ws_all, size_t ws_bytes) {
    if (nranks != NR || n_in < 2 || in_sizes[0] != MLOC * KD || in_sizes[1] != KD * ND ||
        out_size != ND * NSH || ws_bytes < WS_END || (unsigned)rank >= (unsigned)NR) {
        fprintf(stderr, "kernel_launch: unexpected problem shape\n");
        return;
    }
    HIP_CHECK(hipFuncSetAttribute((const void*)gemm_a2a_kernel, hipFuncAttributeMaxDynamicSharedMemorySize, LDS_BYTES));
    hipLaunchKernelGGL(gemm_a2a_kernel, dim3(NWG), dim3(512), LDS_BYTES, stream,
                       (const void*)d_in[0], (const void*)d_in[1], d_out_all, d_ws_all, rank);
    HIP_CHECK(hipGetLastError());
}
```

```cpp
#include <hip/hip_runtime.h>
#include <rccl/rccl.h>
#include <cstdio>
#include <cstdint>

#define HIP_CHECK(expr) do { hipError_t _e = (expr); if (_e != hipSuccess) { \
    fprintf(stderr, "HIP error %s at %s:%d\n", hipGetErrorString(_e), __FILE__, __LINE__); \
    return; } } while (0)

namespace {

typedef short          s16x4 __attribute__((ext_vector_type(4)));
typedef short          s16x8 __attribute__((ext_vector_type(8)));
typedef float          f32x4 __attribute__((ext_vector_type(4)));
typedef unsigned       u32x4 __attribute__((ext_vector_type(4)));
typedef unsigned       u32x2 __attribute__((ext_vector_type(2)));
#define LAS __attribute__((address_space(3)))
#define GAS __attribute__((address_space(1)))

constexpr int NR = 8;
constexpr int MLOC = 512, KD = 4096, ND = 4096, NSH = 512;
constexpr int BM = 128, BN = 128, BK = 64;
constexpr int NKT = KD / BK;
#define KA_STEPS 28
constexpr int KA = KA_STEPS;
#define UNIT 1
constexpr int XBYTES = BM * BK * 2;
constexpr int SLOT = XBYTES + BK * BN * 2;
constexpr int NS = 4;
constexpr int CTRL = NS * SLOT;
constexpr int LDS_BYTES = CTRL + 256;
constexpr int NWG = 256, NTILE = 128;
constexpr int TROW = BN * 4 + 16;
static_assert(KA % 4 == 0 && KA >= 8 && NKT - KA >= 8 && BM * TROW <= CTRL, "");

constexpr size_t WS_CNT   = 0;
constexpr size_t WS_TMO   = 4096;
constexpr size_t WS_AMAX  = 8192;
constexpr size_t WS_DONE  = WS_AMAX + 2 * 8192;
constexpr size_t WS_PFLAG = WS_DONE + NTILE * 128;
constexpr size_t WS_PART  = 1 << 20;
constexpr size_t WS_END   = WS_PART + (size_t)NTILE * 65536;
static_assert(WS_PFLAG + NTILE * 512 <= WS_PART, "");
constexpr unsigned long long DEADLINE = 100000000ull;

#define BARRIER() do { asm volatile("" ::: "memory"); __builtin_amdgcn_s_barrier(); asm volatile("" ::: "memory"); } while (0)
#define WAIT_VM(n) asm volatile("s_waitcnt vmcnt(%0)" :: "n"(n) : "memory")
#define WAIT_LGKM0() __builtin_amdgcn_s_waitcnt(0xC07F)
#define SB() __builtin_amdgcn_sched_barrier(0)

__device__ __forceinline__ unsigned rfl(unsigned v) { return __builtin_amdgcn_readfirstlane(v); }

__device__ __forceinline__ unsigned wave_max_u32(unsigned v) {
    v = max(v, (unsigned)__builtin_amdgcn_update_dpp(0, (int)v, 0xB1, 0xF, 0xF, true));
    v = max(v, (unsigned)__builtin_amdgcn_update_dpp(0, (int)v, 0x4E, 0xF, 0xF, true));
    v = max(v, (unsigned)__builtin_amdgcn_update_dpp(0, (int)v, 0x141, 0xF, 0xF, true));
    v = max(v, (unsigned)__builtin_amdgcn_update_dpp(0, (int)v, 0x140, 0xF, 0xF, true));
    unsigned a = __builtin_amdgcn_readlane(v, 0), b = __builtin_amdgcn_readlane(v, 16);
    unsigned c = __builtin_amdgcn_readlane(v, 32), d = __builtin_amdgcn_readlane(v, 48);
    return max(max(a, b), max(c, d));
}

__device__ __forceinline__ bool wait_flag(GAS unsigned* flag, GAS unsigned* tmo, unsigned call) {
    const unsigned long long t0 = __builtin_amdgcn_s_memrealtime();
    while (rfl(__hip_atomic_load(flag, __ATOMIC_RELAXED, __HIP_MEMORY_SCOPE_SYSTEM)) < call) {
        if (rfl(__hip_atomic_load(tmo, __ATOMIC_RELAXED, __HIP_MEMORY_SCOPE_AGENT)) == call ||
            __builtin_amdgcn_s_memrealtime() - t0 > DEADLINE) {
            __hip_atomic_store(tmo, call, __ATOMIC_RELAXED, __HIP_MEMORY_SCOPE_AGENT);
            return true;
        }
        __builtin_amdgcn_s_sleep(1);
    }
    return false;
}

#define GLDS(rsrc, ldsaddr, voff, soff) \
    asm volatile("s_mov_b32 m0, %0\n\ts_nop 0\n\tbuffer_load_dwordx4 %1, %2, %3 offen lds" \
                 :: "s"(ldsaddr), "v"(voff), "s"(rsrc), "s"(soff) : "memory")
#define GLDS_SYS(rsrc, ldsaddr, voff, soff) \
    asm volatile("s_mov_b32 m0, %0\n\ts_nop 0\n\tbuffer_load_dwordx4 %1, %2, %3 offen sc0 sc1 lds" \
                 :: "s"(ldsaddr), "v"(voff), "s"(rsrc), "s"(soff) : "memory")

#define STAMP(i) do {} while (0)
#define STAMP_OUT() do {} while (0)

struct Frag { s16x8 x[8]; s16x4 w[16]; };

template <int SL, int R>
__device__ __forceinline__ void rd(Frag& f, const unsigned (&ax)[2][2], const unsigned (&aw)[2][4]) {
    constexpr int g = R / 3, e = R % 3, so = (SL & 1) * SLOT, ks = g >> 2, i = g & 3;
    if constexpr (e == 0) f.x[g] = *(LAS s16x8*)(uintptr_t)(ax[SL >> 1][ks] + so + i * 2048);
    else f.w[g * 2 + e - 1] = __builtin_amdgcn_ds_read_tr16_b64_v4i16(
             (LAS s16x4*)(uintptr_t)(aw[SL >> 1][i] + so + ks * 8192 + (e - 1) * 1024));
}
template <int SL, int R>
__device__ __forceinline__ void rd_all(Frag& f, const unsigned (&ax)[2][2], const unsigned (&aw)[2][4]) {
    if constexpr (R < 24) { rd<SL, R>(f, ax, aw); rd_all<SL, R + 1>(f, ax, aw); }
}
template <int SL, int G>
__device__ __forceinline__ void kstep(f32x4 (&acc)[16], const Frag& cur, Frag& nxt,
                                      const unsigned (&ax)[2][2], const unsigned (&aw)[2][4]) {
    if constexpr (G < 32) {
        constexpr int ks = G >> 4, c = (G >> 2) & 3, mi = G & 3;
        const s16x8 wf = __builtin_shufflevector(cur.w[(ks * 4 + c) * 2], cur.w[(ks * 4 + c) * 2 + 1], 0, 1, 2, 3, 4, 5, 6, 7);
        asm volatile("v_mfma_f32_16x16x32_bf16 %0, %1, %2, %0" : "+v"(acc[mi * 4 + c]) : "v"(wf), "v"(cur.x[ks * 4 + mi]));
        if constexpr (SL >= 0 && G < 24) rd<SL, G>(nxt, ax, aw);
        if constexpr (SL >= 0 && G == 31) { SB(); WAIT_LGKM0(); BARRIER(); }
        SB();
        kstep<SL, G + 1>(acc, cur, nxt, ax, aw);
    }
}

struct Args {
    const void* x; const void* w;
    void* myws; void* myout;
    void* out[NR]; void* ws[NR];
    int rank, pad;
};

__global__ void __launch_bounds__(512, 2)
gemm_a2a_kernel(const Args args)
{
    const void* x = args.x; const void* w = args.w;
    const int rank = args.rank;
    asm volatile("" :: "s"(x), "s"(w));
    extern __shared__ __attribute__((aligned(16))) char smem[];
    LAS char* lds = (LAS char*)smem;
    const unsigned ldsb = (unsigned)(uintptr_t)lds;
    const int tid = threadIdx.x, lane = tid & 63;
    const int wid = rfl(tid >> 6);
    const int b = blockIdx.x;
    const int dst = b & 7;
    const int isB = (b >> 3) & 1, tile = b >> 4, tm = tile & 3, tn = tile >> 2;
    const int gt = dst * 16 + tile;
    const int m0 = tm * BM, nc0 = tn * BN;
    const int nst = isB ? NKT - KA : KA, kbase = isB ? KA : 0;
    char* myws = (char*)args.myws;
    GAS unsigned* tmo = (GAS unsigned*)(myws + WS_TMO);
    GAS unsigned* pflag = (GAS unsigned*)(myws + WS_PFLAG + gt * 512);
    char* part = myws + WS_PART + (size_t)gt * 65536;
    LAS unsigned* ctrl = (LAS unsigned*)(lds + CTRL);
    STAMP(0);

    f32x4 v[2][4];
    const int orow = wid * 16 + (lane >> 3), ocol = (lane & 7) * 16;
    unsigned call;

    if (__builtin_expect(wid >= 4, 1)) {
        const int lw = wid - 4;
        __amdgpu_buffer_rsrc_t rx = __builtin_amdgcn_make_buffer_rsrc((void*)x, 0, MLOC * KD * 2, 0x00020000);
        __amdgpu_buffer_rsrc_t rw = __builtin_amdgcn_make_buffer_rsrc((void*)w, 0, KD * ND * 2, 0x00020000);
        unsigned vx[4], vw[4];
#pragma unroll
        for (int i = 0; i < 4; ++i) {
            const int row = 32 * lw + 8 * i + (lane >> 3);
            vx[i] = (unsigned)((m0 + row) * (KD * 2) + (((lane & 7) ^ ((row >> 1) & 7)) << 4));
            const int k = 16 * lw + 4 * i + (lane >> 4), f = ((k & 3) << 1) | (k & 8);
            vw[i] = (unsigned)(k * (ND * 2) + (dst * NSH + nc0) * 2 + (((lane & 15) ^ f) << 4));
        }
        const unsigned lx = ldsb + lw * 4096, lwb = ldsb + XBYTES + lw * 4096;
        int ktab;
        {
            const int ph = (dst + 2 * isB) & 3, rot = (tm + tn) & 3, tu = lane / UNIT, nu = nst / UNIT;
            const int bs = tu < ph ? 0 : ph + ((tu - ph) & ~3);
            const int sz = tu < ph ? ph : max(1, min(4, nu - bs));
            ktab = kbase + (bs + (tu - bs + rot) % sz) * UNIT + lane % UNIT;
        }
#define ISSUE(t) do { \
            const unsigned kt_ = __builtin_amdgcn_readlane(ktab, t), so_ = ((t) & 3) * SLOT; \
            const unsigned sx_ = kt_ * (BK * 2), sw_ = kt_ * (BK * ND * 2); \
            _Pragma("unroll") for (int i = 0; i < 4; ++i) { \
                GLDS(rw, lwb + so_ + i * 1024, vw[i], sw_); GLDS(rx, lx + so_ + i * 1024, vx[i], sx_); } } while (0)
        asm volatile("s_nop 4" ::: "memory");
        ISSUE(0); ISSUE(1);
        WAIT_VM(8); BARRIER();
        STAMP(1);
        ISSUE(2);
        WAIT_VM(8); BARRIER();
        unsigned cv, fl = 0;
        asm volatile("global_load_dword %0, %1, off sc1" : "=v"(cv) : "v"(myws + WS_CNT + b * 4) : "memory");
        ISSUE(3);
        for (int t = 0; t < nst - NS; ++t) {
            ISSUE(t + NS);
            if (t == nst - NS - 2) asm volatile("global_load_dword %0, %1, off sc0 sc1" : "=v"(fl) : "v"(pflag + lw * 32) : "memory");
            WAIT_VM(16); BARRIER();
        }
        WAIT_VM(8);
        asm volatile("" : "+v"(fl), "+v"(cv));
        call = rfl(cv) + 1;
        if (!isB) {
            BARRIER(); WAIT_VM(0); BARRIER(); BARRIER();
            STAMP(2);
            if (lw == 0) {
                if (lane == 0) __hip_atomic_store((GAS unsigned*)(myws + WS_CNT) + b, call, __ATOMIC_RELAXED, __HIP_MEMORY_SCOPE_AGENT);
                if (dst != rank && wait_flag((GAS unsigned*)(myws + WS_DONE + gt * 128), tmo, call) && lane == 0) {
                    GAS unsigned char* po = (GAS unsigned char*)args.myout + (size_t)(dst * MLOC + m0) * NSH + nc0;
                    __hip_atomic_store(po, (unsigned char)0x7f, __ATOMIC_RELAXED, __HIP_MEMORY_SCOPE_SYSTEM);
                }
                STAMP(6);
            }
            STAMP_OUT();
            return;
        }
        unsigned bad = 0;
        if (rfl(fl) < call) bad = wait_flag(pflag + lw * 32, tmo, call);
        STAMP(7);
        {
            __amdgpu_buffer_rsrc_t rp = __builtin_amdgcn_make_buffer_rsrc((void*)part, 0, 65536, 0x00020000);
            const unsigned vp = lane * 16;
            asm volatile("s_nop 4" ::: "memory");
#pragma unroll
            for (int i = 0; i < 8; ++i) GLDS_SYS(rp, ldsb + lw * 8192 + i * 1024, vp, lw * 16384 + i * 1024);
            BARRIER();
#pragma unroll
            for (int i = 0; i < 8; ++i) GLDS_SYS(rp, ldsb + SLOT + lw * 8192 + i * 1024, vp, lw * 16384 + 8192 + i * 1024);
            WAIT_VM(16); BARRIER();
            WAIT_VM(0); BARRIER();
        }
        STAMP(2);
        BARRIER();
        BARRIER();
#pragma unroll
        for (int h = 0; h < 2; ++h)
#pragma unroll
            for (int i = 0; i < 4; ++i) v[h][i] = *(LAS f32x4*)(lds + (orow + 8 * h) * TROW + (ocol + 4 * i) * 4);
        const char* g = myws + WS_AMAX + (call & 1) * 8192 + lw * 2048 + lane * 16;
        unsigned mx = 0;
        const unsigned long long t0 = __builtin_amdgcn_s_memrealtime();
        for (;;) {
            u32x4 a, c;
            asm volatile("global_load_dwordx4 %0, %2, off sc0 sc1\n\t"
                         "global_load_dwordx4 %1, %2, off offset:1024 sc0 sc1\n\t"
                         "s_waitcnt vmcnt(0)"
                         : "=&v"(a), "=&v"(c) : "v"(g) : "memory");
            const bool ok = a.y == call && a.w == call && c.y == call && c.w == call;
            mx = max(max(a.x, a.z), max(c.x, c.z));
            if (__all(ok)) break;
            if (rfl(__hip_atomic_load(tmo, __ATOMIC_RELAXED, __HIP_MEMORY_SCOPE_AGENT)) == call ||
                __builtin_amdgcn_s_memrealtime() - t0 > DEADLINE) {
                __hip_atomic_store(tmo, call, __ATOMIC_RELAXED, __HIP_MEMORY_SCOPE_AGENT);
                bad = 1;
                break;
            }
            __builtin_amdgcn_s_sleep(1);
        }
        mx = wave_max_u32(mx);
        if (lane == 0) { ctrl[8 + lw] = mx; ctrl[12 + lw] = bad; }
        STAMP(3);
    } else {
        const int wm = wid >> 1, wn = wid & 1;
        const int fr = lane & 15, fq = lane >> 4, q = (lane & 15) >> 2, p = lane & 3;
        unsigned callv = __hip_atomic_load((GAS unsigned*)(myws + WS_CNT) + b, __ATOMIC_RELAXED, __HIP_MEMORY_SCOPE_AGENT);
        char* peerws = (char*)args.ws[lane & 7];
        unsigned ax[2][2], aw[2][4];
#pragma unroll
        for (int ks = 0; ks < 2; ++ks) {
            const int row = wm * 64 + fr;
            ax[0][ks] = ldsb + row * 128 + (((ks * 4 + fq) ^ ((row >> 1) & 7)) << 4);
            ax[1][ks] = ax[0][ks] + 2 * SLOT;
            asm volatile("" : "+v"(ax[0][ks]), "+v"(ax[1][ks]));
        }
#pragma unroll
        for (int c = 0; c < 4; ++c) {
            const int k = 8 * fq + q, f = ((k & 3) << 1) | (k & 8);
            aw[0][c] = ldsb + XBYTES + k * 256 + (((wn * 8 + 2 * c + (p >> 1)) ^ f) << 4) + (p & 1) * 8;
            aw[1][c] = aw[0][c] + 2 * SLOT;
            asm volatile("" : "+v"(aw[0][c]), "+v"(aw[1][c]));
        }
        f32x4 acc[16];
#pragma unroll
        for (int i = 0; i < 16; ++i) acc[i] = (f32x4){0.f, 0.f, 0.f, 0.f};
        Frag f0, f1;
        BARRIER();
        rd_all<0, 0>(f0, ax, aw);
        WAIT_LGKM0(); BARRIER();
        asm volatile("s_nop 1");
        for (int t = 0; t < nst - 4; t += 4) {
            kstep<1, 0>(acc, f0, f1, ax, aw); kstep<2, 0>(acc, f1, f0, ax, aw);
            kstep<3, 0>(acc, f0, f1, ax, aw); kstep<0, 0>(acc, f1, f0, ax, aw);
        }
        kstep<1, 0>(acc, f0, f1, ax, aw); kstep<2, 0>(acc, f1, f0, ax, aw); kstep<3, 0>(acc, f0, f1, ax, aw);
        f32x4 pa[16];
        if (isB) {
#pragma unroll
            for (int i = 0; i < 16; ++i) pa[i] = *(LAS f32x4*)(lds + (i < 8 ? 0 : SLOT) + wid * 8192 + (i & 7) * 1024 + lane * 16);
        } else {
#pragma unroll
            for (int i = 0; i < 16; ++i) pa[i] = (f32x4){0.f, 0.f, 0.f, 0.f};
        }
        SB();
        kstep<-1, 0>(acc, f1, f0, ax, aw);
        asm volatile("s_nop 15" ::: "memory");
#pragma unroll
        for (int i = 0; i < 16; ++i) asm volatile("" : "+v"(acc[i]));
        STAMP(2);
        call = rfl(callv) + 1;
        if (!isB) {
            char* pp = part + wid * 16384 + lane * 16;
#pragma unroll
            for (int i = 0; i < 16; ++i)
                asm volatile("global_store_dwordx4 %0, %1, off offset:%2 sc0 sc1\n\ts_nop 1"
                             :: "v"(pp + (i >> 2) * 4096), "v"(acc[i]), "n"((i & 3) * 1024) : "memory");
            WAIT_VM(0);
            if (lane == 0) __hip_atomic_store(pflag + wid * 32, call, __ATOMIC_RELAXED, __HIP_MEMORY_SCOPE_SYSTEM);
            STAMP(3);
            STAMP_OUT();
            return;
        }
        float mxf = 0.f;
#pragma unroll
        for (int i = 0; i < 16; ++i) {
            acc[i] += pa[i];
            mxf = fmaxf(fmaxf(fmaxf(acc[i][0], acc[i][1]), fmaxf(acc[i][2], acc[i][3])), mxf);
        }
        const unsigned wmx = wave_max_u32(__float_as_uint(mxf) & 0x7fffffffu);
        if (lane == 0) ctrl[wid] = wmx;
        WAIT_LGKM0(); BARRIER();
        if (wid == 0) {
            const u32x4 m4 = *(LAS u32x4*)ctrl;
            const unsigned tmx = max(max(m4.x, m4.y), max(m4.z, m4.w));
            if (lane < NR) {
                char* gp = peerws + WS_AMAX + (call & 1) * 8192 + (rank * NTILE + gt) * 8;
                u32x2 gr = {tmx, call};
                asm volatile("global_store_dwordx2 %0, %1, off sc0 sc1\n\ts_nop 1" :: "v"(gp), "v"(gr) : "memory");
            }
            if (lane == 0)
                __hip_atomic_store((GAS unsigned*)(myws + WS_CNT) + b, call, __ATOMIC_RELAXED, __HIP_MEMORY_SCOPE_AGENT);
        }
        STAMP(3);
#pragma unroll
        for (int i = 0; i < 16; ++i)
            *(LAS f32x4*)(lds + (wm * 64 + (i >> 2) * 16 + fr) * TROW + (wn * 64 + (i & 3) * 16 + fq * 4) * 4) = acc[i];
        WAIT_LGKM0(); BARRIER();
#pragma unroll
        for (int h = 0; h < 2; ++h)
#pragma unroll
            for (int i = 0; i < 4; ++i) v[h][i] = *(LAS f32x4*)(lds + (orow + 8 * h) * TROW + (ocol + 4 * i) * 4);
    }
    char* optr = (char*)args.out[dst] + (size_t)(rank * MLOC + m0 + orow) * NSH + nc0 + ocol;
    GAS unsigned* df = (GAS unsigned*)((char*)args.ws[dst] + WS_DONE + (rank * 16 + tile) * 128);
    WAIT_LGKM0(); BARRIER();
    STAMP(4);
    const u32x4 g4 = *(LAS u32x4*)(ctrl + 8), b4 = *(LAS u32x4*)(ctrl + 12);
    const float amax = __uint_as_float(max(max(g4.x, g4.y), max(g4.z, g4.w)));
    const bool timed_out = (b4.x | b4.y | b4.z | b4.w) != 0;
    const float scale = amax / 448.0f;
    const float inv = amax > 0.f ? 1.0f / scale : 0.f;
#pragma unroll
    for (int h = 0; h < 2; ++h) {
        u32x4 o;
#pragma unroll
        for (int i = 0; i < 4; ++i) {
            int r = __builtin_amdgcn_cvt_pk_fp8_f32(__builtin_amdgcn_fmed3f(v[h][i][0] * inv, 0.f, 448.f),
                                                    __builtin_amdgcn_fmed3f(v[h][i][1] * inv, 0.f, 448.f), 0, false);
            r = __builtin_amdgcn_cvt_pk_fp8_f32(__builtin_amdgcn_fmed3f(v[h][i][2] * inv, 0.f, 448.f),
                                                __builtin_amdgcn_fmed3f(v[h][i][3] * inv, 0.f, 448.f), r, true);
            o[i] = timed_out ? 0x7f7f7f7fu : (unsigned)r;
        }
        asm volatile("global_store_dwordx4 %0, %1, off sc0 sc1\n\ts_nop 1" :: "v"(optr + h * 8 * NSH), "v"(o) : "memory");
    }
    WAIT_VM(0);
    BARRIER();
    if (tid == 0 && dst != rank) __hip_atomic_store(df, call, __ATOMIC_RELAXED, __HIP_MEMORY_SCOPE_SYSTEM);
    STAMP(5);
    STAMP_OUT();
}

}

extern "C" void kernel_launch(int rank, int nranks, ncclComm_t comm,
                              hipStream_t stream,
                              void* const* d_in, const int* in_sizes, int n_in,
                              void* d_out, int out_size,
                              const int* in_dtypes, int out_dtype,
                              void* const* d_out_all,
                              void* const* d_ws_all, size_t ws_bytes) {
    if (nranks != NR || n_in < 2 || in_sizes[0] != MLOC * KD || in_sizes[1] != KD * ND ||
        out_size != ND * NSH || ws_bytes < WS_END || (unsigned)rank >= (unsigned)NR) {
        fprintf(stderr, "kernel_launch: unexpected problem shape\n");
        return;
    }
    HIP_CHECK(hipFuncSetAttribute((const void*)gemm_a2a_kernel, hipFuncAttributeMaxDynamicSharedMemorySize, LDS_BYTES));
    Args a;
    a.x = d_in[0]; a.w = d_in[1]; a.myws = d_ws_all[rank]; a.myout = d_out; a.rank = rank; a.pad = 0;
    for (int p = 0; p < NR; ++p) { a.out[p] = d_out_all[p]; a.ws[p] = d_ws_all[p]; }
    hipLaunchKernelGGL(gemm_a2a_kernel, dim3(NWG), dim3(512), LDS_BYTES, stream, a);
    HIP_CHECK(hipGetLastError());
}
```

```cpp
#include <hip/hip_runtime.h>
#include <rccl/rccl.h>
#include <cstdio>
#include <cstdint>
#include <cstddef>

#define HIP_CHECK(expr) do { hipError_t _e = (expr); if (_e != hipSuccess) { \
    fprintf(stderr, "HIP error %s at %s:%d\n", hipGetErrorString(_e), __FILE__, __LINE__); \
    return; } } while (0)

namespace {

typedef short          s16x4 __attribute__((ext_vector_type(4)));
typedef short          s16x8 __attribute__((ext_vector_type(8)));
typedef float          f32x4 __attribute__((ext_vector_type(4)));
typedef unsigned       u32x4 __attribute__((ext_vector_type(4)));
typedef unsigned       u32x2 __attribute__((ext_vector_type(2)));
#define LAS __attribute__((address_space(3)))
#define GAS __attribute__((address_space(1)))

constexpr int NR = 8;
constexpr int MLOC = 512, KD = 4096, ND = 4096, NSH = 512;
constexpr int BM = 128, BN = 128, BK = 64;
constexpr int NKT = KD / BK;
#define KA_STEPS 28
constexpr int KA = KA_STEPS;
#define UNIT 1
constexpr int XBYTES = BM * BK * 2;
constexpr int SLOT = XBYTES + BK * BN * 2;
constexpr int NS = 4;
constexpr int CTRL = NS * SLOT;
constexpr int LDS_BYTES = CTRL + 256;
constexpr int NWG = 256, NTILE = 128;
constexpr int CSTR = 8192;
static_assert(KA % 4 == 0 && KA >= 8 && NKT - KA >= 8, "");

constexpr size_t WS_CNT   = 0;
constexpr size_t WS_TMO   = 4096;
constexpr size_t WS_AMAX  = 8192;
constexpr size_t WS_GO    = WS_AMAX + 2 * 8192;
constexpr size_t WS_PFLAG = WS_GO + NTILE * 128;
constexpr size_t WS_MASK  = 1 << 20;
constexpr size_t WS_DATA  = 2 << 20;
constexpr size_t WS_PART  = 4 << 20;
constexpr size_t WS_END   = WS_PART + (size_t)NTILE * 65536;
static_assert(WS_PFLAG + NTILE * 512 <= WS_MASK, "");
#define POLL_SLEEP 1
#define OUT_POL "sc0 sc1"
#define MASK_SLEEP 2
constexpr unsigned long long DEADLINE = 100000000ull;

#define BARRIER() do { asm volatile("" ::: "memory"); __builtin_amdgcn_s_barrier(); asm volatile("" ::: "memory"); } while (0)
#define WAIT_VM(n) asm volatile("s_waitcnt vmcnt(%0)" :: "n"(n) : "memory")
#define WAIT_LGKM0() __builtin_amdgcn_s_waitcnt(0xC07F)
#define SB() __builtin_amdgcn_sched_barrier(0)

__device__ __forceinline__ unsigned rfl(unsigned v) { return __builtin_amdgcn_readfirstlane(v); }

__device__ __forceinline__ unsigned wave_max_u32(unsigned v) {
    v = max(v, (unsigned)__builtin_amdgcn_update_dpp(0, (int)v, 0xB1, 0xF, 0xF, true));
    v = max(v, (unsigned)__builtin_amdgcn_update_dpp(0, (int)v, 0x4E, 0xF, 0xF, true));
    v = max(v, (unsigned)__builtin_amdgcn_update_dpp(0, (int)v, 0x141, 0xF, 0xF, true));
    v = max(v, (unsigned)__builtin_amdgcn_update_dpp(0, (int)v, 0x140, 0xF, 0xF, true));
    unsigned a = __builtin_amdgcn_readlane(v, 0), b = __builtin_amdgcn_readlane(v, 16);
    unsigned c = __builtin_amdgcn_readlane(v, 32), d = __builtin_amdgcn_readlane(v, 48);
    return max(max(a, b), max(c, d));
}

__device__ __forceinline__ unsigned wave_scan_u32(unsigned v) {
    v += (unsigned)__builtin_amdgcn_update_dpp(0, (int)v, 0x111, 0xF, 0xF, true);
    v += (unsigned)__builtin_amdgcn_update_dpp(0, (int)v, 0x112, 0xF, 0xF, true);
    v += (unsigned)__builtin_amdgcn_update_dpp(0, (int)v, 0x114, 0xF, 0xF, true);
    v += (unsigned)__builtin_amdgcn_update_dpp(0, (int)v, 0x118, 0xF, 0xF, true);
    v += (unsigned)__builtin_amdgcn_update_dpp(0, (int)v, 0x142, 0xA, 0xF, false);
    v += (unsigned)__builtin_amdgcn_update_dpp(0, (int)v, 0x143, 0xC, 0xF, false);
    return v;
}
__device__ __forceinline__ unsigned stream_offsets(unsigned m, unsigned (&pos)[8]) {
    unsigned c = m - ((m >> 1) & 0x55555555u);
    c = (c & 0x33333333u) + ((c >> 2) & 0x33333333u);
    unsigned ex[4], tot[4];
#pragma unroll
    for (int j = 0; j < 4; ++j) {
        const unsigned f = (c >> (4 * j)) & 0x000f000fu, inc = wave_scan_u32(f);
        tot[j] = __builtin_amdgcn_readlane(inc, 63); ex[j] = inc - f;
    }
    unsigned base = 0;
#pragma unroll
    for (int g = 0; g < 8; ++g) {
        pos[g] = base + (g < 4 ? ex[g] & 0xffffu : ex[g - 4] >> 16);
        base += g < 4 ? tot[g] & 0xffffu : tot[g - 4] >> 16;
    }
    return base;
}
__device__ __forceinline__ unsigned tag_bits(unsigned n) { return ((n * 0x00204081u) & 0x01010101u) << 7; }

__device__ __forceinline__ bool give_up(GAS unsigned* tmo, unsigned call, unsigned long long t0, unsigned& polls) {
    if (__builtin_expect((++polls & 31u) != 0u, 1)) return false;
    if (rfl(__hip_atomic_load(tmo, __ATOMIC_RELAXED, __HIP_MEMORY_SCOPE_AGENT)) == call ||
        __builtin_amdgcn_s_memrealtime() - t0 > DEADLINE) {
        __hip_atomic_store(tmo, call, __ATOMIC_RELAXED, __HIP_MEMORY_SCOPE_AGENT);
        return true;
    }
    return false;
}

template <int SLEEP>
__device__ __forceinline__ bool wait_flag(GAS unsigned* flag, GAS unsigned* tmo, unsigned call) {
    const unsigned long long t0 = __builtin_amdgcn_s_memrealtime();
    unsigned polls = 0;
    while (rfl(__hip_atomic_load(flag, __ATOMIC_RELAXED, __HIP_MEMORY_SCOPE_SYSTEM)) < call) {
        if (give_up(tmo, call, t0, polls)) return true;
        __builtin_amdgcn_s_sleep(SLEEP);
    }
    return false;
}

#define GLDS(rsrc, ldsaddr, voff, soff) \
    asm volatile("s_mov_b32 m0, %0\n\ts_nop 0\n\tbuffer_load_dwordx4 %1, %2, %3 offen lds" \
                 :: "s"(ldsaddr), "v"(voff), "s"(rsrc), "s"(soff) : "memory")
#define GLDS_SYS(rsrc, ldsaddr, voff, soff) \
    asm volatile("s_mov_b32 m0, %0\n\ts_nop 0\n\tbuffer_load_dwordx4 %1, %2, %3 offen sc0 sc1 lds" \
                 :: "s"(ldsaddr), "v"(voff), "s"(rsrc), "s"(soff) : "memory")

#define STAMP(i) do {} while (0)
#define STAMP_OUT() do {} while (0)

struct Frag { s16x8 x[8]; s16x4 w[16]; };

template <int SL, int R>
__device__ __forceinline__ void rd(Frag& f, const unsigned (&ax)[2][2], const unsigned (&aw)[2][4]) {
    constexpr int g = R / 3, e = R % 3, so = (SL & 1) * SLOT, ks = g >> 2, i = g & 3;
    if constexpr (e == 0) f.x[g] = *(LAS s16x8*)(uintptr_t)(ax[SL >> 1][ks] + so + i * 2048);
    else f.w[g * 2 + e - 1] = __builtin_amdgcn_ds_read_tr16_b64_v4i16(
             (LAS s16x4*)(uintptr_t)(aw[SL >> 1][i] + so + ks * 8192 + (e - 1) * 1024));
}
template <int SL, int R>
__device__ __forceinline__ void rd_all(Frag& f, const unsigned (&ax)[2][2], const unsigned (&aw)[2][4]) {
    if constexpr (R < 24) { rd<SL, R>(f, ax, aw); rd_all<SL, R + 1>(f, ax, aw); }
}
template <int SL, int G>
__device__ __forceinline__ void kstep(f32x4 (&acc)[16], const Frag& cur, Frag& nxt,
                                      const unsigned (&ax)[2][2], const unsigned (&aw)[2][4]) {
    if constexpr (G < 32) {
        constexpr int ks = G >> 4, c = (G >> 2) & 3, mi = G & 3;
        const s16x8 wf = __builtin_shufflevector(cur.w[(ks * 4 + c) * 2], cur.w[(ks * 4 + c) * 2 + 1], 0, 1, 2, 3, 4, 5, 6, 7);
        asm volatile("v_mfma_f32_16x16x32_bf16 %0, %1, %2, %0" : "+v"(acc[mi * 4 + c]) : "v"(wf), "v"(cur.x[ks * 4 + mi]));
        if constexpr (SL >= 0 && G < 24) rd<SL, G>(nxt, ax, aw);
        if constexpr (SL >= 0 && G == 31) { SB(); WAIT_LGKM0(); BARRIER(); }
        SB();
        kstep<SL, G + 1>(acc, cur, nxt, ax, aw);
    }
}

struct Args {
    const void* x; const void* w;
    void* myws; void* myout;
    void* out[NR]; void* ws[NR];
    int rank, pad;
};

__global__ void __launch_bounds__(512, 2)
gemm_a2a_kernel(const Args args)
{
    const void* x = args.x; const void* w = args.w;
    const int rank = args.rank;
    asm volatile("" :: "s"(x), "s"(w));
    extern __shared__ __attribute__((aligned(16))) char smem[];
    LAS char* lds = (LAS char*)smem;
    const unsigned ldsb = (unsigned)(uintptr_t)lds;
    const int tid = threadIdx.x, lane = tid & 63;
    const int wid = rfl(tid >> 6);
    const int b = blockIdx.x;
    const int dst = b & 7;
#define TX 0
    const int isB = (b >> 3) & 1, tile = (b >> 4) ^ TX, tm = tile & 3, tn = tile >> 2;
    const int gt = dst * 16 + tile;
    const int m0 = tm * BM;
#define COLMAP 1
    const int nc0 = COLMAP == 0 ? tn * 128 : COLMAP == 1 ? tn * 64 : ((tn & 1) + (tn & 2) * 2) * 64;
    constexpr int NCH = COLMAP == 0 ? 64 : COLMAP == 1 ? 256 : 128;
    const int nst = isB ? NKT - KA : KA, kbase = isB ? KA : 0;
    char* myws = (char*)args.myws;
    GAS unsigned* tmo = (GAS unsigned*)(myws + WS_TMO);
    GAS unsigned* pflag = (GAS unsigned*)(myws + WS_PFLAG + gt * 512);
    char* part = myws + WS_PART + (size_t)gt * 65536;
    LAS unsigned* ctrl = (LAS unsigned*)(lds + CTRL);
    STAMP(0);

    f32x4 v[2][4];
    unsigned call, total = 0;
#define FETCH() do { \
        total = rfl(ctrl[16 + wid]); \
        const unsigned ca_ = ldsb + wid * CSTR + lane * 64, sw_ = (lane << 2) & 48; \
        _Pragma("unroll") for (int i = 0; i < 4; ++i) v[0][i] = *(LAS f32x4*)(uintptr_t)(ca_ + ((i * 16) ^ sw_)); \
        if (total > 1024u) { \
            _Pragma("unroll") for (int i = 0; i < 4; ++i) v[1][i] = *(LAS f32x4*)(uintptr_t)(ca_ + 4096 + ((i * 16) ^ sw_)); } \
    } while (0)

    if (__builtin_expect(wid >= 4, 1)) {
        const int lw = wid - 4;
        __amdgpu_buffer_rsrc_t rx = __builtin_amdgcn_make_buffer_rsrc((void*)x, 0, MLOC * KD * 2, 0x00020000);
        __amdgpu_buffer_rsrc_t rw = __builtin_amdgcn_make_buffer_rsrc((void*)w, 0, KD * ND * 2, 0x00020000);
        unsigned vx[4], vw[4];
#pragma unroll
        for (int i = 0; i < 4; ++i) {
            const int row = 32 * lw + 8 * i + (lane >> 3);
            vx[i] = (unsigned)((m0 + row) * (KD * 2) + (((lane & 7) ^ ((row >> 1) & 7)) << 4));
            const int k = 16 * lw + 4 * i + (lane >> 4), f = ((k & 3) << 1) | (k & 8);
            const int ch = (lane & 15) ^ f;
            vw[i] = (unsigned)(k * (ND * 2) + (dst * NSH + nc0 + (ch >> 3) * NCH) * 2 + ((ch & 7) << 4));
        }
        const unsigned lx = ldsb + lw * 4096, lwb = ldsb + XBYTES + lw * 4096;
        int ktab;
        {
            const int ph = (dst + 2 * isB) & 3, rot = (tm + tn) & 3, tu = lane / UNIT, nu = nst / UNIT;
            const int bs = tu < ph ? 0 : ph + ((tu - ph) & ~3);
            const int sz = tu < ph ? ph : max(1, min(4, nu - bs));
            ktab = kbase + (bs + (tu - bs + rot) % sz) * UNIT + lane % UNIT;
        }
#define ISSUE(t) do { \
            const unsigned kt_ = __builtin_amdgcn_readlane(ktab, t), so_ = ((t) & 3) * SLOT; \
            const unsigned sx_ = kt_ * (BK * 2), sw_ = kt_ * (BK * ND * 2); \
            _Pragma("unroll") for (int i = 0; i < 4; ++i) { \
                GLDS(rw, lwb + so_ + i * 1024, vw[i], sw_); GLDS(rx, lx + so_ + i * 1024, vx[i], sx_); } } while (0)
        asm volatile("s_nop 4" ::: "memory");
        ISSUE(0); ISSUE(1);
        WAIT_VM(8); BARRIER();
        if (!isB) STAMP(1);
        ISSUE(2);
        WAIT_VM(8); BARRIER();
        unsigned cv, fl = 0;
        asm volatile("global_load_dword %0, %1, off sc1" : "=v"(cv) : "v"(myws + WS_CNT + b * 4) : "memory");
        char* peerws;
        asm volatile("global_load_dwordx2 %0, %1, off" : "=v"(peerws)
                     : "v"((const char*)__builtin_amdgcn_kernarg_segment_ptr() + offsetof(Args, ws) + (lane & 7) * 8) : "memory");
        ISSUE(3);
        for (int t = 0; t < nst - NS; ++t) {
            ISSUE(t + NS);
            if (t == nst - NS - 2) asm volatile("global_load_dword %0, %1, off sc0 sc1" : "=v"(fl) : "v"(pflag + lw * 32) : "memory");
            WAIT_VM(16); BARRIER();
        }
        WAIT_VM(8);
        asm volatile("" : "+v"(fl), "+v"(cv), "+v"(peerws));
        call = rfl(cv) + 1;
        char* gp = peerws + WS_AMAX + (call & 1) * 8192 + (rank * NTILE + gt) * 8;
        asm volatile("" : "+v"(gp));
        if (!isB) {
            BARRIER(); WAIT_VM(0); BARRIER(); BARRIER();
            STAMP(2);
        } else {
        unsigned bad = 0;
        if (rfl(fl) < call) bad = wait_flag<1>(pflag + lw * 32, tmo, call);
        STAMP(7);
        {
            __amdgpu_buffer_rsrc_t rp = __builtin_amdgcn_make_buffer_rsrc((void*)part, 0, 65536, 0x00020000);
            const unsigned vp = lane * 16;
            asm volatile("s_nop 4" ::: "memory");
#pragma unroll
            for (int i = 0; i < 8; ++i) GLDS_SYS(rp, ldsb + lw * 8192 + i * 1024, vp, lw * 16384 + i * 1024);
            BARRIER();
#pragma unroll
            for (int i = 0; i < 8; ++i) GLDS_SYS(rp, ldsb + SLOT + lw * 8192 + i * 1024, vp, lw * 16384 + 8192 + i * 1024);
            WAIT_VM(16); BARRIER();
            WAIT_VM(0); BARRIER();
        }
        STAMP(2);
        BARRIER();
        if (lw == 0) {
            const u32x4 m4 = *(LAS u32x4*)ctrl;
            const u32x2 gr = {max(max(m4.x, m4.y), max(m4.z, m4.w)), call};
            if (lane < NR) asm volatile("global_store_dwordx2 %0, %1, off sc0 sc1\n\ts_nop 1" :: "v"(gp), "v"(gr) : "memory");
            STAMP(5);
        }
        BARRIER();
        STAMP(1);
        FETCH();
        STAMP(6);
        const char* g = myws + WS_AMAX + (call & 1) * 8192 + lw * 2048 + lane * 16;
        unsigned mx = 0;
        const unsigned long long t0 = __builtin_amdgcn_s_memrealtime();
    unsigned polls = 0;
        for (;;) {
            u32x4 a, c;
            asm volatile("global_load_dwordx4 %0, %2, off sc0 sc1\n\t"
                         "global_load_dwordx4 %1, %2, off offset:1024 sc0 sc1\n\t"
                         "s_waitcnt vmcnt(0)"
                         : "=&v"(a), "=&v"(c) : "v"(g) : "memory");
            const bool ok = a.y == call && a.w == call && c.y == call && c.w == call;
            mx = max(max(a.x, a.z), max(c.x, c.z));
            if (__all(ok)) break;
            if (give_up(tmo, call, t0, polls)) { bad = 1; break; }
            __builtin_amdgcn_s_sleep(1);
        }
        mx = wave_max_u32(mx);
        if (lane == 0) { ctrl[8 + lw] = mx; ctrl[12 + lw] = bad; }
        STAMP(3);
        }
    } else {
        const int wm = wid >> 1, wn = wid & 1;
        const int fr = lane & 15, fq = lane >> 4, q = (lane & 15) >> 2, p = lane & 3;
        unsigned callv = __hip_atomic_load((GAS unsigned*)(myws + WS_CNT) + b, __ATOMIC_RELAXED, __HIP_MEMORY_SCOPE_AGENT);
        char* mptr = (char*)args.ws[dst] + WS_MASK + (((rank * 16 + tile) * 8 + wid * 2) * 64 + lane) * 8;
        unsigned ax[2][2], aw[2][4];
#pragma unroll
        for (int ks = 0; ks < 2; ++ks) {
            const int row = wm * 64 + fr;
            ax[0][ks] = ldsb + row * 128 + (((ks * 4 + fq) ^ ((row >> 1) & 7)) << 4);
            ax[1][ks] = ax[0][ks] + 2 * SLOT;
            asm volatile("" : "+v"(ax[0][ks]), "+v"(ax[1][ks]));
        }
#pragma unroll
        for (int c = 0; c < 4; ++c) {
            const int k = 8 * fq + q, f = ((k & 3) << 1) | (k & 8);
            aw[0][c] = ldsb + XBYTES + k * 256 + (((wn * 8 + 2 * c + (p >> 1)) ^ f) << 4) + (p & 1) * 8;
            aw[1][c] = aw[0][c] + 2 * SLOT;
            asm volatile("" : "+v"(aw[0][c]), "+v"(aw[1][c]));
        }
        f32x4 acc[16];
#pragma unroll
        for (int i = 0; i < 16; ++i) acc[i] = (f32x4){0.f, 0.f, 0.f, 0.f};
        asm volatile("" : "+v"(mptr));
        Frag f0, f1;
        BARRIER();
        rd_all<0, 0>(f0, ax, aw);
        WAIT_LGKM0(); BARRIER();
        asm volatile("s_nop 1");
        for (int t = 0; t < nst - 4; t += 4) {
            kstep<1, 0>(acc, f0, f1, ax, aw); kstep<2, 0>(acc, f1, f0, ax, aw);
            kstep<3, 0>(acc, f0, f1, ax, aw); kstep<0, 0>(acc, f1, f0, ax, aw);
        }
        call = rfl(callv) + 1;
        SB();
        kstep<1, 0>(acc, f0, f1, ax, aw); kstep<2, 0>(acc, f1, f0, ax, aw); kstep<3, 0>(acc, f0, f1, ax, aw);
        f32x4 pa[16];
        if (isB) {
#pragma unroll
            for (int i = 0; i < 16; ++i) pa[i] = *(LAS f32x4*)(lds + (i < 8 ? 0 : SLOT) + wid * 8192 + (i & 7) * 1024 + lane * 16);
        } else {
#pragma unroll
            for (int i = 0; i < 16; ++i) pa[i] = (f32x4){0.f, 0.f, 0.f, 0.f};
        }
        SB();
        kstep<-1, 0>(acc, f1, f0, ax, aw);
        asm volatile("s_nop 15" ::: "memory");
#pragma unroll
        for (int i = 0; i < 16; ++i) asm volatile("" : "+v"(acc[i]));
        STAMP(2);
        if (!isB) {
            char* pp = part + wid * 16384 + lane * 16;
#pragma unroll
            for (int i = 0; i < 16; ++i)
                asm volatile("global_store_dwordx4 %0, %1, off offset:%2 sc0 sc1\n\ts_nop 1"
                             :: "v"(pp + (i >> 2) * 4096), "v"(acc[i]), "n"((i & 3) * 1024) : "memory");
            WAIT_VM(0);
            if (lane == 0) __hip_atomic_store(pflag + wid * 32, call, __ATOMIC_RELAXED, __HIP_MEMORY_SCOPE_SYSTEM);
            STAMP(3);
        } else {
        float mx4[4] = {0.f, 0.f, 0.f, 0.f};
#pragma unroll
        for (int i = 0; i < 16; ++i) acc[i] += pa[i];
#pragma unroll
        for (int i = 0; i < 16; i += 2)
#pragma unroll
            for (int e = 0; e < 4; ++e) mx4[e] = __builtin_fmaxf(__builtin_fmaxf(acc[i][e], acc[i + 1][e]), mx4[e]);
        const float mxf = fmaxf(fmaxf(mx4[0], mx4[1]), fmaxf(mx4[2], mx4[3]));
        const unsigned wmx = wave_max_u32(__float_as_uint(mxf) & 0x7fffffffu);
        if (lane == 0) ctrl[wid] = wmx;
        WAIT_LGKM0(); BARRIER();
        STAMP(3);
        const unsigned trash = ldsb + 8 * CSTR + wid * 512 + lane * 4;
        unsigned mq[8];
#pragma unroll
        for (int c = 0; c < 8; ++c) mq[c] = __float_as_uint(acc[2 * c + 1][3]) >> 31;
#pragma unroll
        for (int e = 6; e >= 0; --e)
#pragma unroll
            for (int c = 0; c < 8; ++c) mq[c] = __builtin_amdgcn_alignbit(mq[c], __float_as_uint(acc[2 * c + (e >> 2)][e & 3]), 31);
        unsigned wa[16];
#pragma unroll
        for (int hh = 0; hh < 2; ++hh) {
            const unsigned m = ~(mq[hh * 4] | (mq[hh * 4 + 1] << 8) | (mq[hh * 4 + 2] << 16) | (mq[hh * 4 + 3] << 24));
            const u32x2 mg = {m, call};
            asm volatile("global_store_dwordx2 %0, %1, off offset:%2 sc0 sc1\n\ts_nop 1" :: "v"(mptr + (call & 1) * 524288), "v"(mg), "n"(hh * 512) : "memory");
            unsigned pos[8];
            const unsigned tot = stream_offsets(m, pos);
            if (lane == 0) ctrl[16 + wid * 2 + hh] = tot;
#pragma unroll
            for (int g = 0; g < 8; ++g) wa[hh * 8 + g] = ldsb + (wid * 2 + hh) * CSTR + pos[g] * 4;
        }
#pragma unroll
        for (int e = 0; e < 4; ++e) {
            unsigned sg[16], ad[16];
#pragma unroll
            for (int i = 0; i < 16; ++i) asm("v_ashrrev_i32 %0, 31, %1" : "=v"(sg[i]) : "v"(acc[i][e]));
#pragma unroll
            for (int i = 0; i < 16; ++i) asm("v_bfi_b32 %0, %1, %2, %3" : "=v"(ad[i]) : "v"(sg[i]), "v"(trash), "v"(wa[i]));
#pragma unroll
            for (int i = 0; i < 16; ++i) {
                *(LAS float*)(uintptr_t)(ad[i] + 4 * e) = acc[i][e];
                if (e < 3) asm("v_lshl_add_u32 %0, %1, 2, %0" : "+v"(wa[i]) : "v"(sg[i]));
            }
        }
        WAIT_LGKM0(); BARRIER();
        STAMP(1);
        FETCH();
        STAMP(6);
        }
    }
    const unsigned tag = call % 255u + 1u, tg0 = tag_bits(tag & 15u), tg1 = tag_bits(tag >> 4);
    if (isB) {
        char* dptr = (char*)args.ws[dst] + WS_DATA + ((rank * 16 + tile) * 8 + wid) * 2048 + lane * 16;
        asm volatile("" : "+v"(dptr));
        WAIT_LGKM0(); BARRIER();
        STAMP(4);
        if (tid == 0) __hip_atomic_store((GAS unsigned*)(myws + WS_GO + gt * 128), call, __ATOMIC_RELAXED, __HIP_MEMORY_SCOPE_AGENT);
        const u32x4 g4 = *(LAS u32x4*)(ctrl + 8), b4 = *(LAS u32x4*)(ctrl + 12);
        const float amax = __uint_as_float(max(max(g4.x, g4.y), max(g4.z, g4.w)));
        const bool timed_out = (b4.x | b4.y | b4.z | b4.w) != 0;
        const float scale = amax / 448.0f;
        const float inv = amax > 0.f ? 1.0f / scale : 0.f;
#pragma unroll
        for (int h = 0; h < 2; ++h) {
            if (h * 1024u < total) {
                u32x4 o;
#pragma unroll
                for (int i = 0; i < 4; ++i) {
                    int r = __builtin_amdgcn_cvt_pk_fp8_f32(fminf(v[h][i][0] * inv, 448.f), fminf(v[h][i][1] * inv, 448.f), 0, false);
                    r = __builtin_amdgcn_cvt_pk_fp8_f32(fminf(v[h][i][2] * inv, 448.f), fminf(v[h][i][3] * inv, 448.f), r, true);
                    o[i] = ((timed_out ? 0x7f7f7f7fu : (unsigned)r) & 0x7f7f7f7fu) | ((i & 1) ? tg1 : tg0);
                }
                if (h * 1024u + lane * 16u < total)
                    asm volatile("global_store_dwordx4 %0, %1, off offset:%2 sc0 sc1\n\ts_nop 1" :: "v"(dptr), "v"(o), "n"(h * 1024) : "memory");
            }
        }
        if (wid < 4) STAMP(5);
    } else {
        bool bad = false;
        unsigned m;
        {
            const char* mp = myws + WS_MASK + (call & 1) * 524288 + ((gt * 8 + wid) * 64 + lane) * 8;
            const unsigned long long t0 = __builtin_amdgcn_s_memrealtime();
    unsigned polls = 0;
            for (;;) {
                u32x2 g;
                asm volatile("global_load_dwordx2 %0, %1, off sc0 sc1\n\ts_waitcnt vmcnt(0)" : "=&v"(g) : "v"(mp) : "memory");
                m = g.x;
                if (__all(g.y == call)) break;
                if (give_up(tmo, call, t0, polls)) { bad = true; m = 0; break; }
                __builtin_amdgcn_s_sleep(MASK_SLEEP);
            }
        }
        STAMP(4);
        unsigned pos[8];
        const unsigned tot = stream_offsets(m, pos);
        const unsigned sbase = ldsb + wid * 8192;
#pragma unroll
        for (int g = 0; g < 8; ++g) *(LAS unsigned*)(uintptr_t)(sbase + 4096 + g * 256 + lane * 4) = pos[g] * 16 + ((m >> (4 * g)) & 15u);
        unsigned ra[8], rb[8], sel[8];
#pragma unroll
        for (int j = 0; j < 8; ++j)
            sel[j] = *(LAS unsigned*)(uintptr_t)(sbase + 4096 + ((j >> 2) * 4 + (lane & 3)) * 256 + ((j & 3) * 16 + (lane >> 2)) * 4);
#pragma unroll
        for (int j = 0; j < 8; ++j) {
            const unsigned e = sel[j], d0 = e >> 6, d1 = d0 + 1;
            const unsigned sp = ((e & 15u) * 0x00204081u) & 0x01010101u, mk = (sp << 8) - sp, s2 = sp + (sp << 8);
            ra[j] = sbase + (d0 ^ ((d0 >> 4) & 3u)) * 4;
            rb[j] = sbase + (d1 ^ ((d1 >> 4) & 3u)) * 4;
            sel[j] = ((((s2 + (sp << 16)) << 8) + __builtin_amdgcn_perm(0u, (e >> 4) & 3u, 0u)) & mk) | (0x0c0c0c0cu & ~mk);
        }
#pragma unroll
        for (int j = 0; j < 8; ++j) asm volatile("" : "+v"(ra[j]), "+v"(rb[j]), "+v"(sel[j]));
        char* optr = (char*)args.myout + (size_t)(dst * MLOC + m0 + (wid >> 2) * 64 + (wid & 1) * 32 + (lane >> 2)) * NSH
                     + nc0 + ((wid >> 1) & 1) * NCH + (lane & 3) * 16;
        char* sp = myws + WS_DATA + (gt * 8 + wid) * 2048 + lane * 16;
        asm volatile("" : "+v"(optr), "+v"(sp));
        if (!bad) bad = wait_flag<1>((GAS unsigned*)(myws + WS_GO + gt * 128), tmo, call);
        STAMP(5);
        const bool n0 = lane * 16u < tot, n1 = 1024u + lane * 16u < tot;
        u32x4 d0 = {0, 0, 0, 0}, d1 = {0, 0, 0, 0};
        if (!bad) {
            const unsigned long long t0 = __builtin_amdgcn_s_memrealtime();
    unsigned polls = 0;
            bool p0 = n0, p1 = n1;
            for (;;) {
                asm volatile("s_mov_b64 exec, %3\n\tglobal_load_dwordx4 %0, %2, off sc0 sc1\n\t"
                             "s_mov_b64 exec, %4\n\tglobal_load_dwordx4 %1, %2, off offset:1024 sc0 sc1\n\t"
                             "s_mov_b64 exec, -1\n\ts_waitcnt vmcnt(0)"
                             : "+v"(d0), "+v"(d1) : "v"(sp), "s"(__ballot(p0)), "s"(__ballot(p1)) : "memory");
                p0 = n0 && (((d0.x & 0x80808080u) ^ tg0) | ((d0.y & 0x80808080u) ^ tg1) | ((d0.z & 0x80808080u) ^ tg0) | ((d0.w & 0x80808080u) ^ tg1)) != 0;
                p1 = n1 && (((d1.x & 0x80808080u) ^ tg0) | ((d1.y & 0x80808080u) ^ tg1) | ((d1.z & 0x80808080u) ^ tg0) | ((d1.w & 0x80808080u) ^ tg1)) != 0;
                if (!__any(p0 || p1)) break;
                if (give_up(tmo, call, t0, polls)) { bad = true; break; }
                __builtin_amdgcn_s_sleep(POLL_SLEEP);
            }
        }
        STAMP(7);
        *(LAS u32x4*)(uintptr_t)(sbase + lane * 16) = d0 & 0x7f7f7f7fu;
        *(LAS u32x4*)(uintptr_t)(sbase + 1024 + lane * 16) = d1 & 0x7f7f7f7fu;
        const unsigned poison = bad ? 0x7f7f7f7fu : 0u;
        unsigned lo[8], hi[8];
#pragma unroll
        for (int g = 0; g < 8; ++g) { lo[g] = *(LAS unsigned*)(uintptr_t)ra[g]; hi[g] = *(LAS unsigned*)(uintptr_t)rb[g]; }
#pragma unroll
        for (int h = 0; h < 2; ++h) {
            u32x4 o;
#pragma unroll
            for (int i = 0; i < 4; ++i) o[i] = __builtin_amdgcn_perm(hi[h * 4 + i], lo[h * 4 + i], sel[h * 4 + i]) | poison;
            asm volatile("global_store_dwordx4 %0, %1, off " OUT_POL "\n\ts_nop 1" :: "v"(optr + h * 16 * NSH), "v"(o) : "memory");
        }
        const u32x4 z = {0, 0, 0, 0};
        if (n0) asm volatile("global_store_dwordx4 %0, %1, off sc0 sc1\n\ts_nop 1" :: "v"(sp), "v"(z) : "memory");
        if (n1) asm volatile("global_store_dwordx4 %0, %1, off offset:1024 sc0 sc1\n\ts_nop 1" :: "v"(sp), "v"(z) : "memory");
        WAIT_VM(0);
        STAMP(6);
    }
    if (tid == 256) __hip_atomic_store((GAS unsigned*)(myws + WS_CNT) + b, call, __ATOMIC_RELAXED, __HIP_MEMORY_SCOPE_AGENT);
    STAMP_OUT();
}

}

extern "C" void kernel_launch(int rank, int nranks, ncclComm_t comm,
                              hipStream_t stream,
                              void* const* d_in, const int* in_sizes, int n_in,
                              void* d_out, int out_size,
                              const int* in_dtypes, int out_dtype,
                              void* const* d_out_all,
                              void* const* d_ws_all, size_t ws_bytes) {
    if (nranks != NR || n_in < 2 || in_sizes[0] != MLOC * KD || in_sizes[1] != KD * ND ||
        out_size != ND * NSH || ws_bytes < WS_END || (unsigned)rank >= (unsigned)NR) {
        fprintf(stderr, "kernel_launch: unexpected problem shape\n");
        return;
    }
    HIP_CHECK(hipFuncSetAttribute((const void*)gemm_a2a_kernel, hipFuncAttributeMaxDynamicSharedMemorySize, LDS_BYTES));
    Args a;
    a.x = d_in[0]; a.w = d_in[1]; a.myws = d_ws_all[rank]; a.myout = d_out; a.rank = rank; a.pad = 0;
    for (int p = 0; p < NR; ++p) { a.out[p] = d_out_all[p]; a.ws[p] = d_ws_all[p]; }
    hipLaunchKernelGGL(gemm_a2a_kernel, dim3(NWG), dim3(512), LDS_BYTES, stream, a);
    HIP_CHECK(hipGetLastError());
}
```

```cpp
#include <hip/hip_runtime.h>
#include <rccl/rccl.h>
#include <cstdio>
#include <cstdint>
#include <cstddef>

#define HIP_CHECK(expr) do { hipError_t _e = (expr); if (_e != hipSuccess) { \
    fprintf(stderr, "HIP error %s at %s:%d\n", hipGetErrorString(_e), __FILE__, __LINE__); \
    return; } } while (0)

namespace {

typedef short          s16x4 __attribute__((ext_vector_type(4)));
typedef short          s16x8 __attribute__((ext_vector_type(8)));
typedef float          f32x4 __attribute__((ext_vector_type(4)));
typedef unsigned       u32x4 __attribute__((ext_vector_type(4)));
typedef unsigned       u32x2 __attribute__((ext_vector_type(2)));
#define LAS __attribute__((address_space(3)))
#define GAS __attribute__((address_space(1)))

constexpr int NR = 8;
constexpr int MLOC = 512, KD = 4096, ND = 4096, NSH = 512;
constexpr int BM = 128, BN = 128, BK = 64;
constexpr int NKT = KD / BK;
#define KA_STEPS 28
constexpr int KA = KA_STEPS;
constexpr int S0 = KA & 2;
#define UNIT 1
constexpr int XBYTES = BM * BK * 2;
constexpr int SLOT = XBYTES + BK * BN * 2;
constexpr int NS = 4;
constexpr int CTRL = NS * SLOT;
constexpr int LDS_BYTES = CTRL + 512;
constexpr int NWG = 256, NTILE = 128;
constexpr int CSTR = 8192;
static_assert(KA % 2 == 0 && KA >= 8 && NKT - KA >= 8, "");

constexpr size_t WS_CNT   = 0;
constexpr size_t WS_TMO   = 4096;
constexpr size_t WS_AMAX  = 8192;
constexpr size_t WS_GO    = WS_AMAX + 2 * 8192;
constexpr size_t WS_PFLAG = WS_GO + NTILE * 128;
constexpr size_t WS_MASK  = 1 << 20;
constexpr size_t WS_DATA  = 2 << 20;
constexpr size_t WS_PART  = 4 << 20;
constexpr size_t WS_END   = WS_PART + (size_t)NTILE * 65536;
static_assert(WS_PFLAG + NTILE * 512 <= WS_MASK, "");
#define POLL_SLEEP 1
#define OUT_POL "sc0 sc1"
#define MASK_SLEEP 2
constexpr unsigned long long DEADLINE = 100000000ull;

#define BARRIER() do { asm volatile("" ::: "memory"); __builtin_amdgcn_s_barrier(); asm volatile("" ::: "memory"); } while (0)
#define WAIT_VM(n) asm volatile("s_waitcnt vmcnt(%0)" :: "n"(n) : "memory")
#define WAIT_LGKM0() __builtin_amdgcn_s_waitcnt(0xC07F)
#define SB() __builtin_amdgcn_sched_barrier(0)

__device__ __forceinline__ unsigned rfl(unsigned v) { return __builtin_amdgcn_readfirstlane(v); }

__device__ __forceinline__ unsigned wave_max_u32(unsigned v) {
    v = max(v, (unsigned)__builtin_amdgcn_update_dpp(0, (int)v, 0xB1, 0xF, 0xF, true));
    v = max(v, (unsigned)__builtin_amdgcn_update_dpp(0, (int)v, 0x4E, 0xF, 0xF, true));
    v = max(v, (unsigned)__builtin_amdgcn_update_dpp(0, (int)v, 0x141, 0xF, 0xF, true));
    v = max(v, (unsigned)__builtin_amdgcn_update_dpp(0, (int)v, 0x140, 0xF, 0xF, true));
    unsigned a = __builtin_amdgcn_readlane(v, 0), b = __builtin_amdgcn_readlane(v, 16);
    unsigned c = __builtin_amdgcn_readlane(v, 32), d = __builtin_amdgcn_readlane(v, 48);
    return max(max(a, b), max(c, d));
}

__device__ __forceinline__ unsigned wave_scan_u32(unsigned v) {
    v += (unsigned)__builtin_amdgcn_update_dpp(0, (int)v, 0x111, 0xF, 0xF, true);
    v += (unsigned)__builtin_amdgcn_update_dpp(0, (int)v, 0x112, 0xF, 0xF, true);
    v += (unsigned)__builtin_amdgcn_update_dpp(0, (int)v, 0x114, 0xF, 0xF, true);
    v += (unsigned)__builtin_amdgcn_update_dpp(0, (int)v, 0x118, 0xF, 0xF, true);
    v += (unsigned)__builtin_amdgcn_update_dpp(0, (int)v, 0x142, 0xA, 0xF, false);
    v += (unsigned)__builtin_amdgcn_update_dpp(0, (int)v, 0x143, 0xC, 0xF, false);
    return v;
}
__device__ __forceinline__ unsigned stream_offsets(unsigned m, unsigned (&pos)[8]) {
    unsigned c = m - ((m >> 1) & 0x55555555u);
    c = (c & 0x33333333u) + ((c >> 2) & 0x33333333u);
    unsigned ex[4], tot[4];
#pragma unroll
    for (int j = 0; j < 4; ++j) {
        const unsigned f = (c >> (4 * j)) & 0x000f000fu, inc = wave_scan_u32(f);
        tot[j] = __builtin_amdgcn_readlane(inc, 63); ex[j] = inc - f;
    }
    unsigned base = 0;
#pragma unroll
    for (int g = 0; g < 8; ++g) {
        pos[g] = base + (g < 4 ? ex[g] & 0xffffu : ex[g - 4] >> 16);
        base += g < 4 ? tot[g] & 0xffffu : tot[g - 4] >> 16;
    }
    return base;
}
__device__ __forceinline__ unsigned tag_bits(unsigned n) { return ((n * 0x00204081u) & 0x01010101u) << 7; }

__device__ __forceinline__ bool give_up(GAS unsigned* tmo, unsigned call, unsigned long long t0, unsigned& polls) {
    if (__builtin_expect((++polls & 31u) != 0u, 1)) return false;
    if (rfl(__hip_atomic_load(tmo, __ATOMIC_RELAXED, __HIP_MEMORY_SCOPE_AGENT)) == call ||
        __builtin_amdgcn_s_memrealtime() - t0 > DEADLINE) {
        __hip_atomic_store(tmo, call, __ATOMIC_RELAXED, __HIP_MEMORY_SCOPE_AGENT);
        return true;
    }
    return false;
}

template <int SLEEP>
__device__ __forceinline__ bool wait_flag(GAS unsigned* flag, GAS unsigned* tmo, unsigned call) {
    const unsigned long long t0 = __builtin_amdgcn_s_memrealtime();
    unsigned polls = 0;
    while (rfl(__hip_atomic_load(flag, __ATOMIC_RELAXED, __HIP_MEMORY_SCOPE_SYSTEM)) < call) {
        if (give_up(tmo, call, t0, polls)) return true;
        __builtin_amdgcn_s_sleep(SLEEP);
    }
    return false;
}

#define GLDS(rsrc, ldsaddr, voff, soff) \
    asm volatile("s_mov_b32 m0, %0\n\ts_nop 0\n\tbuffer_load_dwordx4 %1, %2, %3 offen lds" \
                 :: "s"(ldsaddr), "v"(voff), "s"(rsrc), "s"(soff) : "memory")
#define GLDSW GLDS
#define GLDS_SYS(rsrc, ldsaddr, voff, soff) \
    asm volatile("s_mov_b32 m0, %0\n\ts_nop 0\n\tbuffer_load_dwordx4 %1, %2, %3 offen sc0 sc1 lds" \
                 :: "s"(ldsaddr), "v"(voff), "s"(rsrc), "s"(soff) : "memory")

#define GLDS4(rsrc, ldsaddr, voff) \
    asm volatile("s_mov_b32 m0, %0\n\ts_nop 0\n\tbuffer_load_dword %1, %2, 0 offen lds" \
                 :: "s"(ldsaddr), "v"(voff), "s"(rsrc) : "memory")
#define PART_POL "sc0 sc1"
#define ORDER 1
#define PFN 0

#define STAMP(i) do {} while (0)
#define STAMP_OUT() do {} while (0)

struct Frag { s16x8 x[8]; s16x4 w[16]; };

template <int SL, int R>
__device__ __forceinline__ void rd(Frag& f, const unsigned (&ax)[2][2], const unsigned (&aw)[2][4]) {
    constexpr int g = R / 3, e = R % 3, so = (SL & 1) * SLOT, ks = g >> 2, i = g & 3;
    if constexpr (e == 0) f.x[g] = *(LAS s16x8*)(uintptr_t)(ax[SL >> 1][ks] + so + i * 2048);
    else f.w[g * 2 + e - 1] = __builtin_amdgcn_ds_read_tr16_b64_v4i16(
             (LAS s16x4*)(uintptr_t)(aw[SL >> 1][i] + so + ks * 8192 + (e - 1) * 1024));
}
template <int SL, int R>
__device__ __forceinline__ void rd_all(Frag& f, const unsigned (&ax)[2][2], const unsigned (&aw)[2][4]) {
    if constexpr (R < 24) { rd<SL, R>(f, ax, aw); rd_all<SL, R + 1>(f, ax, aw); }
}
template <int G>
__device__ __forceinline__ void mfma(f32x4 (&acc)[16], const Frag& cur) {
    constexpr int ks = G >> 4, c = (G >> 2) & 3, mi = G & 3;
    const s16x8 wf = __builtin_shufflevector(cur.w[(ks * 4 + c) * 2], cur.w[(ks * 4 + c) * 2 + 1], 0, 1, 2, 3, 4, 5, 6, 7);
    asm volatile("v_mfma_f32_16x16x32_bf16 %0, %1, %2, %0" : "+v"(acc[mi * 4 + c]) : "v"(wf), "v"(cur.x[ks * 4 + mi]));
}
template <int SL, int G>
__device__ __forceinline__ void kstep(f32x4 (&acc)[16], const Frag& cur, Frag& nxt,
                                      const unsigned (&ax)[2][2], const unsigned (&aw)[2][4]) {
    if constexpr (G < 32) {
        mfma<G>(acc, cur);
        if constexpr (G < 24) rd<SL, G>(nxt, ax, aw);
        if constexpr (G == 31) { SB(); WAIT_LGKM0(); BARRIER(); }
        SB();
        kstep<SL, G + 1>(acc, cur, nxt, ax, aw);
    }
}
template <int G, int END>
__device__ __forceinline__ void klast(f32x4 (&acc)[16], const Frag& cur) {
    if constexpr (G < END) { mfma<G>(acc, cur); SB(); klast<G + 1, END>(acc, cur); }
}
#define BPG 16

struct Args {
    const void* x; const void* w;
    void* myws; void* myout;
    void* out[NR]; void* ws[NR];
    int rank, pad;
};

__global__ void __launch_bounds__(512, 2)
gemm_a2a_kernel(const Args args)
{
    typedef __attribute__((address_space(4))) const Args* ArgP;
    ArgP ap = (ArgP)__builtin_amdgcn_kernarg_segment_ptr();
    int rank;
#define ARGS_READY() \
    asm volatile("" : "+s"(ap)); \
    rank = ap->rank; myws = (char*)ap->myws; tmo = (GAS unsigned*)(myws + WS_TMO); \
    pflag = (GAS unsigned*)(myws + WS_PFLAG + gt * 512); part = myws + WS_PART + (size_t)gt * 65536
    extern __shared__ __attribute__((aligned(16))) char smem[];
    LAS char* lds = (LAS char*)smem;
    const unsigned ldsb = (unsigned)(uintptr_t)lds;
    const int tid = threadIdx.x, lane = tid & 63;
    const int wid = rfl(tid >> 6);
    const int b = blockIdx.x;
    const int dst = b & 7;
#define TX 0
    const int isB = (b >> 3) & 1, tile = (b >> 4) ^ TX, tm = tile & 3, tn = tile >> 2;
    const int gt = dst * 16 + tile;
    const int m0 = tm * BM;
#define COLMAP 1
    const int nc0 = COLMAP == 0 ? tn * 128 : COLMAP == 1 ? tn * 64 : ((tn & 1) + (tn & 2) * 2) * 64;
    constexpr int NCH = COLMAP == 0 ? 64 : COLMAP == 1 ? 256 : 128;
    const int nst = isB ? NKT - KA : KA, kbase = isB ? KA : 0;
    char *myws, *part; GAS unsigned *tmo, *pflag;
    LAS unsigned* ctrl = (LAS unsigned*)(lds + CTRL);
    STAMP(0);

    f32x4 v[2][4];
    unsigned call, total = 0;
#define FETCH() do { \
        total = rfl(ctrl[16 + wid]); \
        const unsigned ca_ = ldsb + wid * CSTR + lane * 64, sw_ = (lane << 2) & 48; \
        _Pragma("unroll") for (int i = 0; i < 4; ++i) v[0][i] = *(LAS f32x4*)(uintptr_t)(ca_ + ((i * 16) ^ sw_)); \
        if (total > 1024u) { \
            _Pragma("unroll") for (int i = 0; i < 4; ++i) v[1][i] = *(LAS f32x4*)(uintptr_t)(ca_ + 4096 + ((i * 16) ^ sw_)); } \
    } while (0)

    if (__builtin_expect(wid >= 4, 1)) {
        const int lw = wid - 4;
        unsigned long long xp, wp;
        static_assert(offsetof(Args, x) == 0 && offsetof(Args, w) == 8, "");
        asm volatile("s_load_dwordx2 %0, %2, 0x0\n\ts_load_dwordx2 %1, %2, 0x8" : "=&s"(xp), "=&s"(wp) : "s"(ap));
        SB();
        unsigned vx[4], vw[4];
#pragma unroll
        for (int i = 0; i < 4; ++i) {
            const int row = 32 * lw + 8 * i + (lane >> 3);
            vx[i] = (unsigned)((m0 + row) * (KD * 2) + (((lane & 7) ^ ((row >> 1) & 7)) << 4));
            const int k = 16 * lw + 4 * i + (lane >> 4), f = ((k & 3) << 1) | (k & 8);
            const int ch = (lane & 15) ^ f;
            vw[i] = (unsigned)(k * (ND * 2) + (dst * NSH + nc0 + (ch >> 3) * NCH) * 2 + ((ch & 7) << 4));
        }
        const unsigned lx = ldsb + lw * 4096, lwb = ldsb + XBYTES + lw * 4096;
        int ktab;
        {
            const int ph = (dst + 2 * isB) & 3, rot = (tm + tn) & 3, tu = lane / UNIT, nu = nst / UNIT;
            const int bs = tu < ph ? 0 : ph + ((tu - ph) & ~3);
            const int sz = tu < ph ? ph : max(1, min(4, nu - bs));
            ktab = kbase + (bs + (tu - bs + rot) % sz) * UNIT + lane % UNIT;
        }
#define ISSUE(t) do { \
            const unsigned kt_ = __builtin_amdgcn_readlane(ktab, t), so_ = (((t) + S0) & 3) * SLOT; \
            const unsigned sx_ = kt_ * (BK * 2), sw_ = kt_ * (BK * ND * 2); \
            if (ORDER == 0) { _Pragma("unroll") for (int i = 0; i < 4; ++i) { \
                GLDSW(rw, lwb + so_ + i * 1024, vw[i], sw_); GLDS(rx, lx + so_ + i * 1024, vx[i], sx_); } } \
            if (ORDER == 2) { _Pragma("unroll") for (int i = 0; i < 4; ++i) GLDS(rx, lx + so_ + i * 1024, vx[i], sx_); } \
            if (ORDER != 0) { _Pragma("unroll") for (int i = 0; i < 4; ++i) GLDSW(rw, lwb + so_ + i * 1024, vw[i], sw_); } \
            if (ORDER == 1) { _Pragma("unroll") for (int i = 0; i < 4; ++i) GLDS(rx, lx + so_ + i * 1024, vx[i], sx_); } } while (0)
#pragma unroll
        for (int i = 0; i < 4; ++i) asm volatile("" : "+v"(vx[i]), "+v"(vw[i]));
        asm volatile("" : "+v"(ktab));
        asm volatile("s_waitcnt lgkmcnt(0)" : "+s"(xp), "+s"(wp));
        __amdgpu_buffer_rsrc_t rx = __builtin_amdgcn_make_buffer_rsrc((void*)xp, 0, MLOC * KD * 2, 0x00020000);
        __amdgpu_buffer_rsrc_t rw = __builtin_amdgcn_make_buffer_rsrc((void*)wp, 0, KD * ND * 2, 0x00020000);
        ISSUE(0); ISSUE(1);
        ARGS_READY();
        WAIT_VM(8); BARRIER();
        if (!isB) STAMP(1);
        ISSUE(2);
        WAIT_VM(8); BARRIER();
        unsigned cv, fl = 0;
        asm volatile("global_load_dword %0, %1, off sc1" : "=v"(cv) : "v"(myws + WS_CNT + b * 4) : "memory");
        char* peerws;
        asm volatile("global_load_dwordx2 %0, %1, off" : "=v"(peerws)
                     : "v"((const char*)ap + offsetof(Args, ws) + (lane & 7) * 8) : "memory");
        ISSUE(3);
        for (int t = 0; t < nst - NS; ++t) {
            ISSUE(t + NS);
            if (t == nst - NS - 2) asm volatile("global_load_dword %0, %1, off sc0 sc1" : "=v"(fl) : "v"(pflag + lw * 32) : "memory");
            WAIT_VM(16); BARRIER();
        }
        WAIT_VM(8);
        asm volatile("" : "+v"(fl), "+v"(cv), "+v"(peerws));
        call = rfl(cv) + 1;
        char* gp = peerws + WS_AMAX + (call & 1) * 8192 + (rank * NTILE + gt) * 8;
        asm volatile("" : "+v"(gp));
        if (!isB) {
            BARRIER(); WAIT_VM(0); BARRIER(); BARRIER();
            STAMP(2);
        } else {
        unsigned bad = 0;
        {
            __amdgpu_buffer_rsrc_t rp = __builtin_amdgcn_make_buffer_rsrc((void*)part, 0, 65536, 0x00020000);
            const unsigned vp = lane * 16;
#define POLL_PART() asm volatile("global_load_dword %0, %1, off sc0 sc1" : "=v"(fl) : "v"(pflag + lw * 32) : "memory")
#define FETCH_PART() do { STAMP(7); _Pragma("unroll") for (int i = 0; i < 16; ++i) \
                GLDS_SYS(rp, ldsb + (i >> 3) * SLOT + lw * 8192 + (i & 7) * 1024, vp, lw * 16384 + i * 1024); } while (0)
            bool seen = rfl(fl) >= call, got = false;
            if (!seen) POLL_PART();
            BARRIER();
            if (!seen) { WAIT_VM(0); asm volatile("" : "+v"(fl)); seen = rfl(fl) >= call; }
            if (seen) { FETCH_PART(); got = true; WAIT_VM(16); } else POLL_PART();
            BARRIER();
            if (!got) {
                WAIT_VM(0); asm volatile("" : "+v"(fl));
                if (rfl(fl) >= call) { FETCH_PART(); got = true; }
            }
            BARRIER();
            if (!got) { bad = wait_flag<1>(pflag + lw * 32, tmo, call); FETCH_PART(); }
            WAIT_VM(0); BARRIER();
        }
        STAMP(2);
        BARRIER();
        if (lw == 0) {
            const u32x4 m4 = *(LAS u32x4*)ctrl;
            const u32x2 gr = {max(max(m4.x, m4.y), max(m4.z, m4.w)), call};
            if (lane < NR) asm volatile("global_store_dwordx2 %0, %1, off sc0 sc1\n\ts_nop 1" :: "v"(gp), "v"(gr) : "memory");
            STAMP(5);
        }
        BARRIER();
        STAMP(1);
        FETCH();
        STAMP(6);
        const char* g = myws + WS_AMAX + (call & 1) * 8192 + lw * 2048 + lane * 16;
        unsigned mx = 0;
        const unsigned long long t0 = __builtin_amdgcn_s_memrealtime();
    unsigned polls = 0;
        for (;;) {
            u32x4 a, c;
            asm volatile("global_load_dwordx4 %0, %2, off sc0 sc1\n\t"
                         "global_load_dwordx4 %1, %2, off offset:1024 sc0 sc1\n\t"
                         "s_waitcnt vmcnt(0)"
                         : "=&v"(a), "=&v"(c) : "v"(g) : "memory");
            const bool ok = a.y == call && a.w == call && c.y == call && c.w == call;
            mx = max(max(a.x, a.z), max(c.x, c.z));
            if (__all(ok)) break;
            if (give_up(tmo, call, t0, polls)) { bad = 1; break; }
            __builtin_amdgcn_s_sleep(1);
        }
        mx = wave_max_u32(mx);
        if (lane == 0) { ctrl[8 + lw] = mx; ctrl[12 + lw] = bad; }
        STAMP(3);
        }
    } else {
        const int wm = wid >> 1, wn = wid & 1;
        const int fr = lane & 15, fq = lane >> 4, q = (lane & 15) >> 2, p = lane & 3;
        ARGS_READY();
        unsigned callv = __hip_atomic_load((GAS unsigned*)(myws + WS_CNT) + b, __ATOMIC_RELAXED, __HIP_MEMORY_SCOPE_AGENT);
        char* mptr = (char*)ap->ws[dst] + WS_MASK + (((rank * 16 + tile) * 8 + wid * 2) * 64 + lane) * 8;
        unsigned ax[2][2], aw[2][4];
#pragma unroll
        for (int ks = 0; ks < 2; ++ks) {
            const int row = wm * 64 + fr;
            ax[0][ks] = ldsb + row * 128 + (((ks * 4 + fq) ^ ((row >> 1) & 7)) << 4);
            ax[1][ks] = ax[0][ks] + 2 * SLOT;
            asm volatile("" : "+v"(ax[0][ks]), "+v"(ax[1][ks]));
        }
#pragma unroll
        for (int c = 0; c < 4; ++c) {
            const int k = 8 * fq + q, f = ((k & 3) << 1) | (k & 8);
            aw[0][c] = ldsb + XBYTES + k * 256 + (((wn * 8 + 2 * c + (p >> 1)) ^ f) << 4) + (p & 1) * 8;
            aw[1][c] = aw[0][c] + 2 * SLOT;
            asm volatile("" : "+v"(aw[0][c]), "+v"(aw[1][c]));
        }
        f32x4 acc[16];
#pragma unroll
        for (int i = 0; i < 16; ++i) { acc[i] = (f32x4){0.f, 0.f, 0.f, 0.f}; asm volatile("" : "+v"(acc[i])); }
        asm volatile("" : "+v"(mptr));
        Frag f0, f1;
        BARRIER();
        rd_all<S0, 0>(f0, ax, aw);
        WAIT_LGKM0(); BARRIER();
        asm volatile("s_nop 1");
        if constexpr (S0 != 0) { kstep<3, 0>(acc, f0, f1, ax, aw); kstep<0, 0>(acc, f1, f0, ax, aw); }
        for (int t = S0; t < nst - 4; t += 4) {
            kstep<1, 0>(acc, f0, f1, ax, aw); kstep<2, 0>(acc, f1, f0, ax, aw);
            kstep<3, 0>(acc, f0, f1, ax, aw); kstep<0, 0>(acc, f1, f0, ax, aw);
        }
        call = rfl(callv) + 1;
        SB();
        kstep<1, 0>(acc, f0, f1, ax, aw); kstep<2, 0>(acc, f1, f0, ax, aw); kstep<3, 0>(acc, f0, f1, ax, aw);
        klast<0, BPG>(acc, f1);
        f32x4 pa[16];
        if (isB) {
            BARRIER();
#pragma unroll
            for (int i = 0; i < 16; ++i) pa[i] = *(LAS f32x4*)(lds + (i < 8 ? 0 : SLOT) + wid * 8192 + (i & 7) * 1024 + lane * 16);
        } else {
#pragma unroll
            for (int i = 0; i < 16; ++i) pa[i] = (f32x4){0.f, 0.f, 0.f, 0.f};
        }
        SB();
        klast<BPG, 32>(acc, f1);
        asm volatile("s_nop 15" ::: "memory");
#pragma unroll
        for (int i = 0; i < 16; ++i) asm volatile("" : "+v"(acc[i]));
        STAMP(2);
        if (!isB) {
            char* pp = part + wid * 16384 + lane * 16;
#pragma unroll
            for (int i = 0; i < 16; ++i)
                asm volatile("global_store_dwordx4 %0, %1, off offset:%2 " PART_POL "\n\ts_nop 1"
                             :: "v"(pp + (i >> 2) * 4096), "v"(acc[i]), "n"((i & 3) * 1024) : "memory");
            WAIT_VM(0);
            if (lane == 0) __hip_atomic_store(pflag + wid * 32, call, __ATOMIC_RELAXED, __HIP_MEMORY_SCOPE_SYSTEM);
            STAMP(3);
        } else {
        float mx4[4] = {0.f, 0.f, 0.f, 0.f};
#pragma unroll
        for (int i = 0; i < 16; ++i) acc[i] += pa[i];
#pragma unroll
        for (int i = 0; i < 16; i += 2)
#pragma unroll
            for (int e = 0; e < 4; ++e) mx4[e] = __builtin_fmaxf(__builtin_fmaxf(acc[i][e], acc[i + 1][e]), mx4[e]);
        const float mxf = fmaxf(fmaxf(mx4[0], mx4[1]), fmaxf(mx4[2], mx4[3]));
        const unsigned wmx = wave_max_u32(__float_as_uint(mxf) & 0x7fffffffu);
        if (lane == 0) ctrl[wid] = wmx;
        WAIT_LGKM0(); BARRIER();
        STAMP(3);
        const unsigned trash = ldsb + 8 * CSTR + wid * 512 + lane * 4;
        unsigned mq[8];
#pragma unroll
        for (int c = 0; c < 8; ++c) mq[c] = __float_as_uint(acc[2 * c + 1][3]) >> 31;
#pragma unroll
        for (int e = 6; e >= 0; --e)
#pragma unroll
            for (int c = 0; c < 8; ++c) mq[c] = __builtin_amdgcn_alignbit(mq[c], __float_as_uint(acc[2 * c + (e >> 2)][e & 3]), 31);
        unsigned wa[16];
#pragma unroll
        for (int hh = 0; hh < 2; ++hh) {
            const unsigned m = ~(mq[hh * 4] | (mq[hh * 4 + 1] << 8) | (mq[hh * 4 + 2] << 16) | (mq[hh * 4 + 3] << 24));
            const u32x2 mg = {m, call};
            asm volatile("global_store_dwordx2 %0, %1, off offset:%2 sc0 sc1\n\ts_nop 1" :: "v"(mptr + (call & 1) * 524288), "v"(mg), "n"(hh * 512) : "memory");
            unsigned pos[8];
            const unsigned tot = stream_offsets(m, pos);
            if (lane == 0) ctrl[16 + wid * 2 + hh] = tot;
#pragma unroll
            for (int g = 0; g < 8; ++g) wa[hh * 8 + g] = ldsb + (wid * 2 + hh) * CSTR + pos[g] * 4;
        }
#pragma unroll
        for (int e = 0; e < 4; ++e) {
            unsigned sg[16], ad[16];
#pragma unroll
            for (int i = 0; i < 16; ++i) asm("v_ashrrev_i32 %0, 31, %1" : "=v"(sg[i]) : "v"(acc[i][e]));
#pragma unroll
            for (int i = 0; i < 16; ++i) asm("v_bfi_b32 %0, %1, %2, %3" : "=v"(ad[i]) : "v"(sg[i]), "v"(trash), "v"(wa[i]));
#pragma unroll
            for (int i = 0; i < 16; ++i) {
                *(LAS float*)(uintptr_t)(ad[i] + 4 * e) = acc[i][e];
                if (e < 3) asm("v_lshl_add_u32 %0, %1, 2, %0" : "+v"(wa[i]) : "v"(sg[i]));
            }
        }
        WAIT_LGKM0(); BARRIER();
        STAMP(1);
        FETCH();
        STAMP(6);
        }
    }
    const unsigned tag = call % 255u + 1u, tg0 = tag_bits(tag & 15u), tg1 = tag_bits(tag >> 4);
    if (isB) {
        char* dptr = (char*)ap->ws[dst] + WS_DATA + ((rank * 16 + tile) * 8 + wid) * 2048 + lane * 16;
        asm volatile("" : "+v"(dptr));
        WAIT_LGKM0(); BARRIER();
        STAMP(4);
        if (tid == 0) __hip_atomic_store((GAS unsigned*)(myws + WS_GO + gt * 128), call, __ATOMIC_RELAXED, __HIP_MEMORY_SCOPE_AGENT);
        const u32x4 g4 = *(LAS u32x4*)(ctrl + 8), b4 = *(LAS u32x4*)(ctrl + 12);
        const float amax = __uint_as_float(max(max(g4.x, g4.y), max(g4.z, g4.w)));
        const bool timed_out = (b4.x | b4.y | b4.z | b4.w) != 0;
        const float scale = amax / 448.0f;
        const float inv = amax > 0.f ? 1.0f / scale : 0.f;
#pragma unroll
        for (int h = 0; h < 2; ++h) {
            if (h * 1024u < total) {
                u32x4 o;
#pragma unroll
                for (int i = 0; i < 4; ++i) {
                    int r = __builtin_amdgcn_cvt_pk_fp8_f32(fminf(v[h][i][0] * inv, 448.f), fminf(v[h][i][1] * inv, 448.f), 0, false);
                    r = __builtin_amdgcn_cvt_pk_fp8_f32(fminf(v[h][i][2] * inv, 448.f), fminf(v[h][i][3] * inv, 448.f), r, true);
                    o[i] = ((timed_out ? 0x7f7f7f7fu : (unsigned)r) & 0x7f7f7f7fu) | ((i & 1) ? tg1 : tg0);
                }
                if (h * 1024u + lane * 16u < total)
                    asm volatile("global_store_dwordx4 %0, %1, off offset:%2 sc0 sc1\n\ts_nop 1" :: "v"(dptr), "v"(o), "n"(h * 1024) : "memory");
            }
        }
        if (wid < 4) STAMP(5);
    } else {
        bool bad = false;
        unsigned m;
        {
            const char* mp = myws + WS_MASK + (call & 1) * 524288 + ((gt * 8 + wid) * 64 + lane) * 8;
            const unsigned long long t0 = __builtin_amdgcn_s_memrealtime();
    unsigned polls = 0;
            for (;;) {
                u32x2 g;
                asm volatile("global_load_dwordx2 %0, %1, off sc0 sc1\n\ts_waitcnt vmcnt(0)" : "=&v"(g) : "v"(mp) : "memory");
                m = g.x;
                if (__all(g.y == call)) break;
                if (give_up(tmo, call, t0, polls)) { bad = true; m = 0; break; }
                __builtin_amdgcn_s_sleep(MASK_SLEEP);
            }
        }
        STAMP(4);
        unsigned pos[8];
        const unsigned tot = stream_offsets(m, pos);
        const unsigned sbase = ldsb + wid * 8192;
#pragma unroll
        for (int g = 0; g < 8; ++g) *(LAS unsigned*)(uintptr_t)(sbase + 4096 + g * 256 + lane * 4) = pos[g] * 16 + ((m >> (4 * g)) & 15u);
        unsigned ra[8], rb[8], sel[8];
#pragma unroll
        for (int j = 0; j < 8; ++j)
            sel[j] = *(LAS unsigned*)(uintptr_t)(sbase + 4096 + ((j >> 2) * 4 + (lane & 3)) * 256 + ((j & 3) * 16 + (lane >> 2)) * 4);
#pragma unroll
        for (int j = 0; j < 8; ++j) {
            const unsigned e = sel[j], d0 = e >> 6, d1 = d0 + 1;
            const unsigned sp = ((e & 15u) * 0x00204081u) & 0x01010101u, mk = (sp << 8) - sp, s2 = sp + (sp << 8);
            ra[j] = sbase + (d0 ^ ((d0 >> 4) & 3u)) * 4;
            rb[j] = sbase + (d1 ^ ((d1 >> 4) & 3u)) * 4;
            sel[j] = ((((s2 + (sp << 16)) << 8) + __builtin_amdgcn_perm(0u, (e >> 4) & 3u, 0u)) & mk) | (0x0c0c0c0cu & ~mk);
        }
#pragma unroll
        for (int j = 0; j < 8; ++j) asm volatile("" : "+v"(ra[j]), "+v"(rb[j]), "+v"(sel[j]));
        char* optr = (char*)ap->myout + (size_t)(dst * MLOC + m0 + (wid >> 2) * 64 + (wid & 1) * 32 + (lane >> 2)) * NSH
                     + nc0 + ((wid >> 1) & 1) * NCH + (lane & 3) * 16;
        char* sp = myws + WS_DATA + (gt * 8 + wid) * 2048 + lane * 16;
        asm volatile("" : "+v"(optr), "+v"(sp));
        if (!bad) bad = wait_flag<1>((GAS unsigned*)(myws + WS_GO + gt * 128), tmo, call);
        STAMP(5);
        const bool n0 = lane * 16u < tot, n1 = 1024u + lane * 16u < tot;
        u32x4 d0 = {0, 0, 0, 0}, d1 = {0, 0, 0, 0};
        if (!bad) {
            const unsigned long long t0 = __builtin_amdgcn_s_memrealtime();
    unsigned polls = 0;
            bool p0 = n0, p1 = n1;
            for (;;) {
                asm volatile("s_mov_b64 exec, %3\n\tglobal_load_dwordx4 %0, %2, off sc0 sc1\n\t"
                             "s_mov_b64 exec, %4\n\tglobal_load_dwordx4 %1, %2, off offset:1024 sc0 sc1\n\t"
                             "s_mov_b64 exec, -1\n\ts_waitcnt vmcnt(0)"
                             : "+v"(d0), "+v"(d1) : "v"(sp), "s"(__ballot(p0)), "s"(__ballot(p1)) : "memory");
                p0 = n0 && (((d0.x & 0x80808080u) ^ tg0) | ((d0.y & 0x80808080u) ^ tg1) | ((d0.z & 0x80808080u) ^ tg0) | ((d0.w & 0x80808080u) ^ tg1)) != 0;
                p1 = n1 && (((d1.x & 0x80808080u) ^ tg0) | ((d1.y & 0x80808080u) ^ tg1) | ((d1.z & 0x80808080u) ^ tg0) | ((d1.w & 0x80808080u) ^ tg1)) != 0;
                if (!__any(p0 || p1)) break;
                if (give_up(tmo, call, t0, polls)) { bad = true; break; }
                __builtin_amdgcn_s_sleep(POLL_SLEEP);
            }
        }
        STAMP(7);
        *(LAS u32x4*)(uintptr_t)(sbase + lane * 16) = d0 & 0x7f7f7f7fu;
        *(LAS u32x4*)(uintptr_t)(sbase + 1024 + lane * 16) = d1 & 0x7f7f7f7fu;
        const unsigned poison = bad ? 0x7f7f7f7fu : 0u;
        unsigned lo[8], hi[8];
#pragma unroll
        for (int g = 0; g < 8; ++g) { lo[g] = *(LAS unsigned*)(uintptr_t)ra[g]; hi[g] = *(LAS unsigned*)(uintptr_t)rb[g]; }
#pragma unroll
        for (int h = 0; h < 2; ++h) {
            u32x4 o;
#pragma unroll
            for (int i = 0; i < 4; ++i) o[i] = __builtin_amdgcn_perm(hi[h * 4 + i], lo[h * 4 + i], sel[h * 4 + i]) | poison;
            asm volatile("global_store_dwordx4 %0, %1, off " OUT_POL "\n\ts_nop 1" :: "v"(optr + h * 16 * NSH), "v"(o) : "memory");
        }
        const u32x4 z = {0, 0, 0, 0};
        if (n0) asm volatile("global_store_dwordx4 %0, %1, off sc0 sc1\n\ts_nop 1" :: "v"(sp), "v"(z) : "memory");
        if (n1) asm volatile("global_store_dwordx4 %0, %1, off offset:1024 sc0 sc1\n\ts_nop 1" :: "v"(sp), "v"(z) : "memory");
        WAIT_VM(0);
        STAMP(6);
    }
    if (tid == 256) __hip_atomic_store((GAS unsigned*)(myws + WS_CNT) + b, call, __ATOMIC_RELAXED, __HIP_MEMORY_SCOPE_AGENT);
    STAMP_OUT();
}

}

extern "C" void kernel_launch(int rank, int nranks, ncclComm_t comm,
                              hipStream_t stream,
                              void* const* d_in, const int* in_sizes, int n_in,
                              void* d_out, int out_size,
                              const int* in_dtypes, int out_dtype,
                              void* const* d_out_all,
                              void* const* d_ws_all, size_t ws_bytes) {
    if (nranks != NR || n_in < 2 || in_sizes[0] != MLOC * KD || in_sizes[1] != KD * ND ||
        out_size != ND * NSH || ws_bytes < WS_END || (unsigned)rank >= (unsigned)NR) {
        fprintf(stderr, "kernel_launch: unexpected problem shape\n");
        return;
    }
    HIP_CHECK(hipFuncSetAttribute((const void*)gemm_a2a_kernel, hipFuncAttributeMaxDynamicSharedMemorySize, LDS_BYTES));
    Args a;
    a.x = d_in[0]; a.w = d_in[1]; a.myws = d_ws_all[rank]; a.myout = d_out; a.rank = rank; a.pad = 0;
    for (int p = 0; p < NR; ++p) { a.out[p] = d_out_all[p]; a.ws[p] = d_ws_all[p]; }
    hipLaunchKernelGGL(gemm_a2a_kernel, dim3(NWG), dim3(512), LDS_BYTES, stream, a);
    HIP_CHECK(hipGetLastError());
}
```

```cpp
#include <hip/hip_runtime.h>
#include <rccl/rccl.h>
#include <cstdio>
#include <cstdint>
#include <cstddef>

#define HIP_CHECK(expr) do { hipError_t _e = (expr); if (_e != hipSuccess) { \
    fprintf(stderr, "HIP error %s at %s:%d\n", hipGetErrorString(_e), __FILE__, __LINE__); \
    return; } } while (0)

namespace {

typedef short          s16x4 __attribute__((ext_vector_type(4)));
typedef short          s16x8 __attribute__((ext_vector_type(8)));
typedef float          f32x4 __attribute__((ext_vector_type(4)));
typedef unsigned       u32x4 __attribute__((ext_vector_type(4)));
typedef unsigned       u32x2 __attribute__((ext_vector_type(2)));
#define LAS __attribute__((address_space(3)))
#define GAS __attribute__((address_space(1)))

constexpr int NR = 8;
constexpr int MLOC = 512, KD = 4096, ND = 4096, NSH = 512;
constexpr int BM = 128, BN = 128, BK = 64;
constexpr int NKT = KD / BK;
#define KA_STEPS 28
constexpr int KA = KA_STEPS;
constexpr int S0 = KA & 2;
#define UNIT 1
constexpr int XBYTES = BM * BK * 2;
constexpr int SLOT = XBYTES + BK * BN * 2;
constexpr int NS = 4;
constexpr int CTRL = NS * SLOT;
constexpr int LDS_BYTES = CTRL + 512;
constexpr int NWG = 256, NTILE = 128;
constexpr int CSTR = 8192;
static_assert(KA % 2 == 0 && KA >= 8 && NKT - KA >= 8, "");

constexpr size_t WS_CNT   = 0;
constexpr size_t WS_TMO   = 4096;
constexpr size_t WS_AMAX  = 8192;
constexpr size_t WS_GO    = WS_AMAX + 2 * 8192;
constexpr size_t WS_PFLAG = WS_GO + NTILE * 128;
constexpr size_t WS_MASK  = 1 << 20;
constexpr size_t WS_DATA  = 2 << 20;
constexpr size_t WS_PART  = 4 << 20;
constexpr size_t WS_END   = WS_PART + (size_t)NTILE * 65536;
static_assert(WS_PFLAG + NTILE * 512 <= WS_MASK, "");
#define POLL_MODE 0
#define POLL_SLEEP 1
#define OUT_POL "sc0 sc1"
#define MASK_SLEEP 2
constexpr unsigned long long DEADLINE = 100000000ull;

#define BARRIER() do { asm volatile("" ::: "memory"); __builtin_amdgcn_s_barrier(); asm volatile("" ::: "memory"); } while (0)
#define WAIT_VM(n) asm volatile("s_waitcnt vmcnt(%0)" :: "n"(n) : "memory")
#define WAIT_LGKM0() __builtin_amdgcn_s_waitcnt(0xC07F)
#define SB() __builtin_amdgcn_sched_barrier(0)

__device__ __forceinline__ unsigned rfl(unsigned v) { return __builtin_amdgcn_readfirstlane(v); }

__device__ __forceinline__ unsigned wave_max_u32(unsigned v) {
    v = max(v, (unsigned)__builtin_amdgcn_update_dpp(0, (int)v, 0xB1, 0xF, 0xF, true));
    v = max(v, (unsigned)__builtin_amdgcn_update_dpp(0, (int)v, 0x4E, 0xF, 0xF, true));
    v = max(v, (unsigned)__builtin_amdgcn_update_dpp(0, (int)v, 0x141, 0xF, 0xF, true));
    v = max(v, (unsigned)__builtin_amdgcn_update_dpp(0, (int)v, 0x140, 0xF, 0xF, true));
    unsigned a = __builtin_amdgcn_readlane(v, 0), b = __builtin_amdgcn_readlane(v, 16);
    unsigned c = __builtin_amdgcn_readlane(v, 32), d = __builtin_amdgcn_readlane(v, 48);
    return max(max(a, b), max(c, d));
}

__device__ __forceinline__ unsigned wave_scan_u32(unsigned v) {
    v += (unsigned)__builtin_amdgcn_update_dpp(0, (int)v, 0x111, 0xF, 0xF, true);
    v += (unsigned)__builtin_amdgcn_update_dpp(0, (int)v, 0x112, 0xF, 0xF, true);
    v += (unsigned)__builtin_amdgcn_update_dpp(0, (int)v, 0x114, 0xF, 0xF, true);
    v += (unsigned)__builtin_amdgcn_update_dpp(0, (int)v, 0x118, 0xF, 0xF, true);
    v += (unsigned)__builtin_amdgcn_update_dpp(0, (int)v, 0x142, 0xA, 0xF, false);
    v += (unsigned)__builtin_amdgcn_update_dpp(0, (int)v, 0x143, 0xC, 0xF, false);
    return v;
}
__device__ __forceinline__ unsigned stream_offsets(unsigned m, unsigned (&pos)[8]) {
    unsigned c = m - ((m >> 1) & 0x55555555u);
    c = (c & 0x33333333u) + ((c >> 2) & 0x33333333u);
    unsigned ex[4], tot[4];
#pragma unroll
    for (int j = 0; j < 4; ++j) {
        const unsigned f = (c >> (4 * j)) & 0x000f000fu, inc = wave_scan_u32(f);
        tot[j] = __builtin_amdgcn_readlane(inc, 63); ex[j] = inc - f;
    }
    unsigned base = 0;
#pragma unroll
    for (int g = 0; g < 8; ++g) {
        pos[g] = base + (g < 4 ? ex[g] & 0xffffu : ex[g - 4] >> 16);
        base += g < 4 ? tot[g] & 0xffffu : tot[g - 4] >> 16;
    }
    return base;
}
__device__ __forceinline__ unsigned tag_bits(unsigned n) { return ((n * 0x00204081u) & 0x01010101u) << 7; }

__device__ __forceinline__ bool give_up(GAS unsigned* tmo, unsigned call, unsigned long long t0, unsigned& polls) {
    if (__builtin_expect((++polls & 31u) != 0u, 1)) return false;
    if (rfl(__hip_atomic_load(tmo, __ATOMIC_RELAXED, __HIP_MEMORY_SCOPE_AGENT)) == call ||
        __builtin_amdgcn_s_memrealtime() - t0 > DEADLINE) {
        __hip_atomic_store(tmo, call, __ATOMIC_RELAXED, __HIP_MEMORY_SCOPE_AGENT);
        return true;
    }
    return false;
}

template <int SLEEP>
__device__ __forceinline__ bool wait_flag(GAS unsigned* flag, GAS unsigned* tmo, unsigned call) {
    const unsigned long long t0 = __builtin_amdgcn_s_memrealtime();
    unsigned polls = 0;
    while (rfl(__hip_atomic_load(flag, __ATOMIC_RELAXED, __HIP_MEMORY_SCOPE_SYSTEM)) < call) {
        if (give_up(tmo, call, t0, polls)) return true;
        __builtin_amdgcn_s_sleep(SLEEP);
    }
    return false;
}

#define GLDS(rsrc, ldsaddr, voff, soff) \
    asm volatile("s_mov_b32 m0, %0\n\ts_nop 0\n\tbuffer_load_dwordx4 %1, %2, %3 offen lds" \
                 :: "s"(ldsaddr), "v"(voff), "s"(rsrc), "s"(soff) : "memory")
#define GLDSW GLDS
#define GLDS_SYS(rsrc, ldsaddr, voff, soff) \
    asm volatile("s_mov_b32 m0, %0\n\ts_nop 0\n\tbuffer_load_dwordx4 %1, %2, %3 offen sc0 sc1 lds" \
                 :: "s"(ldsaddr), "v"(voff), "s"(rsrc), "s"(soff) : "memory")

#define GLDS4(rsrc, ldsaddr, voff) \
    asm volatile("s_mov_b32 m0, %0\n\ts_nop 0\n\tbuffer_load_dword %1, %2, 0 offen lds" \
                 :: "s"(ldsaddr), "v"(voff), "s"(rsrc) : "memory")
#define PART_POL "sc0 sc1"
#define ORDER 1
#define PFN 0

#define STAMP(i) do {} while (0)
#define STAMP_OUT() do {} while (0)

struct Frag { s16x8 x[8]; s16x4 w[16]; };

template <int SL, int R>
__device__ __forceinline__ void rd(Frag& f, const unsigned (&ax)[2][2], const unsigned (&aw)[2][4]) {
    constexpr int g = R / 3, e = R % 3, so = (SL & 1) * SLOT, ks = g >> 2, i = g & 3;
    if constexpr (e == 0) f.x[g] = *(LAS s16x8*)(uintptr_t)(ax[SL >> 1][ks] + so + i * 2048);
    else f.w[g * 2 + e - 1] = __builtin_amdgcn_ds_read_tr16_b64_v4i16(
             (LAS s16x4*)(uintptr_t)(aw[SL >> 1][i] + so + ks * 8192 + (e - 1) * 1024));
}
template <int SL, int R>
__device__ __forceinline__ void rd_all(Frag& f, const unsigned (&ax)[2][2], const unsigned (&aw)[2][4]) {
    if constexpr (R < 24) { rd<SL, R>(f, ax, aw); rd_all<SL, R + 1>(f, ax, aw); }
}
template <int G>
__device__ __forceinline__ void mfma(f32x4 (&acc)[16], const Frag& cur) {
    constexpr int ks = G >> 4, c = (G >> 2) & 3, mi = G & 3;
    const s16x8 wf = __builtin_shufflevector(cur.w[(ks * 4 + c) * 2], cur.w[(ks * 4 + c) * 2 + 1], 0, 1, 2, 3, 4, 5, 6, 7);
    asm volatile("v_mfma_f32_16x16x32_bf16 %0, %1, %2, %0" : "+v"(acc[mi * 4 + c]) : "v"(wf), "v"(cur.x[ks * 4 + mi]));
}
template <int SL, int G>
__device__ __forceinline__ void kstep(f32x4 (&acc)[16], const Frag& cur, Frag& nxt,
                                      const unsigned (&ax)[2][2], const unsigned (&aw)[2][4]) {
    if constexpr (G < 32) {
        mfma<G>(acc, cur);
        if constexpr (G < 24) rd<SL, G>(nxt, ax, aw);
        if constexpr (G == 31) { SB(); WAIT_LGKM0(); BARRIER(); }
        SB();
        kstep<SL, G + 1>(acc, cur, nxt, ax, aw);
    }
}
template <int G, int END>
__device__ __forceinline__ void klast(f32x4 (&acc)[16], const Frag& cur) {
    if constexpr (G < END) { mfma<G>(acc, cur); SB(); klast<G + 1, END>(acc, cur); }
}
#define BPG 16

struct Args {
    const void* x; const void* w;
    void* myws; void* myout;
    void* out[NR]; void* ws[NR];
    int rank, pad;
};

__global__ void __launch_bounds__(512, 2)
gemm_a2a_kernel(const Args args)
{
    typedef __attribute__((address_space(4))) const Args* ArgP;
    ArgP ap = (ArgP)__builtin_amdgcn_kernarg_segment_ptr();
    int rank;
#define ARGS_READY() \
    asm volatile("" : "+s"(ap)); \
    rank = ap->rank; myws = (char*)ap->myws; tmo = (GAS unsigned*)(myws + WS_TMO); \
    pflag = (GAS unsigned*)(myws + WS_PFLAG + gt * 512); part = myws + WS_PART + (size_t)gt * 65536
    extern __shared__ __attribute__((aligned(16))) char smem[];
    LAS char* lds = (LAS char*)smem;
    const unsigned ldsb = (unsigned)(uintptr_t)lds;
    const int tid = threadIdx.x, lane = tid & 63;
    const int wid = rfl(tid >> 6);
    const int b = blockIdx.x;
    const int dst = b & 7;
#define TX 0
    const int isB = (b >> 3) & 1, tile = (b >> 4) ^ TX, tm = tile & 3, tn = tile >> 2;
    const int gt = dst * 16 + tile;
    const int m0 = tm * BM;
#define COLMAP 1
    const int nc0 = COLMAP == 0 ? tn * 128 : COLMAP == 1 ? tn * 64 : ((tn & 1) + (tn & 2) * 2) * 64;
    constexpr int NCH = COLMAP == 0 ? 64 : COLMAP == 1 ? 256 : 128;
    const int nst = isB ? NKT - KA : KA, kbase = isB ? KA : 0;
    char *myws, *part; GAS unsigned *tmo, *pflag;
    LAS unsigned* ctrl = (LAS unsigned*)(lds + CTRL);
    STAMP(0);

    f32x4 v[2][4];
    unsigned call, total = 0;
#define FETCH() do { \
        total = rfl(ctrl[16 + wid]); \
        const unsigned ca_ = ldsb + wid * CSTR + lane * 64, sw_ = (lane << 2) & 48; \
        _Pragma("unroll") for (int i = 0; i < 4; ++i) v[0][i] = *(LAS f32x4*)(uintptr_t)(ca_ + ((i * 16) ^ sw_)); \
        if (total > 1024u) { \
            _Pragma("unroll") for (int i = 0; i < 4; ++i) v[1][i] = *(LAS f32x4*)(uintptr_t)(ca_ + 4096 + ((i * 16) ^ sw_)); } \
    } while (0)

    if (__builtin_expect(wid >= 4, 1)) {
        const int lw = wid - 4;
        unsigned long long xp, wp;
        static_assert(offsetof(Args, x) == 0 && offsetof(Args, w) == 8, "");
        asm volatile("s_load_dwordx2 %0, %2, 0x0\n\ts_load_dwordx2 %1, %2, 0x8" : "=&s"(xp), "=&s"(wp) : "s"(ap));
        SB();
        unsigned vx[4], vw[4];
#pragma unroll
        for (int i = 0; i < 4; ++i) {
            const int row = 32 * lw + 8 * i + (lane >> 3);
            vx[i] = (unsigned)((m0 + row) * (KD * 2) + (((lane & 7) ^ ((row >> 1) & 7)) << 4));
            const int k = 16 * lw + 4 * i + (lane >> 4), f = ((k & 3) << 1) | (k & 8);
            const int ch = (lane & 15) ^ f;
            vw[i] = (unsigned)(k * (ND * 2) + (dst * NSH + nc0 + (ch >> 3) * NCH) * 2 + ((ch & 7) << 4));
        }
        const unsigned lx = ldsb + lw * 4096, lwb = ldsb + XBYTES + lw * 4096;
        int ktab;
        {
            const int ph = (dst + 2 * isB) & 3, rot = (tm + tn) & 3, tu = lane / UNIT, nu = nst / UNIT;
            const int bs = tu < ph ? 0 : ph + ((tu - ph) & ~3);
            const int sz = tu < ph ? ph : max(1, min(4, nu - bs));
            ktab = kbase + (bs + (tu - bs + rot) % sz) * UNIT + lane % UNIT;
        }
#define ISSUE(t) do { \
            const unsigned kt_ = __builtin_amdgcn_readlane(ktab, t), so_ = (((t) + S0) & 3) * SLOT; \
            const unsigned sx_ = kt_ * (BK * 2), sw_ = kt_ * (BK * ND * 2); \
            if (ORDER == 0) { _Pragma("unroll") for (int i = 0; i < 4; ++i) { \
                GLDSW(rw, lwb + so_ + i * 1024, vw[i], sw_); GLDS(rx, lx + so_ + i * 1024, vx[i], sx_); } } \
            if (ORDER == 2) { _Pragma("unroll") for (int i = 0; i < 4; ++i) GLDS(rx, lx + so_ + i * 1024, vx[i], sx_); } \
            if (ORDER != 0) { _Pragma("unroll") for (int i = 0; i < 4; ++i) GLDSW(rw, lwb + so_ + i * 1024, vw[i], sw_); } \
            if (ORDER == 1) { _Pragma("unroll") for (int i = 0; i < 4; ++i) GLDS(rx, lx + so_ + i * 1024, vx[i], sx_); } } while (0)
#pragma unroll
        for (int i = 0; i < 4; ++i) asm volatile("" : "+v"(vx[i]), "+v"(vw[i]));
        asm volatile("" : "+v"(ktab));
        asm volatile("s_waitcnt lgkmcnt(0)" : "+s"(xp), "+s"(wp));
        __amdgpu_buffer_rsrc_t rx = __builtin_amdgcn_make_buffer_rsrc((void*)xp, 0, MLOC * KD * 2, 0x00020000);
        __amdgpu_buffer_rsrc_t rw = __builtin_amdgcn_make_buffer_rsrc((void*)wp, 0, KD * ND * 2, 0x00020000);
        ISSUE(0); ISSUE(1);
        ARGS_READY();
        WAIT_VM(8); BARRIER();
        if (!isB) STAMP(1);
        ISSUE(2);
        WAIT_VM(8); BARRIER();
        unsigned cv, fl = 0;
        asm volatile("global_load_dword %0, %1, off sc1" : "=v"(cv) : "v"(myws + WS_CNT + b * 4) : "memory");
        char* peerws;
        asm volatile("global_load_dwordx2 %0, %1, off" : "=v"(peerws)
                     : "v"((const char*)ap + offsetof(Args, ws) + (lane & 7) * 8) : "memory");
        ISSUE(3);
        for (int t = 0; t < nst - NS; ++t) {
            ISSUE(t + NS);
            if (t == nst - NS - 2) asm volatile("global_load_dword %0, %1, off sc0 sc1" : "=v"(fl) : "v"(pflag + lw * 32) : "memory");
            WAIT_VM(16); BARRIER();
        }
        WAIT_VM(8);
        asm volatile("" : "+v"(fl), "+v"(cv), "+v"(peerws));
        call = rfl(cv) + 1;
        char* gp = peerws + WS_AMAX + (call & 1) * 8192 + (rank * NTILE + gt) * 8;
        asm volatile("" : "+v"(gp));
        if (!isB) {
            BARRIER(); WAIT_VM(0); BARRIER(); BARRIER();
            STAMP(2);
            if (lw == 0) {
                const char* dws = (const char*)ap->ws[dst];
                const char* tp = lane < 8 ? gp : lane < 16 ? dws + WS_DATA + (rank * 16 + tile) * 16384 + (lane & 3) * 4096
                                                          : dws + WS_MASK + (call & 1) * 524288 + (rank * 16 + tile) * 4096;
                unsigned dummy;
                asm volatile("global_load_dword %0, %1, off sc0 sc1\n\ts_waitcnt vmcnt(0)" : "=&v"(dummy) : "v"(tp) : "memory");
            }
        } else {
        unsigned bad = 0;
        {
            __amdgpu_buffer_rsrc_t rp = __builtin_amdgcn_make_buffer_rsrc((void*)part, 0, 65536, 0x00020000);
            const unsigned vp = lane * 16;
#define POLL_PART() asm volatile("global_load_dword %0, %1, off sc0 sc1" : "=v"(fl) : "v"(pflag + lw * 32) : "memory")
#define FETCH_PART() do { STAMP(7); _Pragma("unroll") for (int i = 0; i < 16; ++i) \
                GLDS_SYS(rp, ldsb + (i >> 3) * SLOT + lw * 8192 + (i & 7) * 1024, vp, lw * 16384 + i * 1024); } while (0)
            bool seen = rfl(fl) >= call, got = false;
            if (!seen) POLL_PART();
            BARRIER();
            if (!seen) { WAIT_VM(0); asm volatile("" : "+v"(fl)); seen = rfl(fl) >= call; }
            if (seen) { FETCH_PART(); got = true; WAIT_VM(16); } else POLL_PART();
            BARRIER();
            if (!got) {
                WAIT_VM(0); asm volatile("" : "+v"(fl));
                if (rfl(fl) >= call) { FETCH_PART(); got = true; }
            }
            BARRIER();
            if (!got) { bad = wait_flag<1>(pflag + lw * 32, tmo, call); FETCH_PART(); }
            WAIT_VM(0); BARRIER();
        }
        STAMP(2);
        BARRIER();
        if (lw == 0) {
            const u32x4 m4 = *(LAS u32x4*)ctrl;
            const u32x2 gr = {max(max(m4.x, m4.y), max(m4.z, m4.w)), call};
            if (lane < NR) asm volatile("global_store_dwordx2 %0, %1, off sc0 sc1\n\ts_nop 1" :: "v"(gp), "v"(gr) : "memory");
            STAMP(5);
        }
        BARRIER();
        STAMP(1);
        FETCH();
        STAMP(6);
        const char* g = myws + WS_AMAX + (call & 1) * 8192 + lw * 2048 + lane * 16;
        unsigned mx = 0;
        const unsigned long long t0 = __builtin_amdgcn_s_memrealtime();
    unsigned polls = 0;
        for (;;) {
            u32x4 a, c;
            asm volatile("global_load_dwordx4 %0, %2, off sc0 sc1\n\t"
                         "global_load_dwordx4 %1, %2, off offset:1024 sc0 sc1\n\t"
                         "s_waitcnt vmcnt(0)"
                         : "=&v"(a), "=&v"(c) : "v"(g) : "memory");
            const bool ok = a.y == call && a.w == call && c.y == call && c.w == call;
            mx = max(max(a.x, a.z), max(c.x, c.z));
            if (__all(ok)) break;
            if (give_up(tmo, call, t0, polls)) { bad = 1; break; }
            __builtin_amdgcn_s_sleep(1);
        }
        mx = wave_max_u32(mx);
        if (lane == 0) { ctrl[8 + lw] = mx; ctrl[12 + lw] = bad; }
        STAMP(3);
        }
    } else {
        const int wm = wid >> 1, wn = wid & 1;
        const int fr = lane & 15, fq = lane >> 4, q = (lane & 15) >> 2, p = lane & 3;
        ARGS_READY();
        unsigned callv = __hip_atomic_load((GAS unsigned*)(myws + WS_CNT) + b, __ATOMIC_RELAXED, __HIP_MEMORY_SCOPE_AGENT);
        char* mptr = (char*)ap->ws[dst] + WS_MASK + (((rank * 16 + tile) * 8 + wid * 2) * 64 + lane) * 8;
        unsigned ax[2][2], aw[2][4];
#pragma unroll
        for (int ks = 0; ks < 2; ++ks) {
            const int row = wm * 64 + fr;
            ax[0][ks] = ldsb + row * 128 + (((ks * 4 + fq) ^ ((row >> 1) & 7)) << 4);
            ax[1][ks] = ax[0][ks] + 2 * SLOT;
            asm volatile("" : "+v"(ax[0][ks]), "+v"(ax[1][ks]));
        }
#pragma unroll
        for (int c = 0; c < 4; ++c) {
            const int k = 8 * fq + q, f = ((k & 3) << 1) | (k & 8);
            aw[0][c] = ldsb + XBYTES + k * 256 + (((wn * 8 + 2 * c + (p >> 1)) ^ f) << 4) + (p & 1) * 8;
            aw[1][c] = aw[0][c] + 2 * SLOT;
            asm volatile("" : "+v"(aw[0][c]), "+v"(aw[1][c]));
        }
        f32x4 acc[16];
#pragma unroll
        for (int i = 0; i < 16; ++i) { acc[i] = (f32x4){0.f, 0.f, 0.f, 0.f}; asm volatile("" : "+v"(acc[i])); }
        asm volatile("" : "+v"(mptr));
        Frag f0, f1;
        BARRIER();
        rd_all<S0, 0>(f0, ax, aw);
        WAIT_LGKM0(); BARRIER();
        asm volatile("s_nop 1");
        if constexpr (S0 != 0) { kstep<3, 0>(acc, f0, f1, ax, aw); kstep<0, 0>(acc, f1, f0, ax, aw); }
        for (int t = S0; t < nst - 4; t += 4) {
            kstep<1, 0>(acc, f0, f1, ax, aw); kstep<2, 0>(acc, f1, f0, ax, aw);
            kstep<3, 0>(acc, f0, f1, ax, aw); kstep<0, 0>(acc, f1, f0, ax, aw);
        }
        call = rfl(callv) + 1;
        SB();
        kstep<1, 0>(acc, f0, f1, ax, aw); kstep<2, 0>(acc, f1, f0, ax, aw); kstep<3, 0>(acc, f0, f1, ax, aw);
        klast<0, BPG>(acc, f1);
        f32x4 pa[16];
        if (isB) {
            BARRIER();
#pragma unroll
            for (int i = 0; i < 16; ++i) pa[i] = *(LAS f32x4*)(lds + (i < 8 ? 0 : SLOT) + wid * 8192 + (i & 7) * 1024 + lane * 16);
        } else {
#pragma unroll
            for (int i = 0; i < 16; ++i) pa[i] = (f32x4){0.f, 0.f, 0.f, 0.f};
        }
        SB();
        klast<BPG, 32>(acc, f1);
        asm volatile("s_nop 15" ::: "memory");
#pragma unroll
        for (int i = 0; i < 16; ++i) asm volatile("" : "+v"(acc[i]));
        STAMP(2);
        if (!isB) {
            char* pp = part + wid * 16384 + lane * 16;
#pragma unroll
            for (int i = 0; i < 16; ++i)
                asm volatile("global_store_dwordx4 %0, %1, off offset:%2 " PART_POL "\n\ts_nop 1"
                             :: "v"(pp + (i >> 2) * 4096), "v"(acc[i]), "n"((i & 3) * 1024) : "memory");
            WAIT_VM(0);
            if (lane == 0) __hip_atomic_store(pflag + wid * 32, call, __ATOMIC_RELAXED, __HIP_MEMORY_SCOPE_SYSTEM);
            STAMP(3);
        } else {
        float mx4[4] = {0.f, 0.f, 0.f, 0.f};
#pragma unroll
        for (int i = 0; i < 16; ++i) acc[i] += pa[i];
#pragma unroll
        for (int i = 0; i < 16; i += 2)
#pragma unroll
            for (int e = 0; e < 4; ++e) mx4[e] = __builtin_fmaxf(__builtin_fmaxf(acc[i][e], acc[i + 1][e]), mx4[e]);
        const float mxf = fmaxf(fmaxf(mx4[0], mx4[1]), fmaxf(mx4[2], mx4[3]));
        const unsigned wmx = wave_max_u32(__float_as_uint(mxf) & 0x7fffffffu);
        if (lane == 0) ctrl[wid] = wmx;
        WAIT_LGKM0(); BARRIER();
        STAMP(3);
        const unsigned trash = ldsb + 8 * CSTR + wid * 512 + lane * 4;
        unsigned mq[8];
#pragma unroll
        for (int c = 0; c < 8; ++c) mq[c] = __float_as_uint(acc[2 * c + 1][3]) >> 31;
#pragma unroll
        for (int e = 6; e >= 0; --e)
#pragma unroll
            for (int c = 0; c < 8; ++c) mq[c] = __builtin_amdgcn_alignbit(mq[c], __float_as_uint(acc[2 * c + (e >> 2)][e & 3]), 31);
        unsigned wa[16];
#pragma unroll
        for (int hh = 0; hh < 2; ++hh) {
            const unsigned m = ~(mq[hh * 4] | (mq[hh * 4 + 1] << 8) | (mq[hh * 4 + 2] << 16) | (mq[hh * 4 + 3] << 24));
            const u32x2 mg = {m, call};
            asm volatile("global_store_dwordx2 %0, %1, off offset:%2 sc0 sc1\n\ts_nop 1" :: "v"(mptr + (call & 1) * 524288), "v"(mg), "n"(hh * 512) : "memory");
            unsigned pos[8];
            const unsigned tot = stream_offsets(m, pos);
            if (lane == 0) ctrl[16 + wid * 2 + hh] = tot;
#pragma unroll
            for (int g = 0; g < 8; ++g) wa[hh * 8 + g] = ldsb + (wid * 2 + hh) * CSTR + pos[g] * 4;
        }
#pragma unroll
        for (int e = 0; e < 4; ++e) {
            unsigned sg[16], ad[16];
#pragma unroll
            for (int i = 0; i < 16; ++i) asm("v_ashrrev_i32 %0, 31, %1" : "=v"(sg[i]) : "v"(acc[i][e]));
#pragma unroll
            for (int i = 0; i < 16; ++i) asm("v_bfi_b32 %0, %1, %2, %3" : "=v"(ad[i]) : "v"(sg[i]), "v"(trash), "v"(wa[i]));
#pragma unroll
            for (int i = 0; i < 16; ++i) {
                *(LAS float*)(uintptr_t)(ad[i] + 4 * e) = acc[i][e];
                if (e < 3) asm("v_lshl_add_u32 %0, %1, 2, %0" : "+v"(wa[i]) : "v"(sg[i]));
            }
        }
        WAIT_LGKM0(); BARRIER();
        STAMP(1);
        FETCH();
        STAMP(6);
        }
    }
    const unsigned tag = call % 255u + 1u, tg0 = tag_bits(tag & 15u), tg1 = tag_bits(tag >> 4);
    if (isB) {
        char* dptr = (char*)ap->ws[dst] + WS_DATA + ((rank * 16 + tile) * 8 + wid) * 2048 + lane * 16;
        asm volatile("" : "+v"(dptr));
        WAIT_LGKM0(); BARRIER();
        STAMP(4);
        if (tid == 0) __hip_atomic_store((GAS unsigned*)(myws + WS_GO + gt * 128), call, __ATOMIC_RELAXED, __HIP_MEMORY_SCOPE_AGENT);
        const u32x4 g4 = *(LAS u32x4*)(ctrl + 8), b4 = *(LAS u32x4*)(ctrl + 12);
        const float amax = __uint_as_float(max(max(g4.x, g4.y), max(g4.z, g4.w)));
        const bool timed_out = (b4.x | b4.y | b4.z | b4.w) != 0;
        float inv = 448.0f * __builtin_amdgcn_rcpf(amax);
        if (__builtin_expect(!(amax > 1e-30f && amax < 1e30f), 0)) inv = amax > 0.f ? 1.0f / (amax / 448.0f) : 0.f;
#pragma unroll
        for (int h = 0; h < 2; ++h) {
            if (h * 1024u < total) {
                u32x4 o;
#pragma unroll
                for (int i = 0; i < 4; ++i) {
                    int r = __builtin_amdgcn_cvt_pk_fp8_f32(fminf(v[h][i][0] * inv, 448.f), fminf(v[h][i][1] * inv, 448.f), 0, false);
                    r = __builtin_amdgcn_cvt_pk_fp8_f32(fminf(v[h][i][2] * inv, 448.f), fminf(v[h][i][3] * inv, 448.f), r, true);
                    o[i] = ((timed_out ? 0x7f7f7f7fu : (unsigned)r) & 0x7f7f7f7fu) | ((i & 1) ? tg1 : tg0);
                }
                if (h * 1024u + lane * 16u < total)
                    asm volatile("global_store_dwordx4 %0, %1, off offset:%2 sc0 sc1\n\ts_nop 1" :: "v"(dptr), "v"(o), "n"(h * 1024) : "memory");
            }
        }
        if (wid < 4) STAMP(5);
    } else {
        bool bad = false;
        unsigned m;
        {
            const char* mp = myws + WS_MASK + (call & 1) * 524288 + ((gt * 8 + wid) * 64 + lane) * 8;
            const unsigned long long t0 = __builtin_amdgcn_s_memrealtime();
    unsigned polls = 0;
            for (;;) {
                u32x2 g;
                asm volatile("global_load_dwordx2 %0, %1, off sc0 sc1\n\ts_waitcnt vmcnt(0)" : "=&v"(g) : "v"(mp) : "memory");
                m = g.x;
                if (__all(g.y == call)) break;
                if (give_up(tmo, call, t0, polls)) { bad = true; m = 0; break; }
                __builtin_amdgcn_s_sleep(MASK_SLEEP);
            }
        }
        STAMP(4);
        unsigned pos[8];
        const unsigned tot = stream_offsets(m, pos);
        const unsigned sbase = ldsb + wid * 8192;
#pragma unroll
        for (int g = 0; g < 8; ++g) *(LAS unsigned*)(uintptr_t)(sbase + 4096 + g * 256 + lane * 4) = pos[g] * 16 + ((m >> (4 * g)) & 15u);
        unsigned ra[8], rb[8], sel[8];
#pragma unroll
        for (int j = 0; j < 8; ++j)
            sel[j] = *(LAS unsigned*)(uintptr_t)(sbase + 4096 + ((j >> 2) * 4 + (lane & 3)) * 256 + ((j & 3) * 16 + (lane >> 2)) * 4);
#pragma unroll
        for (int j = 0; j < 8; ++j) {
            const unsigned e = sel[j], d0 = e >> 6, d1 = d0 + 1;
            const unsigned sp = ((e & 15u) * 0x00204081u) & 0x01010101u, mk = (sp << 8) - sp, s2 = sp + (sp << 8);
            ra[j] = sbase + (d0 ^ ((d0 >> 4) & 3u)) * 4;
            rb[j] = sbase + (d1 ^ ((d1 >> 4) & 3u)) * 4;
            sel[j] = ((((s2 + (sp << 16)) << 8) + __builtin_amdgcn_perm(0u, (e >> 4) & 3u, 0u)) & mk) | (0x0c0c0c0cu & ~mk);
        }
#pragma unroll
        for (int j = 0; j < 8; ++j) asm volatile("" : "+v"(ra[j]), "+v"(rb[j]), "+v"(sel[j]));
        char* optr = (char*)ap->myout + (size_t)(dst * MLOC + m0 + (wid >> 2) * 64 + (wid & 1) * 32 + (lane >> 2)) * NSH
                     + nc0 + ((wid >> 1) & 1) * NCH + (lane & 3) * 16;
        char* sp = myws + WS_DATA + (gt * 8 + wid) * 2048 + lane * 16;
        asm volatile("" : "+v"(optr), "+v"(sp));
        if (!bad) bad = wait_flag<1>((GAS unsigned*)(myws + WS_GO + gt * 128), tmo, call);
        STAMP(5);
        const bool n0 = lane * 16u < tot, n1 = 1024u + lane * 16u < tot;
        u32x4 d0 = {0, 0, 0, 0}, d1 = {0, 0, 0, 0};
        if (!bad) {
            const unsigned long long t0 = __builtin_amdgcn_s_memrealtime();
    unsigned polls = 0;
            unsigned long long p0 = __ballot(n0), p1 = __ballot(n1);
            unsigned long long q0 = p0, q1 = p1;
            while (p0 | p1) {
                asm volatile("s_mov_b64 exec, %3\n\tglobal_load_dwordx4 %0, %2, off sc0 sc1\n\t"
                             "s_mov_b64 exec, %4\n\tglobal_load_dwordx4 %1, %2, off offset:1024 sc0 sc1\n\t"
                             "s_mov_b64 exec, -1\n\ts_waitcnt vmcnt(0)"
                             : "+v"(d0), "+v"(d1) : "v"(sp), "s"(q0), "s"(q1) : "memory");
                const unsigned long long b0 = __ballot((((d0.x & 0x80808080u) ^ tg0) | ((d0.y & 0x80808080u) ^ tg1) | ((d0.z & 0x80808080u) ^ tg0) | ((d0.w & 0x80808080u) ^ tg1)) != 0);
                const unsigned long long b1 = __ballot((((d1.x & 0x80808080u) ^ tg0) | ((d1.y & 0x80808080u) ^ tg1) | ((d1.z & 0x80808080u) ^ tg0) | ((d1.w & 0x80808080u) ^ tg1)) != 0);
                p0 &= b0 | ~q0; p1 &= b1 | ~q1;
                q0 &= b0; q1 &= b1;
                if ((q0 | q1) == 0) { q0 = p0; q1 = p1; continue; }
                if (give_up(tmo, call, t0, polls)) { bad = true; break; }
                __builtin_amdgcn_s_sleep(POLL_SLEEP);
            }
        }
        STAMP(7);
        *(LAS u32x4*)(uintptr_t)(sbase + lane * 16) = d0 & 0x7f7f7f7fu;
        *(LAS u32x4*)(uintptr_t)(sbase + 1024 + lane * 16) = d1 & 0x7f7f7f7fu;
        const unsigned poison = bad ? 0x7f7f7f7fu : 0u;
        unsigned lo[8], hi[8];
#pragma unroll
        for (int g = 0; g < 8; ++g) { lo[g] = *(LAS unsigned*)(uintptr_t)ra[g]; hi[g] = *(LAS unsigned*)(uintptr_t)rb[g]; }
#pragma unroll
        for (int h = 0; h < 2; ++h) {
            u32x4 o;
#pragma unroll
            for (int i = 0; i < 4; ++i) o[i] = __builtin_amdgcn_perm(hi[h * 4 + i], lo[h * 4 + i], sel[h * 4 + i]) | poison;
            asm volatile("global_store_dwordx4 %0, %1, off " OUT_POL "\n\ts_nop 1" :: "v"(optr + h * 16 * NSH), "v"(o) : "memory");
        }
        const u32x4 z = {0, 0, 0, 0};
        if (n0) asm volatile("global_store_dwordx4 %0, %1, off sc0 sc1\n\ts_nop 1" :: "v"(sp), "v"(z) : "memory");
        if (n1) asm volatile("global_store_dwordx4 %0, %1, off offset:1024 sc0 sc1\n\ts_nop 1" :: "v"(sp), "v"(z) : "memory");
        WAIT_VM(0);
        STAMP(6);
    }
    if (tid == 256) __hip_atomic_store((GAS unsigned*)(myws + WS_CNT) + b, call, __ATOMIC_RELAXED, __HIP_MEMORY_SCOPE_AGENT);
    STAMP_OUT();
}

}

extern "C" void kernel_launch(int rank, int nranks, ncclComm_t comm,
                              hipStream_t stream,
                              void* const* d_in, const int* in_sizes, int n_in,
                              void* d_out, int out_size,
                              const int* in_dtypes, int out_dtype,
                              void* const* d_out_all,
                              void* const* d_ws_all, size_t ws_bytes) {
    if (nranks != NR || n_in < 2 || in_sizes[0] != MLOC * KD || in_sizes[1] != KD * ND ||
        out_size != ND * NSH || ws_bytes < WS_END || (unsigned)rank >= (unsigned)NR) {
        fprintf(stderr, "kernel_launch: unexpected problem shape\n");
        return;
    }
    HIP_CHECK(hipFuncSetAttribute((const void*)gemm_a2a_kernel, hipFuncAttributeMaxDynamicSharedMemorySize, LDS_BYTES));
    Args a;
    a.x = d_in[0]; a.w = d_in[1]; a.myws = d_ws_all[rank]; a.myout = d_out; a.rank = rank; a.pad = 0;
    for (int p = 0; p < NR; ++p) { a.out[p] = d_out_all[p]; a.ws[p] = d_ws_all[p]; }
    hipLaunchKernelGGL(gemm_a2a_kernel, dim3(NWG), dim3(512), LDS_BYTES, stream, a);
    HIP_CHECK(hipGetLastError());
}
```

```cpp
#include <hip/hip_runtime.h>
#include <rccl/rccl.h>
#include <cstdio>
#include <cstdint>
#include <cstddef>

#define HIP_CHECK(expr) do { hipError_t _e = (expr); if (_e != hipSuccess) { \
    fprintf(stderr, "HIP error %s at %s:%d\n", hipGetErrorString(_e), __FILE__, __LINE__); \
    return; } } while (0)

namespace {

typedef short          s16x4 __attribute__((ext_vector_type(4)));
typedef short          s16x8 __attribute__((ext_vector_type(8)));
typedef float          f32x4 __attribute__((ext_vector_type(4)));
typedef unsigned       u32x4 __attribute__((ext_vector_type(4)));
typedef unsigned       u32x2 __attribute__((ext_vector_type(2)));
#define LAS __attribute__((address_space(3)))
#define GAS __attribute__((address_space(1)))

constexpr int NR = 8;
constexpr int MLOC = 512, KD = 4096, ND = 4096, NSH = 512;
constexpr int BM = 128, BN = 128, BK = 64;
constexpr int NKT = KD / BK;
#define KA_STEPS 28
constexpr int KA = KA_STEPS;
constexpr int S0 = KA & 2;
constexpr int XBYTES = BM * BK * 2;
constexpr int SLOT = XBYTES + BK * BN * 2;
constexpr int NS = 4;
constexpr int CTRL = NS * SLOT;
constexpr int LDS_BYTES = CTRL + 512;
constexpr int NWG = 256, NTILE = 128;
constexpr int CSTR = 8192;
static_assert(KA % 2 == 0 && KA >= 8 && NKT - KA >= 8, "");

constexpr size_t WS_CNT   = 0;
constexpr size_t WS_TMO   = 4096;
constexpr size_t WS_AMAX  = 8192;
constexpr size_t WS_GO    = WS_AMAX + 2 * 8192;
constexpr size_t WS_PFLAG = WS_GO + NTILE * 128;
constexpr size_t WS_MASK  = 1 << 20;
constexpr size_t WS_DATA  = 2 << 20;
constexpr size_t WS_PART  = 4 << 20;
constexpr size_t WS_END   = WS_PART + (size_t)NTILE * 65536;
static_assert(WS_PFLAG + NTILE * 512 <= WS_MASK, "");
#define POLL_MODE 0
#define POLL_SLEEP 1
#define OUT_POL "sc0 sc1"
#define MASK_SLEEP 2
constexpr unsigned long long DEADLINE = 100000000ull;

#define BARRIER() do { asm volatile("" ::: "memory"); __builtin_amdgcn_s_barrier(); asm volatile("" ::: "memory"); } while (0)
#define WAIT_VM(n) asm volatile("s_waitcnt vmcnt(%0)" :: "n"(n) : "memory")
#define WAIT_LGKM0() __builtin_amdgcn_s_waitcnt(0xC07F)
#define SB() __builtin_amdgcn_sched_barrier(0)

__device__ __forceinline__ unsigned rfl(unsigned v) { return __builtin_amdgcn_readfirstlane(v); }

__device__ __forceinline__ unsigned wave_max_u32(unsigned v) {
    v = max(v, (unsigned)__builtin_amdgcn_update_dpp(0, (int)v, 0xB1, 0xF, 0xF, true));
    v = max(v, (unsigned)__builtin_amdgcn_update_dpp(0, (int)v, 0x4E, 0xF, 0xF, true));
    v = max(v, (unsigned)__builtin_amdgcn_update_dpp(0, (int)v, 0x141, 0xF, 0xF, true));
    v = max(v, (unsigned)__builtin_amdgcn_update_dpp(0, (int)v, 0x140, 0xF, 0xF, true));
    unsigned a = __builtin_amdgcn_readlane(v, 0), b = __builtin_amdgcn_readlane(v, 16);
    unsigned c = __builtin_amdgcn_readlane(v, 32), d = __builtin_amdgcn_readlane(v, 48);
    return max(max(a, b), max(c, d));
}

__device__ __forceinline__ unsigned wave_scan_u32(unsigned v) {
    v += (unsigned)__builtin_amdgcn_update_dpp(0, (int)v, 0x111, 0xF, 0xF, true);
    v += (unsigned)__builtin_amdgcn_update_dpp(0, (int)v, 0x112, 0xF, 0xF, true);
    v += (unsigned)__builtin_amdgcn_update_dpp(0, (int)v, 0x114, 0xF, 0xF, true);
    v += (unsigned)__builtin_amdgcn_update_dpp(0, (int)v, 0x118, 0xF, 0xF, true);
    v += (unsigned)__builtin_amdgcn_update_dpp(0, (int)v, 0x142, 0xA, 0xF, false);
    v += (unsigned)__builtin_amdgcn_update_dpp(0, (int)v, 0x143, 0xC, 0xF, false);
    return v;
}
__device__ __forceinline__ unsigned stream_offsets(unsigned m, unsigned (&pos)[8]) {
    unsigned c = m - ((m >> 1) & 0x55555555u);
    c = (c & 0x33333333u) + ((c >> 2) & 0x33333333u);
    unsigned ex[4], tot[4];
#pragma unroll
    for (int j = 0; j < 4; ++j) {
        const unsigned f = (c >> (4 * j)) & 0x000f000fu, inc = wave_scan_u32(f);
        tot[j] = __builtin_amdgcn_readlane(inc, 63); ex[j] = inc - f;
    }
    unsigned base = 0;
#pragma unroll
    for (int g = 0; g < 8; ++g) {
        pos[g] = base + (g < 4 ? ex[g] & 0xffffu : ex[g - 4] >> 16);
        base += g < 4 ? tot[g] & 0xffffu : tot[g - 4] >> 16;
    }
    return base;
}
__device__ __forceinline__ unsigned tag_bits(unsigned n) { return ((n * 0x00204081u) & 0x01010101u) << 7; }

__device__ __forceinline__ bool give_up(GAS unsigned* tmo, unsigned call, unsigned long long t0, unsigned& polls) {
    if (__builtin_expect((++polls & 31u) != 0u, 1)) return false;
    if (rfl(__hip_atomic_load(tmo, __ATOMIC_RELAXED, __HIP_MEMORY_SCOPE_AGENT)) == call ||
        __builtin_amdgcn_s_memrealtime() - t0 > DEADLINE) {
        __hip_atomic_store(tmo, call, __ATOMIC_RELAXED, __HIP_MEMORY_SCOPE_AGENT);
        return true;
    }
    return false;
}

template <int SLEEP>
__device__ __forceinline__ bool wait_flag(GAS unsigned* flag, GAS unsigned* tmo, unsigned call) {
    const unsigned long long t0 = __builtin_amdgcn_s_memrealtime();
    unsigned polls = 0;
    while (rfl(__hip_atomic_load(flag, __ATOMIC_RELAXED, __HIP_MEMORY_SCOPE_SYSTEM)) < call) {
        if (give_up(tmo, call, t0, polls)) return true;
        __builtin_amdgcn_s_sleep(SLEEP);
    }
    return false;
}

#define GLDS(rsrc, ldsaddr, voff, soff) \
    asm volatile("s_mov_b32 m0, %0\n\ts_nop 0\n\tbuffer_load_dwordx4 %1, %2, %3 offen lds" \
                 :: "s"(ldsaddr), "v"(voff), "s"(rsrc), "s"(soff) : "memory")
#define GLDSW GLDS
#define GLDS_SYS(rsrc, ldsaddr, voff, soff) \
    asm volatile("s_mov_b32 m0, %0\n\ts_nop 0\n\tbuffer_load_dwordx4 %1, %2, %3 offen sc0 sc1 lds" \
                 :: "s"(ldsaddr), "v"(voff), "s"(rsrc), "s"(soff) : "memory")

#define GLDS4(rsrc, ldsaddr, voff) \
    asm volatile("s_mov_b32 m0, %0\n\ts_nop 0\n\tbuffer_load_dword %1, %2, 0 offen lds" \
                 :: "s"(ldsaddr), "v"(voff), "s"(rsrc) : "memory")
#define PART_POL "sc0 sc1"
#define LASTB 16
#define LASTA 12
#define ORDER 1
#define PFN 0

#define STAMP(i) do {} while (0)
#define TR(i) do {} while (0)
#define STAMP_OUT() do {} while (0)

struct Frag { s16x8 x[8]; s16x4 w[16]; };

template <int SL, int R>
__device__ __forceinline__ void rd(Frag& f, const unsigned (&ax)[2][2], const unsigned (&aw)[2][4]) {
    constexpr int g = R / 3, e = R % 3, so = (SL & 1) * SLOT, ks = g >> 2, i = g & 3;
    if constexpr (e == 0) f.x[g] = *(LAS s16x8*)(uintptr_t)(ax[SL >> 1][ks] + so + i * 2048);
    else f.w[g * 2 + e - 1] = __builtin_amdgcn_ds_read_tr16_b64_v4i16(
             (LAS s16x4*)(uintptr_t)(aw[SL >> 1][i] + so + ks * 8192 + (e - 1) * 1024));
}
template <int SL, int R>
__device__ __forceinline__ void rd_all(Frag& f, const unsigned (&ax)[2][2], const unsigned (&aw)[2][4]) {
    if constexpr (R < 24) { rd<SL, R>(f, ax, aw); rd_all<SL, R + 1>(f, ax, aw); }
}
template <int G>
__device__ __forceinline__ void mfma(f32x4 (&acc)[16], const Frag& cur) {
    constexpr int ks = G >> 4, c = (G >> 2) & 3, mi = G & 3;
    const s16x8 wf = __builtin_shufflevector(cur.w[(ks * 4 + c) * 2], cur.w[(ks * 4 + c) * 2 + 1], 0, 1, 2, 3, 4, 5, 6, 7);
    asm volatile("v_mfma_f32_16x16x32_bf16 %0, %1, %2, %0" : "+v"(acc[mi * 4 + c]) : "v"(wf), "v"(cur.x[ks * 4 + mi]));
}
template <int SL, int G>
__device__ __forceinline__ void kstep(f32x4 (&acc)[16], const Frag& cur, Frag& nxt,
                                      const unsigned (&ax)[2][2], const unsigned (&aw)[2][4]) {
    if constexpr (G < 32) {
        mfma<G>(acc, cur);
        if constexpr (G < 24) rd<SL, G>(nxt, ax, aw);
        if constexpr (G == 31) { SB(); WAIT_LGKM0(); BARRIER(); }
        SB();
        kstep<SL, G + 1>(acc, cur, nxt, ax, aw);
    }
}
template <int G, int END>
__device__ __forceinline__ void klast(f32x4 (&acc)[16], const Frag& cur) {
    if constexpr (G < END) { mfma<G>(acc, cur); SB(); klast<G + 1, END>(acc, cur); }
}
#define BPG 16

struct Args {
    const void* x; const void* w;
    void* myws; void* myout;
    void* out[NR]; void* ws[NR];
    int rank, pad;
};

__global__ void __launch_bounds__(512, 2)
gemm_a2a_kernel(const Args args)
{
    typedef __attribute__((address_space(4))) const Args* ArgP;
    ArgP ap = (ArgP)__builtin_amdgcn_kernarg_segment_ptr();
    int rank;
#define ARGS_READY() \
    asm volatile("" : "+s"(ap)); \
    rank = ap->rank; myws = (char*)ap->myws; tmo = (GAS unsigned*)(myws + WS_TMO); \
    pflag = (GAS unsigned*)(myws + WS_PFLAG + gt * 512); part = myws + WS_PART + (size_t)gt * 65536
    extern __shared__ __attribute__((aligned(16))) char smem[];
    LAS char* lds = (LAS char*)smem;
    const unsigned ldsb = (unsigned)(uintptr_t)lds;
    const int tid = threadIdx.x, lane = tid & 63;
    const int wid = rfl(tid >> 6);
    const int b = blockIdx.x;
    const int dst = b & 7;
#define TX 0
    const int isB = (b >> 3) & 1, tile = (b >> 4) ^ TX, tm = tile & 3, tn = tile >> 2;
    const int gt = dst * 16 + tile;
    const int m0 = tm * BM;
#define COLMAP 1
    const int nc0 = COLMAP == 0 ? tn * 128 : COLMAP == 1 ? tn * 64 : ((tn & 1) + (tn & 2) * 2) * 64;
    constexpr int NCH = COLMAP == 0 ? 64 : COLMAP == 1 ? 256 : 128;
    const int nst = isB ? NKT - KA : KA, kbase = isB ? KA : 0;
    char *myws, *part; GAS unsigned *tmo, *pflag;
    LAS unsigned* ctrl = (LAS unsigned*)(lds + CTRL);
    STAMP(0);

    f32x4 v[2][4];
    unsigned call, total = 0;
#define FETCH() do { \
        total = rfl(ctrl[16 + wid]); \
        const unsigned ca_ = ldsb + wid * CSTR + lane * 64, sw_ = (lane << 2) & 48; \
        _Pragma("unroll") for (int i = 0; i < 4; ++i) v[0][i] = *(LAS f32x4*)(uintptr_t)(ca_ + ((i * 16) ^ sw_)); \
        if (total > 1024u) { \
            _Pragma("unroll") for (int i = 0; i < 4; ++i) v[1][i] = *(LAS f32x4*)(uintptr_t)(ca_ + 4096 + ((i * 16) ^ sw_)); } \
    } while (0)

    if (__builtin_expect(wid >= 4, 1)) {
        const int lw = wid - 4;
        unsigned long long xp, wp;
        static_assert(offsetof(Args, x) == 0 && offsetof(Args, w) == 8, "");
        asm volatile("s_load_dwordx2 %0, %2, 0x0\n\ts_load_dwordx2 %1, %2, 0x8" : "=&s"(xp), "=&s"(wp) : "s"(ap));
        SB();
        unsigned vx[4], vw[4];
#pragma unroll
        for (int i = 0; i < 4; ++i) {
            const int row = 32 * lw + 8 * i + (lane >> 3);
            vx[i] = (unsigned)((m0 + row) * (KD * 2) + (((lane & 7) ^ ((row >> 1) & 7)) << 4));
            const int k = 16 * lw + 4 * i + (lane >> 4), f = ((k & 3) << 1) | (k & 8);
            const int ch = (lane & 15) ^ f;
            vw[i] = (unsigned)(k * (ND * 2) + (dst * NSH + nc0 + (ch >> 3) * NCH) * 2 + ((ch & 7) << 4));
        }
        const unsigned lx = ldsb + lw * 4096, lwb = ldsb + XBYTES + lw * 4096;
        int ktab;
        {
            const int ph = (dst + 2 * isB) & 3, rot = (tm + tn) & 3;
            int bs = lane < ph ? 0 : ph + ((lane - ph) & ~3);
            int sz = lane < ph ? ph : max(1, min(4, nst - bs)), r = rot;
            const int lb = isB ? LASTB : LASTA;
            if (lane >= nst - lb + ph) { bs = nst - lb + ph; sz = lb - ph; r = rot * (lb / 4); }
            ktab = kbase + bs + (lane - bs + r) % sz;
        }
#define ISSUE(t) do { \
            const unsigned kt_ = __builtin_amdgcn_readlane(ktab, t), so_ = (((t) + S0) & 3) * SLOT; \
            const unsigned sx_ = kt_ * (BK * 2), sw_ = kt_ * (BK * ND * 2); \
            if (ORDER == 0) { _Pragma("unroll") for (int i = 0; i < 4; ++i) { \
                GLDSW(rw, lwb + so_ + i * 1024, vw[i], sw_); GLDS(rx, lx + so_ + i * 1024, vx[i], sx_); } } \
            if (ORDER == 2) { _Pragma("unroll") for (int i = 0; i < 4; ++i) GLDS(rx, lx + so_ + i * 1024, vx[i], sx_); } \
            if (ORDER != 0) { _Pragma("unroll") for (int i = 0; i < 4; ++i) GLDSW(rw, lwb + so_ + i * 1024, vw[i], sw_); } \
            if (ORDER == 1) { _Pragma("unroll") for (int i = 0; i < 4; ++i) GLDS(rx, lx + so_ + i * 1024, vx[i], sx_); } } while (0)
#pragma unroll
        for (int i = 0; i < 4; ++i) asm volatile("" : "+v"(vx[i]), "+v"(vw[i]));
        asm volatile("" : "+v"(ktab));
        asm volatile("s_waitcnt lgkmcnt(0)" : "+s"(xp), "+s"(wp));
        __amdgpu_buffer_rsrc_t rx = __builtin_amdgcn_make_buffer_rsrc((void*)xp, 0, MLOC * KD * 2, 0x00020000);
        __amdgpu_buffer_rsrc_t rw = __builtin_amdgcn_make_buffer_rsrc((void*)wp, 0, KD * ND * 2, 0x00020000);
        ISSUE(0); ISSUE(1);
        ARGS_READY();
        WAIT_VM(8); BARRIER();
        if (!isB) STAMP(1);
        ISSUE(2);
        WAIT_VM(8); BARRIER();
        unsigned cv, fl = 0;
        asm volatile("global_load_dword %0, %1, off sc1" : "=v"(cv) : "v"(myws + WS_CNT + b * 4) : "memory");
        char* peerws;
        asm volatile("global_load_dwordx2 %0, %1, off" : "=v"(peerws)
                     : "v"((const char*)ap + offsetof(Args, ws) + (lane & 7) * 8) : "memory");
        ISSUE(3);
        for (int t = 0; t < nst - NS; ++t) {
            ISSUE(t + NS);
            if (t == nst - NS - 2) asm volatile("global_load_dword %0, %1, off sc0 sc1" : "=v"(fl) : "v"(pflag + lw * 32) : "memory");
            WAIT_VM(16); BARRIER();
            TR(t);
        }
        WAIT_VM(8);
        asm volatile("" : "+v"(fl), "+v"(cv), "+v"(peerws));
        call = rfl(cv) + 1;
        char* gp = peerws + WS_AMAX + (call & 1) * 8192 + (rank * NTILE + gt) * 8;
        asm volatile("" : "+v"(gp));
        if (!isB) {
            BARRIER(); WAIT_VM(0); BARRIER(); BARRIER();
            STAMP(2);
            if (lw == 0) {
                const char* dws = (const char*)ap->ws[dst];
                const char* tp = lane < 8 ? gp : lane < 16 ? dws + WS_DATA + (rank * 16 + tile) * 16384 + (lane & 3) * 4096
                                                          : dws + WS_MASK + (call & 1) * 524288 + (rank * 16 + tile) * 4096;
                unsigned dummy;
                asm volatile("global_load_dword %0, %1, off sc0 sc1\n\ts_waitcnt vmcnt(0)" : "=&v"(dummy) : "v"(tp) : "memory");
            }
        } else {
        unsigned bad = 0;
        {
            __amdgpu_buffer_rsrc_t rp = __builtin_amdgcn_make_buffer_rsrc((void*)part, 0, 65536, 0x00020000);
            const unsigned vp = lane * 16;
#define POLL_PART() asm volatile("global_load_dword %0, %1, off sc0 sc1" : "=v"(fl) : "v"(pflag + lw * 32) : "memory")
#define FETCH_HALF() do { STAMP(7); const unsigned hl_ = ldsb + nh * SLOT + lw * 8192, hs_ = lw * 16384 + nh * 8192; \
                _Pragma("unroll") for (int i = 0; i < 8; ++i) GLDS_SYS(rp, hl_ + i * 1024, vp, hs_ + i * 1024); ++nh; } while (0)
            bool seen = rfl(fl) >= call;
            unsigned nh = 0;
            if (seen) FETCH_HALF(); else POLL_PART();
            BARRIER();
            TR(nst - 4);
            if (seen) { FETCH_HALF(); WAIT_VM(16); }
            else {
                WAIT_VM(0); asm volatile("" : "+v"(fl)); seen = rfl(fl) >= call;
                if (seen) FETCH_HALF(); else POLL_PART();
            }
            BARRIER();
            TR(nst - 3);
            if (!seen) { WAIT_VM(0); asm volatile("" : "+v"(fl)); seen = rfl(fl) >= call; }
            if (seen && nh < 2) FETCH_HALF();
            BARRIER();
            TR(nst - 2);
            if (!seen) bad = wait_flag<1>(pflag + lw * 32, tmo, call);
#pragma nounroll
            while (nh < 2) FETCH_HALF();
            WAIT_VM(0); TR(nst - 1); BARRIER();
            TR(nst);
        }
        STAMP(2);
        BARRIER();
        if (lw == 0) {
            const u32x4 m4 = *(LAS u32x4*)ctrl;
            const u32x2 gr = {max(max(m4.x, m4.y), max(m4.z, m4.w)), call};
            if (lane < NR) asm volatile("global_store_dwordx2 %0, %1, off sc0 sc1\n\ts_nop 1" :: "v"(gp), "v"(gr) : "memory");
            STAMP(5);
        }
        BARRIER();
        STAMP(1);
        FETCH();
        STAMP(6);
        const char* g = myws + WS_AMAX + (call & 1) * 8192 + lw * 2048 + lane * 16;
        unsigned mx = 0;
        const unsigned long long t0 = __builtin_amdgcn_s_memrealtime();
        unsigned polls = 0;
        for (;;) {
            u32x4 a, c;
            asm volatile("global_load_dwordx4 %0, %2, off sc0 sc1\n\t"
                         "global_load_dwordx4 %1, %2, off offset:1024 sc0 sc1\n\t"
                         "s_waitcnt vmcnt(0)"
                         : "=&v"(a), "=&v"(c) : "v"(g) : "memory");
            const bool ok = a.y == call && a.w == call && c.y == call && c.w == call;
            mx = max(max(a.x, a.z), max(c.x, c.z));
            if (__all(ok)) break;
            if (give_up(tmo, call, t0, polls)) { bad = 1; break; }
            __builtin_amdgcn_s_sleep(1);
        }
        mx = wave_max_u32(mx);
        if (lane == 0) { ctrl[8 + lw] = mx; ctrl[12 + lw] = bad; }
        STAMP(3);
        }
    } else {
        const int wm = wid >> 1, wn = wid & 1;
        const int fr = lane & 15, fq = lane >> 4, q = (lane & 15) >> 2, p = lane & 3;
        if (b < 64) {
            unsigned long long pc; unsigned len, rel;
            asm volatile("s_getpc_b64 %0\n.Lgemm_a2a_here:\n\t"
                         "s_mov_b32 %1, .Lgemm_a2a_here-_ZN12_GLOBAL__N_115gemm_a2a_kernelENS_4ArgsE\n\t"
                         "s_mov_b32 %2, .Lgemm_a2a_e1-_ZN12_GLOBAL__N_115gemm_a2a_kernelENS_4ArgsE\n\t"
                         "s_max_u32 %2, %2, .Lgemm_a2a_e2-_ZN12_GLOBAL__N_115gemm_a2a_kernelENS_4ArgsE" : "=s"(pc), "=s"(rel), "=s"(len) :: "scc");
            pc -= rel;
            const unsigned off = (((b >> 3) * 4 + wid) * 8 + (lane & 7)) * 128;
            if (lane < 8 && off < len) {
                unsigned dummy;
                asm volatile("global_load_dword %0, %1, off\n\ts_waitcnt vmcnt(0)" : "=&v"(dummy) : "v"((const char*)pc + off) : "memory");
            }
        }
        ARGS_READY();
        unsigned callv = __hip_atomic_load((GAS unsigned*)(myws + WS_CNT) + b, __ATOMIC_RELAXED, __HIP_MEMORY_SCOPE_AGENT);
        char* mptr = (char*)ap->ws[dst] + WS_MASK + (((rank * 16 + tile) * 8 + wid * 2) * 64 + lane) * 8;
        unsigned ax[2][2], aw[2][4];
#pragma unroll
        for (int ks = 0; ks < 2; ++ks) {
            const int row = wm * 64 + fr;
            ax[0][ks] = ldsb + row * 128 + (((ks * 4 + fq) ^ ((row >> 1) & 7)) << 4);
            ax[1][ks] = ax[0][ks] + 2 * SLOT;
            asm volatile("" : "+v"(ax[0][ks]), "+v"(ax[1][ks]));
        }
#pragma unroll
        for (int c = 0; c < 4; ++c) {
            const int k = 8 * fq + q, f = ((k & 3) << 1) | (k & 8);
            aw[0][c] = ldsb + XBYTES + k * 256 + (((wn * 8 + 2 * c + (p >> 1)) ^ f) << 4) + (p & 1) * 8;
            aw[1][c] = aw[0][c] + 2 * SLOT;
            asm volatile("" : "+v"(aw[0][c]), "+v"(aw[1][c]));
        }
        f32x4 acc[16];
#pragma unroll
        for (int i = 0; i < 16; ++i) { acc[i] = (f32x4){0.f, 0.f, 0.f, 0.f}; asm volatile("" : "+v"(acc[i])); }
        asm volatile("" : "+v"(mptr));
        Frag f0, f1;
        BARRIER();
        rd_all<S0, 0>(f0, ax, aw);
        WAIT_LGKM0(); BARRIER();
        asm volatile("s_nop 1");
        if constexpr (S0 != 0) { kstep<3, 0>(acc, f0, f1, ax, aw); kstep<0, 0>(acc, f1, f0, ax, aw); }
        for (int t = S0; t < nst - 4; t += 4) {
            kstep<1, 0>(acc, f0, f1, ax, aw); kstep<2, 0>(acc, f1, f0, ax, aw);
            kstep<3, 0>(acc, f0, f1, ax, aw); kstep<0, 0>(acc, f1, f0, ax, aw);
        }
        call = rfl(callv) + 1;
        SB();
        kstep<1, 0>(acc, f0, f1, ax, aw); kstep<2, 0>(acc, f1, f0, ax, aw); kstep<3, 0>(acc, f0, f1, ax, aw);
        klast<0, BPG>(acc, f1);
        f32x4 pa[16];
        if (isB) {
            BARRIER();
#pragma unroll
            for (int i = 0; i < 16; ++i) pa[i] = *(LAS f32x4*)(lds + (i < 8 ? 0 : SLOT) + wid * 8192 + (i & 7) * 1024 + lane * 16);
        } else {
#pragma unroll
            for (int i = 0; i < 16; ++i) pa[i] = (f32x4){0.f, 0.f, 0.f, 0.f};
        }
        SB();
        klast<BPG, 32>(acc, f1);
        asm volatile("s_nop 15" ::: "memory");
#pragma unroll
        for (int i = 0; i < 16; ++i) asm volatile("" : "+v"(acc[i]));
        STAMP(2);
        if (!isB) {
            char* pp = part + wid * 16384 + lane * 16;
#pragma unroll
            for (int i = 0; i < 16; ++i)
                asm volatile("global_store_dwordx4 %0, %1, off offset:%2 " PART_POL "\n\ts_nop 1"
                             :: "v"(pp + (i >> 2) * 4096), "v"(acc[i]), "n"((i & 3) * 1024) : "memory");
            WAIT_VM(0);
            if (lane == 0) __hip_atomic_store(pflag + wid * 32, call, __ATOMIC_RELAXED, __HIP_MEMORY_SCOPE_SYSTEM);
            STAMP(3);
        } else {
        float mx4[4] = {0.f, 0.f, 0.f, 0.f};
#pragma unroll
        for (int i = 0; i < 16; ++i) acc[i] += pa[i];
#pragma unroll
        for (int i = 0; i < 16; i += 2)
#pragma unroll
            for (int e = 0; e < 4; ++e) mx4[e] = __builtin_fmaxf(__builtin_fmaxf(acc[i][e], acc[i + 1][e]), mx4[e]);
        const float mxf = fmaxf(fmaxf(mx4[0], mx4[1]), fmaxf(mx4[2], mx4[3]));
        const unsigned wmx = wave_max_u32(__float_as_uint(mxf) & 0x7fffffffu);
        if (lane == 0) ctrl[wid] = wmx;
        WAIT_LGKM0(); BARRIER();
        STAMP(3);
        const unsigned trash = ldsb + 8 * CSTR + wid * 512 + lane * 4;
        unsigned mq[8];
#pragma unroll
        for (int c = 0; c < 8; ++c) mq[c] = __float_as_uint(acc[2 * c + 1][3]) >> 31;
#pragma unroll
        for (int e = 6; e >= 0; --e)
#pragma unroll
            for (int c = 0; c < 8; ++c) mq[c] = __builtin_amdgcn_alignbit(mq[c], __float_as_uint(acc[2 * c + (e >> 2)][e & 3]), 31);
        unsigned wa[16];
#pragma unroll
        for (int hh = 0; hh < 2; ++hh) {
            const unsigned m = ~(mq[hh * 4] | (mq[hh * 4 + 1] << 8) | (mq[hh * 4 + 2] << 16) | (mq[hh * 4 + 3] << 24));
            const u32x2 mg = {m, call};
            asm volatile("global_store_dwordx2 %0, %1, off offset:%2 sc0 sc1\n\ts_nop 1" :: "v"(mptr + (call & 1) * 524288), "v"(mg), "n"(hh * 512) : "memory");
            unsigned pos[8];
            const unsigned tot = stream_offsets(m, pos);
            if (lane == 0) ctrl[16 + wid * 2 + hh] = tot;
#pragma unroll
            for (int g = 0; g < 8; ++g) wa[hh * 8 + g] = ldsb + (wid * 2 + hh) * CSTR + pos[g] * 4;
        }
#pragma unroll
        for (int e = 0; e < 4; ++e) {
            unsigned sg[16], ad[16];
#pragma unroll
            for (int i = 0; i < 16; ++i) asm("v_ashrrev_i32 %0, 31, %1" : "=v"(sg[i]) : "v"(acc[i][e]));
#pragma unroll
            for (int i = 0; i < 16; ++i) asm("v_bfi_b32 %0, %1, %2, %3" : "=v"(ad[i]) : "v"(sg[i]), "v"(trash), "v"(wa[i]));
#pragma unroll
            for (int i = 0; i < 16; ++i) {
                *(LAS float*)(uintptr_t)(ad[i] + 4 * e) = acc[i][e];
                if (e < 3) asm("v_lshl_add_u32 %0, %1, 2, %0" : "+v"(wa[i]) : "v"(sg[i]));
            }
        }
        asm volatile(".Lgemm_a2a_e2:");
        WAIT_LGKM0(); BARRIER();
        STAMP(1);
        FETCH();
        STAMP(6);
        }
    }
    const unsigned tag = call % 255u + 1u, tg0 = tag_bits(tag & 15u), tg1 = tag_bits(tag >> 4);
    if (isB) {
        char* dptr = (char*)ap->ws[dst] + WS_DATA + ((rank * 16 + tile) * 8 + wid) * 2048 + lane * 16;
        asm volatile("" : "+v"(dptr));
        WAIT_LGKM0(); BARRIER();
        STAMP(4);
        if (tid == 0) __hip_atomic_store((GAS unsigned*)(myws + WS_GO + gt * 128), call, __ATOMIC_RELAXED, __HIP_MEMORY_SCOPE_AGENT);
        const u32x4 g4 = *(LAS u32x4*)(ctrl + 8), b4 = *(LAS u32x4*)(ctrl + 12);
        const float amax = __uint_as_float(max(max(g4.x, g4.y), max(g4.z, g4.w)));
        const bool timed_out = (b4.x | b4.y | b4.z | b4.w) != 0;
        float inv = 448.0f * __builtin_amdgcn_rcpf(amax);
        if (__builtin_expect(!(amax > 1e-30f && amax < 1e30f), 0)) inv = amax > 0.f ? 1.0f / (amax / 448.0f) : 0.f;
#pragma unroll
        for (int h = 0; h < 2; ++h) {
            if (h * 1024u < total) {
                u32x4 o;
#pragma unroll
                for (int i = 0; i < 4; ++i) {
                    int r = __builtin_amdgcn_cvt_pk_fp8_f32(fminf(v[h][i][0] * inv, 448.f), fminf(v[h][i][1] * inv, 448.f), 0, false);
                    r = __builtin_amdgcn_cvt_pk_fp8_f32(fminf(v[h][i][2] * inv, 448.f), fminf(v[h][i][3] * inv, 448.f), r, true);
                    o[i] = ((timed_out ? 0x7f7f7f7fu : (unsigned)r) & 0x7f7f7f7fu) | ((i & 1) ? tg1 : tg0);
                }
                if (h * 1024u + lane * 16u < total)
                    asm volatile("global_store_dwordx4 %0, %1, off offset:%2 sc0 sc1\n\ts_nop 1" :: "v"(dptr), "v"(o), "n"(h * 1024) : "memory");
            }
        }
        if (wid < 4) STAMP(5);
    } else {
        bool bad = false;
        unsigned m;
        {
            const char* mp = myws + WS_MASK + (call & 1) * 524288 + ((gt * 8 + wid) * 64 + lane) * 8;
            const unsigned long long t0 = __builtin_amdgcn_s_memrealtime();
    unsigned polls = 0;
            for (;;) {
                u32x2 g;
                asm volatile("global_load_dwordx2 %0, %1, off sc0 sc1\n\ts_waitcnt vmcnt(0)" : "=&v"(g) : "v"(mp) : "memory");
                m = g.x;
                if (__all(g.y == call)) break;
                if (give_up(tmo, call, t0, polls)) { bad = true; m = 0; break; }
                __builtin_amdgcn_s_sleep(MASK_SLEEP);
            }
        }
        STAMP(4);
        unsigned pos[8];
        const unsigned tot = stream_offsets(m, pos);
        const unsigned sbase = ldsb + wid * 8192;
#pragma unroll
        for (int g = 0; g < 8; ++g) *(LAS unsigned*)(uintptr_t)(sbase + 4096 + g * 256 + lane * 4) = pos[g] * 16 + ((m >> (4 * g)) & 15u);
        unsigned ra[8], rb[8], sel[8];
#pragma unroll
        for (int j = 0; j < 8; ++j)
            sel[j] = *(LAS unsigned*)(uintptr_t)(sbase + 4096 + ((j >> 2) * 4 + (lane & 3)) * 256 + ((j & 3) * 16 + (lane >> 2)) * 4);
#pragma unroll
        for (int j = 0; j < 8; ++j) {
            const unsigned e = sel[j], d0 = e >> 6, d1 = d0 + 1;
            const unsigned sp = ((e & 15u) * 0x00204081u) & 0x01010101u, mk = (sp << 8) - sp, s2 = sp + (sp << 8);
            ra[j] = sbase + (d0 ^ ((d0 >> 4) & 3u)) * 4;
            rb[j] = sbase + (d1 ^ ((d1 >> 4) & 3u)) * 4;
            sel[j] = ((((s2 + (sp << 16)) << 8) + __builtin_amdgcn_perm(0u, (e >> 4) & 3u, 0u)) & mk) | (0x0c0c0c0cu & ~mk);
        }
#pragma unroll
        for (int j = 0; j < 8; ++j) asm volatile("" : "+v"(ra[j]), "+v"(rb[j]), "+v"(sel[j]));
        char* optr = (char*)ap->myout + (size_t)(dst * MLOC + m0 + (wid >> 2) * 64 + (wid & 1) * 32 + (lane >> 2)) * NSH
                     + nc0 + ((wid >> 1) & 1) * NCH + (lane & 3) * 16;
        char* sp = myws + WS_DATA + (gt * 8 + wid) * 2048 + lane * 16;
        asm volatile("" : "+v"(optr), "+v"(sp));
        if (!bad) bad = wait_flag<1>((GAS unsigned*)(myws + WS_GO + gt * 128), tmo, call);
        STAMP(5);
        const bool n0 = lane * 16u < tot, n1 = 1024u + lane * 16u < tot;
        u32x4 d0 = {0, 0, 0, 0}, d1 = {0, 0, 0, 0};
        if (!bad) {
            const unsigned long long t0 = __builtin_amdgcn_s_memrealtime();
    unsigned polls = 0;
            unsigned long long p0 = __ballot(n0), p1 = __ballot(n1);
            unsigned long long q0 = p0, q1 = p1;
            while (p0 | p1) {
                asm volatile("s_mov_b64 exec, %3\n\tglobal_load_dwordx4 %0, %2, off sc0 sc1\n\t"
                             "s_mov_b64 exec, %4\n\tglobal_load_dwordx4 %1, %2, off offset:1024 sc0 sc1\n\t"
                             "s_mov_b64 exec, -1\n\ts_waitcnt vmcnt(0)"
                             : "+v"(d0), "+v"(d1) : "v"(sp), "s"(q0), "s"(q1) : "memory");
                const unsigned long long b0 = __ballot((((d0.x & 0x80808080u) ^ tg0) | ((d0.y & 0x80808080u) ^ tg1) | ((d0.z & 0x80808080u) ^ tg0) | ((d0.w & 0x80808080u) ^ tg1)) != 0);
                const unsigned long long b1 = __ballot((((d1.x & 0x80808080u) ^ tg0) | ((d1.y & 0x80808080u) ^ tg1) | ((d1.z & 0x80808080u) ^ tg0) | ((d1.w & 0x80808080u) ^ tg1)) != 0);
                p0 &= b0 | ~q0; p1 &= b1 | ~q1;
                q0 &= b0; q1 &= b1;
                if ((q0 | q1) == 0) { q0 = p0; q1 = p1; continue; }
                if (give_up(tmo, call, t0, polls)) { bad = true; break; }
                __builtin_amdgcn_s_sleep(POLL_SLEEP);
            }
        }
        STAMP(7);
        *(LAS u32x4*)(uintptr_t)(sbase + lane * 16) = d0 & 0x7f7f7f7fu;
        *(LAS u32x4*)(uintptr_t)(sbase + 1024 + lane * 16) = d1 & 0x7f7f7f7fu;
        const unsigned poison = bad ? 0x7f7f7f7fu : 0u;
        unsigned lo[8], hi[8];
#pragma unroll
        for (int g = 0; g < 8; ++g) { lo[g] = *(LAS unsigned*)(uintptr_t)ra[g]; hi[g] = *(LAS unsigned*)(uintptr_t)rb[g]; }
#pragma unroll
        for (int h = 0; h < 2; ++h) {
            u32x4 o;
#pragma unroll
            for (int i = 0; i < 4; ++i) o[i] = __builtin_amdgcn_perm(hi[h * 4 + i], lo[h * 4 + i], sel[h * 4 + i]) | poison;
            asm volatile("global_store_dwordx4 %0, %1, off " OUT_POL "\n\ts_nop 1" :: "v"(optr + h * 16 * NSH), "v"(o) : "memory");
        }
        const u32x4 z = {0, 0, 0, 0};
        if (n0) asm volatile("global_store_dwordx4 %0, %1, off sc0 sc1\n\ts_nop 1" :: "v"(sp), "v"(z) : "memory");
        if (n1) asm volatile("global_store_dwordx4 %0, %1, off offset:1024 sc0 sc1\n\ts_nop 1" :: "v"(sp), "v"(z) : "memory");
        WAIT_VM(0);
        STAMP(6);
    }
    asm volatile(".Lgemm_a2a_e1:");
    if (tid == 256) __hip_atomic_store((GAS unsigned*)(myws + WS_CNT) + b, call, __ATOMIC_RELAXED, __HIP_MEMORY_SCOPE_AGENT);
    STAMP_OUT();
}

}

extern "C" void kernel_launch(int rank, int nranks, ncclComm_t comm,
                              hipStream_t stream,
                              void* const* d_in, const int* in_sizes, int n_in,
                              void* d_out, int out_size,
                              const int* in_dtypes, int out_dtype,
                              void* const* d_out_all,
                              void* const* d_ws_all, size_t ws_bytes) {
    if (nranks != NR || n_in < 2 || in_sizes[0] != MLOC * KD || in_sizes[1] != KD * ND ||
        out_size != ND * NSH || ws_bytes < WS_END || (unsigned)rank >= (unsigned)NR) {
        fprintf(stderr, "kernel_launch: unexpected problem shape\n");
        return;
    }
    HIP_CHECK(hipFuncSetAttribute((const void*)gemm_a2a_kernel, hipFuncAttributeMaxDynamicSharedMemorySize, LDS_BYTES));
    Args a;
    a.x = d_in[0]; a.w = d_in[1]; a.myws = d_ws_all[rank]; a.myout = d_out; a.rank = rank; a.pad = 0;
    for (int p = 0; p < NR; ++p) { a.out[p] = d_out_all[p]; a.ws[p] = d_ws_all[p]; }
    hipLaunchKernelGGL(gemm_a2a_kernel, dim3(NWG), dim3(512), LDS_BYTES, stream, a);
    HIP_CHECK(hipGetLastError());
}
```

```cpp
#include <hip/hip_runtime.h>
#include <rccl/rccl.h>
#include <cstdio>
#include <cstdint>
#include <cstddef>

#define HIP_CHECK(expr) do { hipError_t _e = (expr); if (_e != hipSuccess) { \
    fprintf(stderr, "HIP error %s at %s:%d\n", hipGetErrorString(_e), __FILE__, __LINE__); \
    return; } } while (0)

namespace {

typedef short          s16x4 __attribute__((ext_vector_type(4)));
typedef short          s16x8 __attribute__((ext_vector_type(8)));
typedef float          f32x4 __attribute__((ext_vector_type(4)));
typedef unsigned       u32x4 __attribute__((ext_vector_type(4)));
typedef unsigned       u32x2 __attribute__((ext_vector_type(2)));
#define LAS __attribute__((address_space(3)))
#define GAS __attribute__((address_space(1)))

constexpr int NR = 8;
constexpr int MLOC = 512, KD = 4096, ND = 4096, NSH = 512;
constexpr int BM = 128, BN = 128, BK = 64;
constexpr int NKT = KD / BK;
constexpr int KA = 28;
constexpr int LASTA = 12, LASTB = 16;
constexpr int XBYTES = BM * BK * 2;
constexpr int SLOT = XBYTES + BK * BN * 2;
constexpr int NS = 4;
constexpr int CTRL = NS * SLOT;
constexpr int LDS_BYTES = CTRL + 256;
constexpr int NWG = 256, NTILE = 128;
constexpr int CSTR = 8192;
static_assert(KA % 4 == 0 && KA >= 8 && NKT - KA >= 8, "");

constexpr size_t WS_CNT   = 0;
constexpr size_t WS_TMO   = 4096;
constexpr size_t WS_AMAX  = 8192;
constexpr size_t WS_GO    = WS_AMAX + 2 * 8192;
constexpr size_t WS_PFLAG = WS_GO + NTILE * 128;
constexpr size_t WS_MASK  = 1 << 20;
constexpr size_t WS_DATA  = 2 << 20;
constexpr size_t WS_PART  = 4 << 20;
constexpr size_t WS_END   = WS_PART + (size_t)NTILE * 65536;
static_assert(WS_PFLAG + NTILE * 512 <= WS_MASK, "");
constexpr unsigned long long DEADLINE = 100000000ull;

#define BARRIER() do { asm volatile("" ::: "memory"); __builtin_amdgcn_s_barrier(); asm volatile("" ::: "memory"); } while (0)
#define WAIT_VM(n) asm volatile("s_waitcnt vmcnt(%0)" :: "n"(n) : "memory")
#define WAIT_LGKM0() __builtin_amdgcn_s_waitcnt(0xC07F)
#define SB() __builtin_amdgcn_sched_barrier(0)

__device__ __forceinline__ unsigned rfl(unsigned v) { return __builtin_amdgcn_readfirstlane(v); }

__device__ __forceinline__ unsigned wave_max_u32(unsigned v) {
    v = max(v, (unsigned)__builtin_amdgcn_update_dpp(0, (int)v, 0xB1, 0xF, 0xF, true));
    v = max(v, (unsigned)__builtin_amdgcn_update_dpp(0, (int)v, 0x4E, 0xF, 0xF, true));
    v = max(v, (unsigned)__builtin_amdgcn_update_dpp(0, (int)v, 0x141, 0xF, 0xF, true));
    v = max(v, (unsigned)__builtin_amdgcn_update_dpp(0, (int)v, 0x140, 0xF, 0xF, true));
    unsigned a = __builtin_amdgcn_readlane(v, 0), b = __builtin_amdgcn_readlane(v, 16);
    unsigned c = __builtin_amdgcn_readlane(v, 32), d = __builtin_amdgcn_readlane(v, 48);
    return max(max(a, b), max(c, d));
}

__device__ __forceinline__ unsigned wave_scan_u32(unsigned v) {
    v += (unsigned)__builtin_amdgcn_update_dpp(0, (int)v, 0x111, 0xF, 0xF, true);
    v += (unsigned)__builtin_amdgcn_update_dpp(0, (int)v, 0x112, 0xF, 0xF, true);
    v += (unsigned)__builtin_amdgcn_update_dpp(0, (int)v, 0x114, 0xF, 0xF, true);
    v += (unsigned)__builtin_amdgcn_update_dpp(0, (int)v, 0x118, 0xF, 0xF, true);
    v += (unsigned)__builtin_amdgcn_update_dpp(0, (int)v, 0x142, 0xA, 0xF, false);
    v += (unsigned)__builtin_amdgcn_update_dpp(0, (int)v, 0x143, 0xC, 0xF, false);
    return v;
}
__device__ __forceinline__ unsigned stream_offsets(unsigned m, unsigned (&pos)[8]) {
    unsigned c = m - ((m >> 1) & 0x55555555u);
    c = (c & 0x33333333u) + ((c >> 2) & 0x33333333u);
    unsigned ex[4], tot[4];
#pragma unroll
    for (int j = 0; j < 4; ++j) {
        const unsigned f = (c >> (4 * j)) & 0x000f000fu, inc = wave_scan_u32(f);
        tot[j] = __builtin_amdgcn_readlane(inc, 63); ex[j] = inc - f;
    }
    unsigned base = 0;
#pragma unroll
    for (int g = 0; g < 8; ++g) {
        pos[g] = base + (g < 4 ? ex[g] & 0xffffu : ex[g - 4] >> 16);
        base += g < 4 ? tot[g] & 0xffffu : tot[g - 4] >> 16;
    }
    return base;
}
__device__ __forceinline__ unsigned tag_bits(unsigned n) { return ((n * 0x00204081u) & 0x01010101u) << 7; }

__device__ __forceinline__ bool give_up(GAS unsigned* tmo, unsigned call, unsigned long long t0, unsigned& polls) {
    if (__builtin_expect((++polls & 31u) != 0u, 1)) return false;
    if (rfl(__hip_atomic_load(tmo, __ATOMIC_RELAXED, __HIP_MEMORY_SCOPE_AGENT)) == call ||
        __builtin_amdgcn_s_memrealtime() - t0 > DEADLINE) {
        __hip_atomic_store(tmo, call, __ATOMIC_RELAXED, __HIP_MEMORY_SCOPE_AGENT);
        return true;
    }
    return false;
}

__device__ __forceinline__ bool wait_flag(GAS unsigned* flag, GAS unsigned* tmo, unsigned call) {
    const unsigned long long t0 = __builtin_amdgcn_s_memrealtime();
    unsigned polls = 0;
    while (rfl(__hip_atomic_load(flag, __ATOMIC_RELAXED, __HIP_MEMORY_SCOPE_SYSTEM)) < call) {
        if (give_up(tmo, call, t0, polls)) return true;
        __builtin_amdgcn_s_sleep(1);
    }
    return false;
}

#define GLDS(rsrc, ldsaddr, voff, soff) \
    asm volatile("s_mov_b32 m0, %0\n\ts_nop 0\n\tbuffer_load_dwordx4 %1, %2, %3 offen lds" \
                 :: "s"(ldsaddr), "v"(voff), "s"(rsrc), "s"(soff) : "memory")
#define GLDS_SYS(rsrc, ldsaddr, voff, soff) \
    asm volatile("s_mov_b32 m0, %0\n\ts_nop 0\n\tbuffer_load_dwordx4 %1, %2, %3 offen sc0 sc1 lds" \
                 :: "s"(ldsaddr), "v"(voff), "s"(rsrc), "s"(soff) : "memory")

#define STAMP(i) do {} while (0)
#define TR(i) do {} while (0)
#define STAMP_OUT() do {} while (0)

struct Frag { s16x8 x[8]; s16x4 w[16]; };

template <int SL, int R>
__device__ __forceinline__ void rd(Frag& f, const unsigned (&ax)[2][2], const unsigned (&aw)[2][4]) {
    constexpr int g = R / 3, e = R % 3, so = (SL & 1) * SLOT, ks = g >> 2, i = g & 3;
    if constexpr (e == 0) f.x[g] = *(LAS s16x8*)(uintptr_t)(ax[SL >> 1][ks] + so + i * 2048);
    else f.w[g * 2 + e - 1] = __builtin_amdgcn_ds_read_tr16_b64_v4i16(
             (LAS s16x4*)(uintptr_t)(aw[SL >> 1][i] + so + ks * 8192 + (e - 1) * 1024));
}
template <int SL, int R>
__device__ __forceinline__ void rd_all(Frag& f, const unsigned (&ax)[2][2], const unsigned (&aw)[2][4]) {
    if constexpr (R < 24) { rd<SL, R>(f, ax, aw); rd_all<SL, R + 1>(f, ax, aw); }
}
template <int G, bool PAD = false>
__device__ __forceinline__ void mfma(f32x4 (&acc)[16], const Frag& cur) {
    constexpr int ks = G >> 4, c = (G >> 2) & 3, mi = G & 3;
    const s16x8 wf = __builtin_shufflevector(cur.w[(ks * 4 + c) * 2], cur.w[(ks * 4 + c) * 2 + 1], 0, 1, 2, 3, 4, 5, 6, 7);
    if constexpr (PAD)
        asm volatile("s_nop 1\n\tv_mfma_f32_16x16x32_bf16 %0, %1, %2, %0" : "+v"(acc[mi * 4 + c]) : "v"(wf), "v"(cur.x[ks * 4 + mi]));
    else
        asm volatile("v_mfma_f32_16x16x32_bf16 %0, %1, %2, %0" : "+v"(acc[mi * 4 + c]) : "v"(wf), "v"(cur.x[ks * 4 + mi]));
}
template <int G>
__device__ __forceinline__ void give(f32x4 (&acc)[16], char* pp) {
    constexpr int i = (G & 3) * 4 + ((G >> 2) & 3);
    asm volatile("global_store_dwordx4 %0, %1, off offset:%2 sc0 sc1\n\ts_nop 1"
                 :: "v"(pp + (i >> 2) * 4096), "v"(acc[i]), "n"((i & 3) * 1024) : "memory");
}
template <int SL, int G>
__device__ __forceinline__ void kstep(f32x4 (&acc)[16], const Frag& cur, Frag& nxt,
                                      const unsigned (&ax)[2][2], const unsigned (&aw)[2][4]) {
    if constexpr (G < 32) {
        mfma<G>(acc, cur);
        if constexpr (G < 24) rd<SL, G>(nxt, ax, aw);
        if constexpr (G == 31) { SB(); WAIT_LGKM0(); BARRIER(); }
        SB();
        kstep<SL, G + 1>(acc, cur, nxt, ax, aw);
    }
}
template <int G, int END, bool PAD>
__device__ __forceinline__ void klast(f32x4 (&acc)[16], const Frag& cur) {
    if constexpr (G < END) { mfma<G, PAD>(acc, cur); SB(); klast<G + 1, END, PAD>(acc, cur); }
}
template <int G>
__device__ __forceinline__ void klast_give(f32x4 (&acc)[16], const Frag& cur, char* pp) {
    if constexpr (G < 32) mfma<G, true>(acc, cur);
    if constexpr (G == 32) asm volatile("s_nop 7" ::: "memory");
    if constexpr (G >= 20) give<G - 4>(acc, pp);
    SB();
    if constexpr (G < 35) klast_give<G + 1>(acc, cur, pp);
}

struct Args {
    const void* x; const void* w;
    void* myws; void* myout;
    void* ws[NR];
    int rank, pad;
};

__global__ void __launch_bounds__(512, 2)
gemm_a2a_kernel(const Args args)
{
    typedef __attribute__((address_space(4))) const Args* ArgP;
    ArgP ap = (ArgP)__builtin_amdgcn_kernarg_segment_ptr();
    unsigned long long xp, wp;
    static_assert(offsetof(Args, x) == 0 && offsetof(Args, w) == 8, "");
    asm volatile(".Lgemm_a2a_top:\n\ts_load_dwordx2 %0, %2, 0x0\n\ts_load_dwordx2 %1, %2, 0x8" : "=&s"(xp), "=&s"(wp) : "s"(ap));
    SB();
    int rank;
#define ARGS_READY() \
    asm volatile("" : "+s"(ap)); \
    rank = ap->rank; myws = (char*)ap->myws; tmo = (GAS unsigned*)(myws + WS_TMO); \
    pflag = (GAS unsigned*)(myws + WS_PFLAG + gt * 512); part = myws + WS_PART + (size_t)gt * 65536
    extern __shared__ __attribute__((aligned(16))) char smem[];
    LAS char* lds = (LAS char*)smem;
    const unsigned ldsb = (unsigned)(uintptr_t)lds;
    const int tid = threadIdx.x, lane = tid & 63;
    const int wid = rfl(tid >> 6);
    const int b = blockIdx.x;
    const int dst = b & 7;
    const int isB = (b >> 3) & 1, tile = b >> 4, tm = tile & 3, tn = tile >> 2;
    const int gt = dst * 16 + tile;
    const int m0 = tm * BM;
    const int nc0 = tn * BN;
    const int nst = isB ? NKT - KA : KA, kbase = isB ? KA : 0;
    char *myws, *part; GAS unsigned *tmo, *pflag;
    LAS unsigned* ctrl = (LAS unsigned*)(lds + CTRL);
    STAMP(0);

    f32x4 v[2][4];
    unsigned call, total = 0;
#define FETCH() do { \
        total = rfl(ctrl[16 + wid]); \
        const unsigned ca_ = ldsb + wid * CSTR + lane * 64, sw_ = (lane << 2) & 48; \
        _Pragma("unroll") for (int i = 0; i < 4; ++i) v[0][i] = *(LAS f32x4*)(uintptr_t)(ca_ + ((i * 16) ^ sw_)); \
        if (total > 1024u) { \
            _Pragma("unroll") for (int i = 0; i < 4; ++i) v[1][i] = *(LAS f32x4*)(uintptr_t)(ca_ + 4096 + ((i * 16) ^ sw_)); } \
    } while (0)

    if (__builtin_expect(wid >= 4, 1)) {
        const int lw = wid - 4;
        const int ph = (dst + 2 * isB) & 3, rot = (tm + tn) & 3;
        const unsigned k0 = kbase + ((0x244400E4u >> (ph * 8 + rot * 2)) & 3);
        const unsigned lx = ldsb + lw * 4096, lwb = ldsb + XBYTES + lw * 4096;
        unsigned vx[4], vw[4];
#pragma unroll
        for (int i = 0; i < 4; ++i) {
            const int k = 16 * lw + 4 * i + (lane >> 4), f = ((k & 3) << 1) | (k & 8);
            vw[i] = (unsigned)(k * (ND * 2) + (dst * NSH + nc0) * 2 + (((lane & 15) ^ f) << 4));
            asm volatile("" : "+v"(vw[i]));
        }
        asm volatile("s_waitcnt lgkmcnt(0)" : "+s"(xp), "+s"(wp));
        __amdgpu_buffer_rsrc_t rx = __builtin_amdgcn_make_buffer_rsrc((void*)xp, 0, MLOC * KD * 2, 0x00020000);
        __amdgpu_buffer_rsrc_t rw = __builtin_amdgcn_make_buffer_rsrc((void*)wp, 0, KD * ND * 2, 0x00020000);
#pragma unroll
        for (int i = 0; i < 4; ++i) GLDS(rw, lwb + i * 1024, vw[i], k0 * (BK * ND * 2));
        SB();
#pragma unroll
        for (int i = 0; i < 4; ++i) {
            const int row = 32 * lw + 8 * i + (lane >> 3);
            vx[i] = (unsigned)((m0 + row) * (KD * 2) + (((lane & 7) ^ ((row >> 1) & 7)) << 4));
            GLDS(rx, lx + i * 1024, vx[i], k0 * (BK * 2));
        }
        SB();
        int ktab;
        {
            const int ls = ph + (isB ? NKT - KA - LASTB : KA - LASTA);
            const int bs = lane < ph ? 0 : lane >= ls ? ls : ph + ((lane - ph) & ~3);
            const int sz = lane < ph ? ph : lane >= ls ? nst - ls : 4;
            ktab = kbase + bs + (lane - bs + rot * ((sz + 3) >> 2)) % sz;
        }
#define ISSUE(t) do { \
            const unsigned kt_ = __builtin_amdgcn_readlane(ktab, t), so_ = ((t) & 3) * SLOT; \
            _Pragma("unroll") for (int i = 0; i < 4; ++i) GLDS(rw, lwb + so_ + i * 1024, vw[i], kt_ * (BK * ND * 2)); \
            _Pragma("unroll") for (int i = 0; i < 4; ++i) GLDS(rx, lx + so_ + i * 1024, vx[i], kt_ * (BK * 2)); } while (0)
        ISSUE(1);
        ARGS_READY();
        WAIT_VM(8); BARRIER();
        if (!isB) STAMP(1);
        ISSUE(2);
        WAIT_VM(8); BARRIER();
        unsigned cv, fl = 0;
        asm volatile("global_load_dword %0, %1, off sc1" : "=v"(cv) : "v"(myws + WS_CNT + b * 4) : "memory");
        char* peerws;
        asm volatile("global_load_dwordx2 %0, %1, off" : "=v"(peerws)
                     : "v"((const char*)ap + offsetof(Args, ws) + (lane & 7) * 8) : "memory");
        ISSUE(3);
        for (int t = 0; t < nst - NS; ++t) {
            ISSUE(t + NS);
            if (t == nst - NS - 2) asm volatile("global_load_dword %0, %1, off sc0 sc1" : "=v"(fl) : "v"(pflag + lw * 32) : "memory");
            WAIT_VM(16); BARRIER();
            TR(t);
        }
        WAIT_VM(8);
        asm volatile("" : "+v"(fl), "+v"(cv), "+v"(peerws));
        call = rfl(cv) + 1;
        char* gp = peerws + WS_AMAX + (call & 1) * 8192 + (rank * NTILE + gt) * 8;
        asm volatile("" : "+v"(gp));
        if (!isB) {
            BARRIER(); WAIT_VM(0); BARRIER(); BARRIER();
            STAMP(2);
            if (lw == 0) {
                const char* dws = (const char*)ap->ws[dst];
                const char* tp = lane < 8 ? gp : lane < 16 ? dws + WS_DATA + (rank * 16 + tile) * 16384 + (lane & 3) * 4096
                                                          : dws + WS_MASK + (call & 1) * 524288 + (rank * 16 + tile) * 4096;
                unsigned dummy;
                asm volatile("global_load_dword %0, %1, off sc0 sc1\n\ts_waitcnt vmcnt(0)" : "=&v"(dummy) : "v"(tp) : "memory");
            }
        } else {
        unsigned bad = 0;
        {
            __amdgpu_buffer_rsrc_t rp = __builtin_amdgcn_make_buffer_rsrc((void*)part, 0, 65536, 0x00020000);
            const unsigned vp = lane * 16;
#define POLL_PART() asm volatile("global_load_dword %0, %1, off sc0 sc1" : "=v"(fl) : "v"(pflag + lw * 32) : "memory")
#define FETCH_HALF() do { STAMP(7); const unsigned hl_ = ldsb + nh * SLOT + lw * 8192, hs_ = lw * 16384 + nh * 8192; \
                _Pragma("unroll") for (int i = 0; i < 8; ++i) GLDS_SYS(rp, hl_ + i * 1024, vp, hs_ + i * 1024); ++nh; } while (0)
            bool seen = rfl(fl) >= call;
            unsigned nh = 0;
            if (seen) FETCH_HALF(); else POLL_PART();
            BARRIER();
            TR(nst - 4);
            if (seen) { FETCH_HALF(); WAIT_VM(16); }
            else {
                WAIT_VM(0); asm volatile("" : "+v"(fl)); seen = rfl(fl) >= call;
                if (seen) FETCH_HALF(); else POLL_PART();
            }
            BARRIER();
            TR(nst - 3);
            if (!seen) { WAIT_VM(0); asm volatile("" : "+v"(fl)); seen = rfl(fl) >= call; }
            if (seen && nh < 2) FETCH_HALF();
            BARRIER();
            TR(nst - 2);
            if (!seen) bad = wait_flag(pflag + lw * 32, tmo, call);
#pragma nounroll
            while (nh < 2) FETCH_HALF();
            WAIT_VM(0); TR(nst - 1); BARRIER();
            TR(nst);
        }
        STAMP(2);
        BARRIER();
        if (lw == 0) {
            const u32x4 m4 = *(LAS u32x4*)ctrl;
            const u32x2 gr = {max(max(m4.x, m4.y), max(m4.z, m4.w)), call};
            if (lane < NR) asm volatile("global_store_dwordx2 %0, %1, off sc0 sc1\n\ts_nop 1" :: "v"(gp), "v"(gr) : "memory");
            STAMP(5);
        }
        BARRIER();
        STAMP(1);
        FETCH();
        STAMP(6);
        const char* g = myws + WS_AMAX + (call & 1) * 8192 + lw * 2048 + lane * 16;
        unsigned mx = 0;
        const unsigned long long t0 = __builtin_amdgcn_s_memrealtime();
        unsigned polls = 0;
        for (;;) {
            u32x4 a, c;
            asm volatile("global_load_dwordx4 %0, %2, off sc0 sc1\n\t"
                         "global_load_dwordx4 %1, %2, off offset:1024 sc0 sc1\n\t"
                         "s_waitcnt vmcnt(0)"
                         : "=&v"(a), "=&v"(c) : "v"(g) : "memory");
            const bool ok = a.y == call && a.w == call && c.y == call && c.w == call;
            mx = max(max(a.x, a.z), max(c.x, c.z));
            if (__all(ok)) break;
            if (give_up(tmo, call, t0, polls)) { bad = 1; break; }
            __builtin_amdgcn_s_sleep(1);
        }
        mx = wave_max_u32(mx);
        if (lane == 0) { ctrl[8 + lw] = mx; ctrl[12 + lw] = bad; }
        STAMP(3);
        }
    } else {
        const int wm = wid >> 1, wn = wid & 1;
        const int fr = lane & 15, fq = lane >> 4, q = (lane & 15) >> 2, p = lane & 3;
        asm volatile("s_waitcnt lgkmcnt(0)" : "+s"(xp), "+s"(wp));
        if (b < 64) {
            unsigned long long pc; unsigned len, rel;
            asm volatile("s_getpc_b64 %0\n.Lgemm_a2a_here:\n\t"
                         "s_mov_b32 %1, .Lgemm_a2a_here-.Lgemm_a2a_top\n\t"
                         "s_mov_b32 %2, .Lgemm_a2a_e1-.Lgemm_a2a_top\n\t"
                         "s_max_u32 %2, %2, .Lgemm_a2a_e2-.Lgemm_a2a_top" : "=s"(pc), "=s"(rel), "=s"(len) :: "scc");
            pc -= rel;
            const unsigned off = (((b >> 3) * 4 + wid) * 8 + (lane & 7)) * 128;
            if (lane < 8 && off < len) {
                unsigned dummy;
                asm volatile("global_load_dword %0, %1, off\n\ts_waitcnt vmcnt(0)" : "=&v"(dummy) : "v"((const char*)pc + off) : "memory");
            }
        }
        ARGS_READY();
        unsigned callv = __hip_atomic_load((GAS unsigned*)(myws + WS_CNT) + b, __ATOMIC_RELAXED, __HIP_MEMORY_SCOPE_AGENT);
        char* mptr = (char*)ap->ws[dst] + WS_MASK + (((rank * 16 + tile) * 8 + wid * 2) * 64 + lane) * 8;
        unsigned ax[2][2], aw[2][4];
#pragma unroll
        for (int ks = 0; ks < 2; ++ks) {
            const int row = wm * 64 + fr;
            ax[0][ks] = ldsb + row * 128 + (((ks * 4 + fq) ^ ((row >> 1) & 7)) << 4);
            ax[1][ks] = ax[0][ks] + 2 * SLOT;
            asm volatile("" : "+v"(ax[0][ks]), "+v"(ax[1][ks]));
        }
#pragma unroll
        for (int c = 0; c < 4; ++c) {
            const int k = 8 * fq + q, f = ((k & 3) << 1) | (k & 8);
            aw[0][c] = ldsb + XBYTES + k * 256 + (((wn * 8 + 2 * c + (p >> 1)) ^ f) << 4) + (p & 1) * 8;
            aw[1][c] = aw[0][c] + 2 * SLOT;
            asm volatile("" : "+v"(aw[0][c]), "+v"(aw[1][c]));
        }
        f32x4 acc[16];
#pragma unroll
        for (int i = 0; i < 16; ++i) { acc[i] = (f32x4){0.f, 0.f, 0.f, 0.f}; asm volatile("" : "+v"(acc[i])); }
        asm volatile("" : "+v"(mptr));
        Frag f0, f1;
        BARRIER();
        rd_all<0, 0>(f0, ax, aw);
        WAIT_LGKM0(); BARRIER();
        asm volatile("s_nop 1");
        for (int t = 0; t < nst - 4; t += 4) {
            kstep<1, 0>(acc, f0, f1, ax, aw); kstep<2, 0>(acc, f1, f0, ax, aw);
            kstep<3, 0>(acc, f0, f1, ax, aw); kstep<0, 0>(acc, f1, f0, ax, aw);
        }
        call = rfl(callv) + 1;
        SB();
        kstep<1, 0>(acc, f0, f1, ax, aw); kstep<2, 0>(acc, f1, f0, ax, aw); kstep<3, 0>(acc, f0, f1, ax, aw);
        klast<0, 16, false>(acc, f1);
        if (!isB) {
            klast_give<16>(acc, f1, part + wid * 16384 + lane * 16);
            STAMP(2);
            WAIT_VM(0);
            if (lane == 0) __hip_atomic_store(pflag + wid * 32, call, __ATOMIC_RELAXED, __HIP_MEMORY_SCOPE_SYSTEM);
            STAMP(3);
        } else {
        BARRIER();
        f32x4 pa[16];
#pragma unroll
        for (int i = 0; i < 16; ++i) pa[i] = *(LAS f32x4*)(lds + (i < 8 ? 0 : SLOT) + wid * 8192 + (i & 7) * 1024 + lane * 16);
        SB();
        klast<16, 32, true>(acc, f1);
        asm volatile("s_nop 7" ::: "memory");
#pragma unroll
        for (int i = 0; i < 16; ++i) asm volatile("" : "+v"(acc[i]));
        STAMP(2);
        float mx4[4] = {0.f, 0.f, 0.f, 0.f};
#pragma unroll
        for (int i = 0; i < 16; ++i) acc[i] += pa[i];
#pragma unroll
        for (int i = 0; i < 16; i += 2)
#pragma unroll
            for (int e = 0; e < 4; ++e) mx4[e] = __builtin_fmaxf(__builtin_fmaxf(acc[i][e], acc[i + 1][e]), mx4[e]);
        const float mxf = fmaxf(fmaxf(mx4[0], mx4[1]), fmaxf(mx4[2], mx4[3]));
        const unsigned wmx = wave_max_u32(__float_as_uint(mxf) & 0x7fffffffu);
        if (lane == 0) ctrl[wid] = wmx;
        WAIT_LGKM0(); BARRIER();
        STAMP(3);
        const unsigned trash = ldsb + 8 * CSTR + wid * 512 + lane * 4;
        unsigned mq[8];
#pragma unroll
        for (int c = 0; c < 8; ++c) mq[c] = __float_as_uint(acc[2 * c + 1][3]) >> 31;
#pragma unroll
        for (int e = 6; e >= 0; --e)
#pragma unroll
            for (int c = 0; c < 8; ++c) mq[c] = __builtin_amdgcn_alignbit(mq[c], __float_as_uint(acc[2 * c + (e >> 2)][e & 3]), 31);
        unsigned wa[16];
#pragma unroll
        for (int hh = 0; hh < 2; ++hh) {
            const unsigned m = ~(mq[hh * 4] | (mq[hh * 4 + 1] << 8) | (mq[hh * 4 + 2] << 16) | (mq[hh * 4 + 3] << 24));
            const u32x2 mg = {m, call};
            asm volatile("global_store_dwordx2 %0, %1, off offset:%2 sc0 sc1\n\ts_nop 1" :: "v"(mptr + (call & 1) * 524288), "v"(mg), "n"(hh * 512) : "memory");
            unsigned pos[8];
            const unsigned tot = stream_offsets(m, pos);
            if (lane == 0) ctrl[16 + wid * 2 + hh] = tot;
#pragma unroll
            for (int g = 0; g < 8; ++g) wa[hh * 8 + g] = ldsb + (wid * 2 + hh) * CSTR + pos[g] * 4;
        }
#pragma unroll
        for (int e = 0; e < 4; ++e) {
            unsigned sg[16], ad[16];
#pragma unroll
            for (int i = 0; i < 16; ++i) asm("v_ashrrev_i32 %0, 31, %1" : "=v"(sg[i]) : "v"(acc[i][e]));
#pragma unroll
            for (int i = 0; i < 16; ++i) asm("v_bfi_b32 %0, %1, %2, %3" : "=v"(ad[i]) : "v"(sg[i]), "v"(trash), "v"(wa[i]));
#pragma unroll
            for (int i = 0; i < 16; ++i) {
                *(LAS float*)(uintptr_t)(ad[i] + 4 * e) = acc[i][e];
                if (e < 3) asm("v_lshl_add_u32 %0, %1, 2, %0" : "+v"(wa[i]) : "v"(sg[i]));
            }
        }
        asm volatile(".Lgemm_a2a_e2:");
        WAIT_LGKM0(); BARRIER();
        STAMP(1);
        FETCH();
        STAMP(6);
        }
    }
    const unsigned tag = call % 255u + 1u, tg0 = tag_bits(tag & 15u), tg1 = tag_bits(tag >> 4);
    if (isB) {
        char* dptr = (char*)ap->ws[dst] + WS_DATA + ((rank * 16 + tile) * 8 + wid) * 2048 + lane * 16;
        asm volatile("" : "+v"(dptr));
        WAIT_LGKM0(); BARRIER();
        STAMP(4);
        if (tid == 0) __hip_atomic_store((GAS unsigned*)(myws + WS_GO + gt * 128), call, __ATOMIC_RELAXED, __HIP_MEMORY_SCOPE_AGENT);
        const u32x4 g4 = *(LAS u32x4*)(ctrl + 8), b4 = *(LAS u32x4*)(ctrl + 12);
        const float amax = __uint_as_float(max(max(g4.x, g4.y), max(g4.z, g4.w)));
        const bool timed_out = (b4.x | b4.y | b4.z | b4.w) != 0;
        float inv = 448.0f * __builtin_amdgcn_rcpf(amax);
        if (__builtin_expect(!(amax > 1e-30f && amax < 1e30f), 0)) inv = amax > 0.f ? 1.0f / (amax / 448.0f) : 0.f;
#pragma unroll
        for (int h = 0; h < 2; ++h) {
            if (h * 1024u < total) {
                u32x4 o;
#pragma unroll
                for (int i = 0; i < 4; ++i) {
                    int r = __builtin_amdgcn_cvt_pk_fp8_f32(fminf(v[h][i][0] * inv, 448.f), fminf(v[h][i][1] * inv, 448.f), 0, false);
                    r = __builtin_amdgcn_cvt_pk_fp8_f32(fminf(v[h][i][2] * inv, 448.f), fminf(v[h][i][3] * inv, 448.f), r, true);
                    o[i] = ((timed_out ? 0x7f7f7f7fu : (unsigned)r) & 0x7f7f7f7fu) | ((i & 1) ? tg1 : tg0);
                }
                if (h * 1024u + lane * 16u < total)
                    asm volatile("global_store_dwordx4 %0, %1, off offset:%2 sc0 sc1\n\ts_nop 1" :: "v"(dptr), "v"(o), "n"(h * 1024) : "memory");
            }
        }
        if (wid < 4) STAMP(5);
    } else {
        bool bad = false;
        unsigned m;
        {
            const char* mp = myws + WS_MASK + (call & 1) * 524288 + ((gt * 8 + wid) * 64 + lane) * 8;
            const unsigned long long t0 = __builtin_amdgcn_s_memrealtime();
            unsigned polls = 0;
            for (;;) {
                u32x2 g;
                asm volatile("global_load_dwordx2 %0, %1, off sc0 sc1\n\ts_waitcnt vmcnt(0)" : "=&v"(g) : "v"(mp) : "memory");
                m = g.x;
                if (__all(g.y == call)) break;
                if (give_up(tmo, call, t0, polls)) { bad = true; m = 0; break; }
                __builtin_amdgcn_s_sleep(2);
            }
        }
        STAMP(4);
        unsigned pos[8];
        const unsigned tot = stream_offsets(m, pos);
        const unsigned sbase = ldsb + wid * 8192;
#pragma unroll
        for (int g = 0; g < 8; ++g) *(LAS unsigned*)(uintptr_t)(sbase + 4096 + g * 256 + lane * 4) = pos[g] * 16 + ((m >> (4 * g)) & 15u);
        unsigned ra[8], rb[8], sel[8];
#pragma unroll
        for (int j = 0; j < 8; ++j)
            sel[j] = *(LAS unsigned*)(uintptr_t)(sbase + 4096 + ((j >> 2) * 4 + (lane & 3)) * 256 + ((j & 3) * 16 + (lane >> 2)) * 4);
#pragma unroll
        for (int j = 0; j < 8; ++j) {
            const unsigned e = sel[j], d0 = e >> 6, d1 = d0 + 1;
            const unsigned sp = ((e & 15u) * 0x00204081u) & 0x01010101u, mk = (sp << 8) - sp, s2 = sp + (sp << 8);
            ra[j] = sbase + (d0 ^ ((d0 >> 4) & 3u)) * 4;
            rb[j] = sbase + (d1 ^ ((d1 >> 4) & 3u)) * 4;
            sel[j] = ((((s2 + (sp << 16)) << 8) + __builtin_amdgcn_perm(0u, (e >> 4) & 3u, 0u)) & mk) | (0x0c0c0c0cu & ~mk);
        }
#pragma unroll
        for (int j = 0; j < 8; ++j) asm volatile("" : "+v"(ra[j]), "+v"(rb[j]), "+v"(sel[j]));
        char* optr = (char*)ap->myout + (size_t)(dst * MLOC + m0 + (wid >> 2) * 64 + (wid & 1) * 32 + (lane >> 2)) * NSH
                     + nc0 + ((wid >> 1) & 1) * 64 + (lane & 3) * 16;
        char* sp = myws + WS_DATA + (gt * 8 + wid) * 2048 + lane * 16;
        asm volatile("" : "+v"(optr), "+v"(sp));
        if (!bad) bad = wait_flag((GAS unsigned*)(myws + WS_GO + gt * 128), tmo, call);
        STAMP(5);
        const bool n0 = lane * 16u < tot, n1 = 1024u + lane * 16u < tot;
        u32x4 d0 = {0, 0, 0, 0}, d1 = {0, 0, 0, 0};
        if (!bad) {
            const unsigned long long t0 = __builtin_amdgcn_s_memrealtime();
            unsigned polls = 0;
            unsigned long long p0 = __ballot(n0), p1 = __ballot(n1);
            do {
                unsigned tw;
                asm volatile("s_mov_b64 exec, %4\n\tglobal_load_dwordx4 %0, %3, off sc0 sc1\n\t"
                             "s_mov_b64 exec, %5\n\tglobal_load_dwordx4 %1, %3, off offset:1024 sc0 sc1\n\t"
                             "s_mov_b64 exec, -1\n\tglobal_load_dword %2, %6, off sc1\n\ts_waitcnt vmcnt(0)"
                             : "+v"(d0), "+v"(d1), "=&v"(tw) : "v"(sp), "s"(p0), "s"(p1), "v"(tmo) : "memory");
                p0 &= __ballot((((d0.x & 0x80808080u) ^ tg0) | ((d0.y & 0x80808080u) ^ tg1) | ((d0.z & 0x80808080u) ^ tg0) | ((d0.w & 0x80808080u) ^ tg1)) != 0);
                p1 &= __ballot((((d1.x & 0x80808080u) ^ tg0) | ((d1.y & 0x80808080u) ^ tg1) | ((d1.z & 0x80808080u) ^ tg0) | ((d1.w & 0x80808080u) ^ tg1)) != 0);
                if (rfl(tw) == call || ((p0 | p1) && give_up(tmo, call, t0, polls))) { bad = true; break; }
            } while (p0 | p1);
        }
        STAMP(7);
        *(LAS u32x4*)(uintptr_t)(sbase + lane * 16) = d0 & 0x7f7f7f7fu;
        *(LAS u32x4*)(uintptr_t)(sbase + 1024 + lane * 16) = d1 & 0x7f7f7f7fu;
        const unsigned poison = bad ? 0x7f7f7f7fu : 0u;
        unsigned lo[8], hi[8];
#pragma unroll
        for (int g = 0; g < 8; ++g) { lo[g] = *(LAS unsigned*)(uintptr_t)ra[g]; hi[g] = *(LAS unsigned*)(uintptr_t)rb[g]; }
#pragma unroll
        for (int h = 0; h < 2; ++h) {
            u32x4 o;
#pragma unroll
            for (int i = 0; i < 4; ++i) o[i] = __builtin_amdgcn_perm(hi[h * 4 + i], lo[h * 4 + i], sel[h * 4 + i]) | poison;
            asm volatile("global_store_dwordx4 %0, %1, off sc0 sc1\n\ts_nop 1" :: "v"(optr + h * 16 * NSH), "v"(o) : "memory");
        }
        const u32x4 z = {0, 0, 0, 0};
        if (n0) asm volatile("global_store_dwordx4 %0, %1, off sc0 sc1\n\ts_nop 1" :: "v"(sp), "v"(z) : "memory");
        if (n1) asm volatile("global_store_dwordx4 %0, %1, off offset:1024 sc0 sc1\n\ts_nop 1" :: "v"(sp), "v"(z) : "memory");
        WAIT_VM(0);
        STAMP(6);
    }
    asm volatile(".Lgemm_a2a_e1:");
    if (tid == 256) __hip_atomic_store((GAS unsigned*)(myws + WS_CNT) + b, call, __ATOMIC_RELAXED, __HIP_MEMORY_SCOPE_AGENT);
    STAMP_OUT();
}

}

extern "C" void kernel_launch(int rank, int nranks, ncclComm_t comm,
                              hipStream_t stream,
                              void* const* d_in, const int* in_sizes, int n_in,
                              void* d_out, int out_size,
                              const int* in_dtypes, int out_dtype,
                              void* const* d_out_all,
                              void* const* d_ws_all, size_t ws_bytes) {
    if (nranks != NR || n_in < 2 || in_sizes[0] != MLOC * KD || in_sizes[1] != KD * ND ||
        out_size != ND * NSH || ws_bytes < WS_END || (unsigned)rank >= (unsigned)NR) {
        fprintf(stderr, "kernel_launch: unexpected problem shape\n");
        return;
    }
    HIP_CHECK(hipFuncSetAttribute((const void*)gemm_a2a_kernel, hipFuncAttributeMaxDynamicSharedMemorySize, LDS_BYTES));
    Args a;
    a.x = d_in[0]; a.w = d_in[1]; a.myws = d_ws_all[rank]; a.myout = d_out; a.rank = rank; a.pad = 0;
    for (int p = 0; p < NR; ++p) a.ws[p] = d_ws_all[p];
    hipLaunchKernelGGL(gemm_a2a_kernel, dim3(NWG), dim3(512), LDS_BYTES, stream, a);
    HIP_CHECK(hipGetLastError());
}
```

```cpp
#include <hip/hip_runtime.h>
#include <rccl/rccl.h>
#include <cstdio>
#include <cstdint>
#include <cstddef>

#define HIP_CHECK(expr) do { hipError_t _e = (expr); if (_e != hipSuccess) { \
    fprintf(stderr, "HIP error %s at %s:%d\n", hipGetErrorString(_e), __FILE__, __LINE__); \
    return; } } while (0)

namespace {

typedef short          s16x4 __attribute__((ext_vector_type(4)));
typedef short          s16x8 __attribute__((ext_vector_type(8)));
typedef float          f32x4 __attribute__((ext_vector_type(4)));
typedef unsigned       u32x4 __attribute__((ext_vector_type(4)));
typedef unsigned       u32x2 __attribute__((ext_vector_type(2)));
#define LAS __attribute__((address_space(3)))
#define GAS __attribute__((address_space(1)))

constexpr int NR = 8;
constexpr int MLOC = 512, KD = 4096, ND = 4096, NSH = 512;
constexpr int BM = 128, BN = 128, BK = 64;
constexpr int NKT = KD / BK;
constexpr int KA = 28;
constexpr int LASTA = 12, LASTB = 16;
constexpr int XBYTES = BM * BK * 2;
constexpr int SLOT = XBYTES + BK * BN * 2;
constexpr int NS = 4;
constexpr int CTRL = NS * SLOT;
constexpr int LDS_BYTES = CTRL + 256;
constexpr int NWG = 256, NTILE = 128;
constexpr int CSTR = 8192;
static_assert(KA % 4 == 0 && KA >= 8 && NKT - KA >= 8, "");

constexpr size_t WS_CNT   = 0;
constexpr size_t WS_TMO   = 4096;
constexpr size_t WS_AMAX  = 8192;
constexpr size_t WS_GO    = WS_AMAX + 2 * 8192;
constexpr size_t WS_PFLAG = WS_GO + NTILE * 128;
constexpr size_t WS_MASK  = 1 << 20;
constexpr size_t WS_DATA  = 2 << 20;
constexpr size_t WS_PART  = 4 << 20;
constexpr size_t WS_END   = WS_PART + (size_t)NTILE * 65536;
static_assert(WS_PFLAG + NTILE * 512 <= WS_MASK, "");
constexpr unsigned long long DEADLINE = 100000000ull;

#define BARRIER() do { asm volatile("" ::: "memory"); __builtin_amdgcn_s_barrier(); asm volatile("" ::: "memory"); } while (0)
#define WAIT_VM(n) asm volatile("s_waitcnt vmcnt(%0)" :: "n"(n) : "memory")
#define WAIT_LGKM0() __builtin_amdgcn_s_waitcnt(0xC07F)
#define SB() __builtin_amdgcn_sched_barrier(0)

__device__ __forceinline__ unsigned rfl(unsigned v) { return __builtin_amdgcn_readfirstlane(v); }

__device__ __forceinline__ unsigned wave_max_u32(unsigned v) {
    v = max(v, (unsigned)__builtin_amdgcn_update_dpp(0, (int)v, 0xB1, 0xF, 0xF, true));
    v = max(v, (unsigned)__builtin_amdgcn_update_dpp(0, (int)v, 0x4E, 0xF, 0xF, true));
    v = max(v, (unsigned)__builtin_amdgcn_update_dpp(0, (int)v, 0x141, 0xF, 0xF, true));
    v = max(v, (unsigned)__builtin_amdgcn_update_dpp(0, (int)v, 0x140, 0xF, 0xF, true));
    unsigned a = __builtin_amdgcn_readlane(v, 0), b = __builtin_amdgcn_readlane(v, 16);
    unsigned c = __builtin_amdgcn_readlane(v, 32), d = __builtin_amdgcn_readlane(v, 48);
    return max(max(a, b), max(c, d));
}

__device__ __forceinline__ unsigned wave_scan_u32(unsigned v) {
    v += (unsigned)__builtin_amdgcn_update_dpp(0, (int)v, 0x111, 0xF, 0xF, true);
    v += (unsigned)__builtin_amdgcn_update_dpp(0, (int)v, 0x112, 0xF, 0xF, true);
    v += (unsigned)__builtin_amdgcn_update_dpp(0, (int)v, 0x114, 0xF, 0xF, true);
    v += (unsigned)__builtin_amdgcn_update_dpp(0, (int)v, 0x118, 0xF, 0xF, true);
    v += (unsigned)__builtin_amdgcn_update_dpp(0, (int)v, 0x142, 0xA, 0xF, false);
    v += (unsigned)__builtin_amdgcn_update_dpp(0, (int)v, 0x143, 0xC, 0xF, false);
    return v;
}
__device__ __forceinline__ unsigned stream_offsets(unsigned m, unsigned (&pos)[8]) {
    unsigned c = m - ((m >> 1) & 0x55555555u);
    c = (c & 0x33333333u) + ((c >> 2) & 0x33333333u);
    unsigned ex[4], tot[4];
#pragma unroll
    for (int j = 0; j < 4; ++j) {
        const unsigned f = (c >> (4 * j)) & 0x000f000fu, inc = wave_scan_u32(f);
        tot[j] = __builtin_amdgcn_readlane(inc, 63); ex[j] = inc - f;
    }
    unsigned base = 0;
#pragma unroll
    for (int g = 0; g < 8; ++g) {
        pos[g] = base + (g < 4 ? ex[g] & 0xffffu : ex[g - 4] >> 16);
        base += g < 4 ? tot[g] & 0xffffu : tot[g - 4] >> 16;
    }
    return base;
}
__device__ __forceinline__ unsigned tag_bits(unsigned n) { return ((n * 0x00204081u) & 0x01010101u) << 7; }

__device__ __forceinline__ bool give_up(GAS unsigned* tmo, unsigned call, unsigned long long t0, unsigned& polls) {
    if (__builtin_expect((++polls & 31u) != 0u, 1)) return false;
    if (rfl(__hip_atomic_load(tmo, __ATOMIC_RELAXED, __HIP_MEMORY_SCOPE_AGENT)) == call ||
        __builtin_amdgcn_s_memrealtime() - t0 > DEADLINE) {
        __hip_atomic_store(tmo, call, __ATOMIC_RELAXED, __HIP_MEMORY_SCOPE_AGENT);
        return true;
    }
    return false;
}

__device__ __forceinline__ bool wait_flag(GAS unsigned* flag, GAS unsigned* tmo, unsigned call) {
    const unsigned long long t0 = __builtin_amdgcn_s_memrealtime();
    unsigned polls = 0;
    while (rfl(__hip_atomic_load(flag, __ATOMIC_RELAXED, __HIP_MEMORY_SCOPE_SYSTEM)) < call) {
        if (give_up(tmo, call, t0, polls)) return true;
        __builtin_amdgcn_s_sleep(1);
    }
    return false;
}

#define GLDS(rsrc, ldsaddr, voff, soff) \
    asm volatile("s_mov_b32 m0, %0\n\ts_nop 0\n\tbuffer_load_dwordx4 %1, %2, %3 offen lds" \
                 :: "s"(ldsaddr), "v"(voff), "s"(rsrc), "s"(soff) : "memory")
#define GLDS_SYS(rsrc, ldsaddr, voff, soff) \
    asm volatile("s_mov_b32 m0, %0\n\ts_nop 0\n\tbuffer_load_dwordx4 %1, %2, %3 offen sc0 sc1 lds" \
                 :: "s"(ldsaddr), "v"(voff), "s"(rsrc), "s"(soff) : "memory")

struct Frag { s16x8 x[8]; s16x4 w[16]; };

template <int SL, int R>
__device__ __forceinline__ void rd(Frag& f, const unsigned (&ax)[2][2], const unsigned (&aw)[2][4]) {
    constexpr int g = R / 3, e = R % 3, so = (SL & 1) * SLOT, ks = g >> 2, i = g & 3;
    if constexpr (e == 0) f.x[g] = *(LAS s16x8*)(uintptr_t)(ax[SL >> 1][ks] + so + i * 2048);
    else f.w[g * 2 + e - 1] = __builtin_amdgcn_ds_read_tr16_b64_v4i16(
             (LAS s16x4*)(uintptr_t)(aw[SL >> 1][i] + so + ks * 8192 + (e - 1) * 1024));
}
template <int SL, int R>
__device__ __forceinline__ void rd_all(Frag& f, const unsigned (&ax)[2][2], const unsigned (&aw)[2][4]) {
    if constexpr (R < 24) { rd<SL, R>(f, ax, aw); rd_all<SL, R + 1>(f, ax, aw); }
}
template <int G, bool PAD = false>
__device__ __forceinline__ void mfma(f32x4 (&acc)[16], const Frag& cur) {
    constexpr int ks = G >> 4, c = (G >> 2) & 3, mi = G & 3;
    const s16x8 wf = __builtin_shufflevector(cur.w[(ks * 4 + c) * 2], cur.w[(ks * 4 + c) * 2 + 1], 0, 1, 2, 3, 4, 5, 6, 7);
    if constexpr (PAD)
        asm volatile("s_nop 1\n\tv_mfma_f32_16x16x32_bf16 %0, %1, %2, %0" : "+v"(acc[mi * 4 + c]) : "v"(wf), "v"(cur.x[ks * 4 + mi]));
    else
        asm volatile("v_mfma_f32_16x16x32_bf16 %0, %1, %2, %0" : "+v"(acc[mi * 4 + c]) : "v"(wf), "v"(cur.x[ks * 4 + mi]));
}
template <int G>
__device__ __forceinline__ void give(f32x4 (&acc)[16], char* pp) {
    constexpr int i = (G & 3) * 4 + ((G >> 2) & 3);
    asm volatile("global_store_dwordx4 %0, %1, off offset:%2 sc0 sc1\n\ts_nop 1"
                 :: "v"(pp + (i >> 2) * 4096), "v"(acc[i]), "n"((i & 3) * 1024) : "memory");
}
template <int SL, int G>
__device__ __forceinline__ void kstep(f32x4 (&acc)[16], const Frag& cur, Frag& nxt,
                                      const unsigned (&ax)[2][2], const unsigned (&aw)[2][4]) {
    if constexpr (G < 32) {
        mfma<G>(acc, cur);
        if constexpr (G < 24) rd<SL, G>(nxt, ax, aw);
        if constexpr (G == 31) { SB(); WAIT_LGKM0(); BARRIER(); }
        SB();
        kstep<SL, G + 1>(acc, cur, nxt, ax, aw);
    }
}
template <int G, int END, bool PAD>
__device__ __forceinline__ void klast(f32x4 (&acc)[16], const Frag& cur) {
    if constexpr (G < END) { mfma<G, PAD>(acc, cur); SB(); klast<G + 1, END, PAD>(acc, cur); }
}
template <int G>
__device__ __forceinline__ void klast_give(f32x4 (&acc)[16], const Frag& cur, char* pp) {
    if constexpr (G < 32) mfma<G, true>(acc, cur);
    if constexpr (G == 32) asm volatile("s_nop 7" ::: "memory");
    if constexpr (G >= 20) give<G - 4>(acc, pp);
    SB();
    if constexpr (G < 35) klast_give<G + 1>(acc, cur, pp);
}

struct Args {
    const void* x; const void* w;
    void* myws; void* myout;
    void* ws[NR];
    int rank, pad;
};

__global__ void __launch_bounds__(512, 2)
gemm_a2a_kernel(const Args args)
{
    typedef __attribute__((address_space(4))) const Args* ArgP;
    ArgP ap = (ArgP)__builtin_amdgcn_kernarg_segment_ptr();
    unsigned long long xp, wp;
    static_assert(offsetof(Args, x) == 0 && offsetof(Args, w) == 8, "");
    asm volatile(".Lgemm_a2a_top:\n\ts_load_dwordx2 %0, %2, 0x0\n\ts_load_dwordx2 %1, %2, 0x8" : "=&s"(xp), "=&s"(wp) : "s"(ap));
    SB();
    int rank;
#define ARGS_READY() \
    asm volatile("" : "+s"(ap)); \
    rank = ap->rank; myws = (char*)ap->myws; tmo = (GAS unsigned*)(myws + WS_TMO); \
    pflag = (GAS unsigned*)(myws + WS_PFLAG + gt * 512); part = myws + WS_PART + (size_t)gt * 65536
    extern __shared__ __attribute__((aligned(16))) char smem[];
    LAS char* lds = (LAS char*)smem;
    const unsigned ldsb = (unsigned)(uintptr_t)lds;
    const int tid = threadIdx.x, lane = tid & 63;
    const int wid = rfl(tid >> 6);
    const int b = blockIdx.x;
    const int dst = b & 7;
    const int isB = (b >> 3) & 1, tile = b >> 4, tm = tile & 3, tn = tile >> 2;
    const int gt = dst * 16 + tile;
    const int m0 = tm * BM;
    const int nc0 = tn * BN;
    const int nst = isB ? NKT - KA : KA, kbase = isB ? KA : 0;
    char *myws, *part; GAS unsigned *tmo, *pflag;
    LAS unsigned* ctrl = (LAS unsigned*)(lds + CTRL);

    f32x4 v[2][4];
    unsigned call, total = 0;
#define FETCH() do { \
        total = rfl(ctrl[16 + wid]); \
        const unsigned ca_ = ldsb + wid * CSTR + lane * 64, sw_ = (lane << 2) & 48; \
        _Pragma("unroll") for (int i = 0; i < 4; ++i) v[0][i] = *(LAS f32x4*)(uintptr_t)(ca_ + ((i * 16) ^ sw_)); \
        if (total > 1024u) { \
            _Pragma("unroll") for (int i = 0; i < 4; ++i) v[1][i] = *(LAS f32x4*)(uintptr_t)(ca_ + 4096 + ((i * 16) ^ sw_)); } \
    } while (0)

    if (__builtin_expect(wid >= 4, 1)) {
        const int lw = wid - 4;
        const int ph = (dst + 2 * isB) & 3, rot = (tm + tn) & 3;
        const unsigned k0 = kbase + ((0x244400E4u >> (ph * 8 + rot * 2)) & 3);
        const unsigned lx = ldsb + lw * 4096, lwb = ldsb + XBYTES + lw * 4096;
        unsigned vx[4], vw[4];
#pragma unroll
        for (int i = 0; i < 4; ++i) {
            const int k = 16 * lw + 4 * i + (lane >> 4), f = ((k & 3) << 1) | (k & 8);
            vw[i] = (unsigned)(k * (ND * 2) + (dst * NSH + nc0) * 2 + (((lane & 15) ^ f) << 4));
            asm volatile("" : "+v"(vw[i]));
        }
        asm volatile("s_waitcnt lgkmcnt(0)" : "+s"(xp), "+s"(wp));
        __amdgpu_buffer_rsrc_t rx = __builtin_amdgcn_make_buffer_rsrc((void*)xp, 0, MLOC * KD * 2, 0x00020000);
        __amdgpu_buffer_rsrc_t rw = __builtin_amdgcn_make_buffer_rsrc((void*)wp, 0, KD * ND * 2, 0x00020000);
#pragma unroll
        for (int i = 0; i < 4; ++i) GLDS(rw, lwb + i * 1024, vw[i], k0 * (BK * ND * 2));
        SB();
#pragma unroll
        for (int i = 0; i < 4; ++i) {
            const int row = 32 * lw + 8 * i + (lane >> 3);
            vx[i] = (unsigned)((m0 + row) * (KD * 2) + (((lane & 7) ^ ((row >> 1) & 7)) << 4));
            GLDS(rx, lx + i * 1024, vx[i], k0 * (BK * 2));
        }
        SB();
        int ktab;
        {
            const int ls = ph + (isB ? NKT - KA - LASTB : KA - LASTA);
            const int bs = lane < ph ? 0 : lane >= ls ? ls : ph + ((lane - ph) & ~3);
            const int sz = lane < ph ? ph : lane >= ls ? nst - ls : 4;
            ktab = kbase + bs + (lane - bs + rot * ((sz + 3) >> 2)) % sz;
        }
#define ISSUE(t) do { \
            const unsigned kt_ = __builtin_amdgcn_readlane(ktab, t), so_ = ((t) & 3) * SLOT; \
            _Pragma("unroll") for (int i = 0; i < 4; ++i) GLDS(rw, lwb + so_ + i * 1024, vw[i], kt_ * (BK * ND * 2)); \
            _Pragma("unroll") for (int i = 0; i < 4; ++i) GLDS(rx, lx + so_ + i * 1024, vx[i], kt_ * (BK * 2)); } while (0)
        ISSUE(1);
        ARGS_READY();
        WAIT_VM(8); BARRIER();
        ISSUE(2);
        WAIT_VM(8); BARRIER();
        unsigned cv, fl = 0;
        asm volatile("global_load_dword %0, %1, off sc1" : "=v"(cv) : "v"(myws + WS_CNT + b * 4) : "memory");
        char* peerws;
        asm volatile("global_load_dwordx2 %0, %1, off" : "=v"(peerws)
                     : "v"((const char*)ap + offsetof(Args, ws) + (lane & 7) * 8) : "memory");
        ISSUE(3);
        for (int t = 0; t < nst - NS; ++t) {
            ISSUE(t + NS);
            if (t == nst - NS - 2) asm volatile("global_load_dword %0, %1, off sc0 sc1" : "=v"(fl) : "v"(pflag + lw * 32) : "memory");
            WAIT_VM(16); BARRIER();
        }
        WAIT_VM(8);
        asm volatile("" : "+v"(fl), "+v"(cv), "+v"(peerws));
        call = rfl(cv) + 1;
        char* gp = peerws + WS_AMAX + (call & 1) * 8192 + (rank * NTILE + gt) * 8;
        asm volatile("" : "+v"(gp));
        if (!isB) {
            BARRIER(); WAIT_VM(0); BARRIER(); BARRIER();
            if (lw == 0) {
                const char* dws = (const char*)ap->ws[dst];
                const char* tp = lane < 8 ? gp : lane < 16 ? dws + WS_DATA + (rank * 16 + tile) * 16384 + (lane & 3) * 4096
                                                          : dws + WS_MASK + (call & 1) * 524288 + (rank * 16 + tile) * 4096;
                unsigned dummy;
                asm volatile("global_load_dword %0, %1, off sc0 sc1\n\ts_waitcnt vmcnt(0)" : "=&v"(dummy) : "v"(tp) : "memory");
            }
        } else {
        unsigned bad = 0;
        {
            __amdgpu_buffer_rsrc_t rp = __builtin_amdgcn_make_buffer_rsrc((void*)part, 0, 65536, 0x00020000);
            const unsigned vp = lane * 16;
#define POLL_PART() asm volatile("global_load_dword %0, %1, off sc0 sc1" : "=v"(fl) : "v"(pflag + lw * 32) : "memory")
#define FETCH_HALF() do { const unsigned hl_ = ldsb + nh * SLOT + lw * 8192, hs_ = lw * 16384 + nh * 8192; \
                _Pragma("unroll") for (int i = 0; i < 8; ++i) GLDS_SYS(rp, hl_ + i * 1024, vp, hs_ + i * 1024); ++nh; } while (0)
            bool seen = rfl(fl) >= call;
            unsigned nh = 0;
            if (seen) FETCH_HALF(); else POLL_PART();
            BARRIER();
            if (seen) { FETCH_HALF(); WAIT_VM(16); }
            else {
                WAIT_VM(0); asm volatile("" : "+v"(fl)); seen = rfl(fl) >= call;
                if (seen) FETCH_HALF(); else POLL_PART();
            }
            BARRIER();
            if (!seen) { WAIT_VM(0); asm volatile("" : "+v"(fl)); seen = rfl(fl) >= call; }
            if (seen && nh < 2) FETCH_HALF();
            BARRIER();
            if (!seen) bad = wait_flag(pflag + lw * 32, tmo, call);
#pragma nounroll
            while (nh < 2) FETCH_HALF();
            WAIT_VM(0); BARRIER();
        }
        BARRIER();
        if (lw == 0) {
            const u32x4 m4 = *(LAS u32x4*)ctrl;
            const u32x2 gr = {max(max(m4.x, m4.y), max(m4.z, m4.w)), call};
            if (lane < NR) asm volatile("global_store_dwordx2 %0, %1, off sc0 sc1\n\ts_nop 1" :: "v"(gp), "v"(gr) : "memory");
        }
        BARRIER();
        FETCH();
        const char* g = myws + WS_AMAX + (call & 1) * 8192 + lw * 2048 + lane * 16;
        unsigned mx = 0;
        const unsigned long long t0 = __builtin_amdgcn_s_memrealtime();
        unsigned polls = 0;
        for (;;) {
            u32x4 a, c;
            asm volatile("global_load_dwordx4 %0, %2, off sc0 sc1\n\t"
                         "global_load_dwordx4 %1, %2, off offset:1024 sc0 sc1\n\t"
                         "s_waitcnt vmcnt(0)"
                         : "=&v"(a), "=&v"(c) : "v"(g) : "memory");
            const bool ok = a.y == call && a.w == call && c.y == call && c.w == call;
            mx = max(max(a.x, a.z), max(c.x, c.z));
            if (__all(ok)) break;
            if (give_up(tmo, call, t0, polls)) { bad = 1; break; }
            __builtin_amdgcn_s_sleep(1);
        }
        mx = wave_max_u32(mx);
        if (lane == 0) { ctrl[8 + lw] = mx; ctrl[12 + lw] = bad; }
        }
    } else {
        const int wm = wid >> 1, wn = wid & 1;
        const int fr = lane & 15, fq = lane >> 4, q = (lane & 15) >> 2, p = lane & 3;
        asm volatile("s_waitcnt lgkmcnt(0)" : "+s"(xp), "+s"(wp));
        if (b < 64) {
            unsigned long long pc; unsigned len, rel;
            asm volatile("s_getpc_b64 %0\n.Lgemm_a2a_here:\n\t"
                         "s_mov_b32 %1, .Lgemm_a2a_here-.Lgemm_a2a_top\n\t"
                         "s_mov_b32 %2, .Lgemm_a2a_e1-.Lgemm_a2a_top\n\t"
                         "s_max_u32 %2, %2, .Lgemm_a2a_e2-.Lgemm_a2a_top" : "=s"(pc), "=s"(rel), "=s"(len) :: "scc");
            pc -= rel;
            const unsigned off = (((b >> 3) * 4 + wid) * 8 + (lane & 7)) * 128;
            if (lane < 8 && off < len) {
                unsigned dummy;
                asm volatile("global_load_dword %0, %1, off\n\ts_waitcnt vmcnt(0)" : "=&v"(dummy) : "v"((const char*)pc + off) : "memory");
            }
        }
        ARGS_READY();
        unsigned callv = __hip_atomic_load((GAS unsigned*)(myws + WS_CNT) + b, __ATOMIC_RELAXED, __HIP_MEMORY_SCOPE_AGENT);
        char* mptr = (char*)ap->ws[dst] + WS_MASK + (((rank * 16 + tile) * 8 + wid * 2) * 64 + lane) * 8;
        unsigned ax[2][2], aw[2][4];
#pragma unroll
        for (int ks = 0; ks < 2; ++ks) {
            const int row = wm * 64 + fr;
            ax[0][ks] = ldsb + row * 128 + (((ks * 4 + fq) ^ ((row >> 1) & 7)) << 4);
            ax[1][ks] = ax[0][ks] + 2 * SLOT;
            asm volatile("" : "+v"(ax[0][ks]), "+v"(ax[1][ks]));
        }
#pragma unroll
        for (int c = 0; c < 4; ++c) {
            const int k = 8 * fq + q, f = ((k & 3) << 1) | (k & 8);
            aw[0][c] = ldsb + XBYTES + k * 256 + (((wn * 8 + 2 * c + (p >> 1)) ^ f) << 4) + (p & 1) * 8;
            aw[1][c] = aw[0][c] + 2 * SLOT;
            asm volatile("" : "+v"(aw[0][c]), "+v"(aw[1][c]));
        }
        f32x4 acc[16];
#pragma unroll
        for (int i = 0; i < 16; ++i) { acc[i] = (f32x4){0.f, 0.f, 0.f, 0.f}; asm volatile("" : "+v"(acc[i])); }
        asm volatile("" : "+v"(mptr));
        Frag f0, f1;
        BARRIER();
        rd_all<0, 0>(f0, ax, aw);
        WAIT_LGKM0(); BARRIER();
        asm volatile("s_nop 1");
        for (int t = 0; t < nst - 4; t += 4) {
            kstep<1, 0>(acc, f0, f1, ax, aw); kstep<2, 0>(acc, f1, f0, ax, aw);
            kstep<3, 0>(acc, f0, f1, ax, aw); kstep<0, 0>(acc, f1, f0, ax, aw);
        }
        call = rfl(callv) + 1;
        SB();
        kstep<1, 0>(acc, f0, f1, ax, aw); kstep<2, 0>(acc, f1, f0, ax, aw); kstep<3, 0>(acc, f0, f1, ax, aw);
        klast<0, 16, false>(acc, f1);
        if (!isB) {
            klast_give<16>(acc, f1, part + wid * 16384 + lane * 16);
            WAIT_VM(0);
            if (lane == 0) __hip_atomic_store(pflag + wid * 32, call, __ATOMIC_RELAXED, __HIP_MEMORY_SCOPE_SYSTEM);
        } else {
        BARRIER();
        f32x4 pa[16];
#pragma unroll
        for (int i = 0; i < 16; ++i) pa[i] = *(LAS f32x4*)(lds + (i < 8 ? 0 : SLOT) + wid * 8192 + (i & 7) * 1024 + lane * 16);
        SB();
        klast<16, 32, true>(acc, f1);
        asm volatile("s_nop 7" ::: "memory");
#pragma unroll
        for (int i = 0; i < 16; ++i) asm volatile("" : "+v"(acc[i]));
        float mx4[4] = {0.f, 0.f, 0.f, 0.f};
#pragma unroll
        for (int i = 0; i < 16; ++i) acc[i] += pa[i];
#pragma unroll
        for (int i = 0; i < 16; i += 2)
#pragma unroll
            for (int e = 0; e < 4; ++e) mx4[e] = __builtin_fmaxf(__builtin_fmaxf(acc[i][e], acc[i + 1][e]), mx4[e]);
        const float mxf = fmaxf(fmaxf(mx4[0], mx4[1]), fmaxf(mx4[2], mx4[3]));
        const unsigned wmx = wave_max_u32(__float_as_uint(mxf) & 0x7fffffffu);
        if (lane == 0) ctrl[wid] = wmx;
        WAIT_LGKM0(); BARRIER();
        const unsigned trash = ldsb + 8 * CSTR + wid * 512 + lane * 4;
        unsigned mq[8];
#pragma unroll
        for (int c = 0; c < 8; ++c) mq[c] = __float_as_uint(acc[2 * c + 1][3]) >> 31;
#pragma unroll
        for (int e = 6; e >= 0; --e)
#pragma unroll
            for (int c = 0; c < 8; ++c) mq[c] = __builtin_amdgcn_alignbit(mq[c], __float_as_uint(acc[2 * c + (e >> 2)][e & 3]), 31);
        unsigned wa[16];
#pragma unroll
        for (int hh = 0; hh < 2; ++hh) {
            const unsigned m = ~(mq[hh * 4] | (mq[hh * 4 + 1] << 8) | (mq[hh * 4 + 2] << 16) | (mq[hh * 4 + 3] << 24));
            const u32x2 mg = {m, call};
            asm volatile("global_store_dwordx2 %0, %1, off offset:%2 sc0 sc1\n\ts_nop 1" :: "v"(mptr + (call & 1) * 524288), "v"(mg), "n"(hh * 512) : "memory");
            unsigned pos[8];
            const unsigned tot = stream_offsets(m, pos);
            if (lane == 0) ctrl[16 + wid * 2 + hh] = tot;
#pragma unroll
            for (int g = 0; g < 8; ++g) wa[hh * 8 + g] = ldsb + (wid * 2 + hh) * CSTR + pos[g] * 4;
        }
#pragma unroll
        for (int e = 0; e < 4; ++e) {
            unsigned sg[16], ad[16];
#pragma unroll
            for (int i = 0; i < 16; ++i) asm("v_ashrrev_i32 %0, 31, %1" : "=v"(sg[i]) : "v"(acc[i][e]));
#pragma unroll
            for (int i = 0; i < 16; ++i) asm("v_bfi_b32 %0, %1, %2, %3" : "=v"(ad[i]) : "v"(sg[i]), "v"(trash), "v"(wa[i]));
#pragma unroll
            for (int i = 0; i < 16; ++i) {
                *(LAS float*)(uintptr_t)(ad[i] + 4 * e) = acc[i][e];
                if (e < 3) asm("v_lshl_add_u32 %0, %1, 2, %0" : "+v"(wa[i]) : "v"(sg[i]));
            }
        }
        asm volatile(".Lgemm_a2a_e2:");
        WAIT_LGKM0(); BARRIER();
        FETCH();
        }
    }
    const unsigned tag = call % 255u + 1u, tg0 = tag_bits(tag & 15u), tg1 = tag_bits(tag >> 4);
    if (isB) {
        char* dptr = (char*)ap->ws[dst] + WS_DATA + ((rank * 16 + tile) * 8 + wid) * 2048 + lane * 16;
        asm volatile("" : "+v"(dptr));
        WAIT_LGKM0(); BARRIER();
        if (tid == 0) __hip_atomic_store((GAS unsigned*)(myws + WS_GO + gt * 128), call, __ATOMIC_RELAXED, __HIP_MEMORY_SCOPE_AGENT);
        const u32x4 g4 = *(LAS u32x4*)(ctrl + 8), b4 = *(LAS u32x4*)(ctrl + 12);
        const float amax = __uint_as_float(max(max(g4.x, g4.y), max(g4.z, g4.w)));
        const bool timed_out = (b4.x | b4.y | b4.z | b4.w) != 0;
        float inv = 448.0f * __builtin_amdgcn_rcpf(amax);
        if (__builtin_expect(!(amax > 1e-30f && amax < 1e30f), 0)) inv = amax > 0.f ? 1.0f / (amax / 448.0f) : 0.f;
#pragma unroll
        for (int h = 0; h < 2; ++h) {
            if (h * 1024u < total) {
                u32x4 o;
#pragma unroll
                for (int i = 0; i < 4; ++i) {
                    int r = __builtin_amdgcn_cvt_pk_fp8_f32(fminf(v[h][i][0] * inv, 448.f), fminf(v[h][i][1] * inv, 448.f), 0, false);
                    r = __builtin_amdgcn_cvt_pk_fp8_f32(fminf(v[h][i][2] * inv, 448.f), fminf(v[h][i][3] * inv, 448.f), r, true);
                    o[i] = ((timed_out ? 0x7f7f7f7fu : (unsigned)r) & 0x7f7f7f7fu) | ((i & 1) ? tg1 : tg0);
                }
                if (h * 1024u + lane * 16u < total)
                    asm volatile("global_store_dwordx4 %0, %1, off offset:%2 sc0 sc1\n\ts_nop 1" :: "v"(dptr), "v"(o), "n"(h * 1024) : "memory");
            }
        }
    } else {
        bool bad = false;
        unsigned m;
        {
            const char* mp = myws + WS_MASK + (call & 1) * 524288 + ((gt * 8 + wid) * 64 + lane) * 8;
            const unsigned long long t0 = __builtin_amdgcn_s_memrealtime();
            unsigned polls = 0;
            for (;;) {
                u32x2 g;
                asm volatile("global_load_dwordx2 %0, %1, off sc0 sc1\n\ts_waitcnt vmcnt(0)" : "=&v"(g) : "v"(mp) : "memory");
                m = g.x;
                if (__all(g.y == call)) break;
                if (give_up(tmo, call, t0, polls)) { bad = true; m = 0; break; }
                __builtin_amdgcn_s_sleep(2);
            }
        }
        unsigned pos[8];
        const unsigned tot = stream_offsets(m, pos);
        const unsigned sbase = ldsb + wid * 8192;
#pragma unroll
        for (int g = 0; g < 8; ++g) *(LAS unsigned*)(uintptr_t)(sbase + 4096 + g * 256 + lane * 4) = pos[g] * 16 + ((m >> (4 * g)) & 15u);
        unsigned ra[8], rb[8], sel[8];
#pragma unroll
        for (int j = 0; j < 8; ++j)
            sel[j] = *(LAS unsigned*)(uintptr_t)(sbase + 4096 + ((j >> 2) * 4 + (lane & 3)) * 256 + ((j & 3) * 16 + (lane >> 2)) * 4);
#pragma unroll
        for (int j = 0; j < 8; ++j) {
            const unsigned e = sel[j], d0 = e >> 6, d1 = d0 + 1;
            const unsigned sp = ((e & 15u) * 0x00204081u) & 0x01010101u, mk = (sp << 8) - sp, s2 = sp + (sp << 8);
            ra[j] = sbase + (d0 ^ ((d0 >> 4) & 3u)) * 4;
            rb[j] = sbase + (d1 ^ ((d1 >> 4) & 3u)) * 4;
            sel[j] = ((((s2 + (sp << 16)) << 8) + __builtin_amdgcn_perm(0u, (e >> 4) & 3u, 0u)) & mk) | (0x0c0c0c0cu & ~mk);
        }
#pragma unroll
        for (int j = 0; j < 8; ++j) asm volatile("" : "+v"(ra[j]), "+v"(rb[j]), "+v"(sel[j]));
        char* optr = (char*)ap->myout + (size_t)(dst * MLOC + m0 + (wid >> 2) * 64 + (wid & 1) * 32 + (lane >> 2)) * NSH
                     + nc0 + ((wid >> 1) & 1) * 64 + (lane & 3) * 16;
        char* sp = myws + WS_DATA + (gt * 8 + wid) * 2048 + lane * 16;
        asm volatile("" : "+v"(optr), "+v"(sp));
        if (!bad) bad = wait_flag((GAS unsigned*)(myws + WS_GO + gt * 128), tmo, call);
        const bool n0 = lane * 16u < tot, n1 = 1024u + lane * 16u < tot;
        u32x4 d0 = {0, 0, 0, 0}, d1 = {0, 0, 0, 0};
        if (!bad) {
            const unsigned long long t0 = __builtin_amdgcn_s_memrealtime();
            unsigned polls = 0;
            unsigned long long p0 = __ballot(n0), p1 = __ballot(n1);
            do {
                unsigned tw;
                asm volatile("s_mov_b64 exec, %4\n\tglobal_load_dwordx4 %0, %3, off sc0 sc1\n\t"
                             "s_mov_b64 exec, %5\n\tglobal_load_dwordx4 %1, %3, off offset:1024 sc0 sc1\n\t"
                             "s_mov_b64 exec, -1\n\tglobal_load_dword %2, %6, off sc1\n\ts_waitcnt vmcnt(0)"
                             : "+v"(d0), "+v"(d1), "=&v"(tw) : "v"(sp), "s"(p0), "s"(p1), "v"(tmo) : "memory");
                p0 &= __ballot((((d0.x & 0x80808080u) ^ tg0) | ((d0.y & 0x80808080u) ^ tg1) | ((d0.z & 0x80808080u) ^ tg0) | ((d0.w & 0x80808080u) ^ tg1)) != 0);
                p1 &= __ballot((((d1.x & 0x80808080u) ^ tg0) | ((d1.y & 0x80808080u) ^ tg1) | ((d1.z & 0x80808080u) ^ tg0) | ((d1.w & 0x80808080u) ^ tg1)) != 0);
                if (rfl(tw) == call || ((p0 | p1) && give_up(tmo, call, t0, polls))) { bad = true; break; }
            } while (p0 | p1);
        }
        *(LAS u32x4*)(uintptr_t)(sbase + lane * 16) = d0 & 0x7f7f7f7fu;
        *(LAS u32x4*)(uintptr_t)(sbase + 1024 + lane * 16) = d1 & 0x7f7f7f7fu;
        const unsigned poison = bad ? 0x7f7f7f7fu : 0u;
        unsigned lo[8], hi[8];
#pragma unroll
        for (int g = 0; g < 8; ++g) { lo[g] = *(LAS unsigned*)(uintptr_t)ra[g]; hi[g] = *(LAS unsigned*)(uintptr_t)rb[g]; }
#pragma unroll
        for (int h = 0; h < 2; ++h) {
            u32x4 o;
#pragma unroll
            for (int i = 0; i < 4; ++i) o[i] = __builtin_amdgcn_perm(hi[h * 4 + i], lo[h * 4 + i], sel[h * 4 + i]) | poison;
            asm volatile("global_store_dwordx4 %0, %1, off sc0 sc1\n\ts_nop 1" :: "v"(optr + h * 16 * NSH), "v"(o) : "memory");
        }
        const u32x4 z = {0, 0, 0, 0};
        if (n0) asm volatile("global_store_dwordx4 %0, %1, off sc0 sc1\n\ts_nop 1" :: "v"(sp), "v"(z) : "memory");
        if (n1) asm volatile("global_store_dwordx4 %0, %1, off offset:1024 sc0 sc1\n\ts_nop 1" :: "v"(sp), "v"(z) : "memory");
        WAIT_VM(0);
    }
    asm volatile(".Lgemm_a2a_e1:");
    if (tid == 256) __hip_atomic_store((GAS unsigned*)(myws + WS_CNT) + b, call, __ATOMIC_RELAXED, __HIP_MEMORY_SCOPE_AGENT);
}

}

extern "C" void kernel_launch(int rank, int nranks, ncclComm_t comm,
                              hipStream_t stream,
                              void* const* d_in, const int* in_sizes, int n_in,
                              void* d_out, int out_size,
                              const int* in_dtypes, int out_dtype,
                              void* const* d_out_all,
                              void* const* d_ws_all, size_t ws_bytes) {
    if (nranks != NR || n_in < 2 || in_sizes[0] != MLOC * KD || in_sizes[1] != KD * ND ||
        out_size != ND * NSH || ws_bytes < WS_END || (unsigned)rank >= (unsigned)NR) {
        fprintf(stderr, "kernel_launch: unexpected problem shape\n");
        return;
    }
    HIP_CHECK(hipFuncSetAttribute((const void*)gemm_a2a_kernel, hipFuncAttributeMaxDynamicSharedMemorySize, LDS_BYTES));
    Args a;
    a.x = d_in[0]; a.w = d_in[1]; a.myws = d_ws_all[rank]; a.myout = d_out; a.rank = rank; a.pad = 0;
    for (int p = 0; p < NR; ++p) a.ws[p] = d_ws_all[p];
    hipLaunchKernelGGL(gemm_a2a_kernel, dim3(NWG), dim3(512), LDS_BYTES, stream, a);
    HIP_CHECK(hipGetLastError());
}
```

```cpp
#include <hip/hip_runtime.h>
#include <rccl/rccl.h>
#include <cstdio>
#include <cstdint>
#include <cstddef>

#define HIP_CHECK(expr) do { hipError_t _e = (expr); if (_e != hipSuccess) { \
    fprintf(stderr, "HIP error %s at %s:%d\n", hipGetErrorString(_e), __FILE__, __LINE__); \
    return; } } while (0)

namespace {

typedef short          s16x4 __attribute__((ext_vector_type(4)));
typedef short          s16x8 __attribute__((ext_vector_type(8)));
typedef float          f32x4 __attribute__((ext_vector_type(4)));
typedef unsigned       u32x4 __attribute__((ext_vector_type(4)));
typedef unsigned       u32x2 __attribute__((ext_vector_type(2)));
#define LAS __attribute__((address_space(3)))
#define GAS __attribute__((address_space(1)))

constexpr int NR = 8;
constexpr int MLOC = 512, KD = 4096, ND = 4096, NSH = 512;
constexpr int BM = 128, BN = 128, BK = 64;
constexpr int NKT = KD / BK;
constexpr int KA = 28;
constexpr int LASTA = 12, LASTB = 16;
constexpr int XBYTES = BM * BK * 2;
constexpr int SLOT = XBYTES + BK * BN * 2;
constexpr int NS = 4;
constexpr int CTRL = NS * SLOT;
constexpr int LDS_BYTES = CTRL + 256;
constexpr int NWG = 256, NTILE = 128;
constexpr int CSTR = 8192;
static_assert(KA % 4 == 0 && KA >= 8 && NKT - KA >= 8, "");

constexpr size_t WS_CNT   = 0;
constexpr size_t WS_TMO   = 4096;
constexpr size_t WS_AMAX  = 8192;
constexpr size_t WS_GO    = WS_AMAX + 2 * 8192;
constexpr size_t WS_PFLAG = WS_GO + NTILE * 128;
constexpr size_t WS_MASK  = 1 << 20;
constexpr size_t WS_DATA  = 2 << 20;
constexpr size_t WS_PART  = 4 << 20;
constexpr size_t WS_END   = WS_PART + (size_t)NTILE * 65536;
static_assert(WS_PFLAG + NTILE * 512 <= WS_MASK, "");
constexpr unsigned long long DEADLINE = 100000000ull;

#define BARRIER() do { asm volatile("" ::: "memory"); __builtin_amdgcn_s_barrier(); asm volatile("" ::: "memory"); } while (0)
#define WAIT_VM(n) asm volatile("s_waitcnt vmcnt(%0)" :: "n"(n) : "memory")
#define WAIT_LGKM0() __builtin_amdgcn_s_waitcnt(0xC07F)
#define SB() __builtin_amdgcn_sched_barrier(0)

__device__ __forceinline__ unsigned rfl(unsigned v) { return __builtin_amdgcn_readfirstlane(v); }

__device__ __forceinline__ unsigned wave_max_u32(unsigned v) {
    v = max(v, (unsigned)__builtin_amdgcn_update_dpp(0, (int)v, 0xB1, 0xF, 0xF, true));
    v = max(v, (unsigned)__builtin_amdgcn_update_dpp(0, (int)v, 0x4E, 0xF, 0xF, true));
    v = max(v, (unsigned)__builtin_amdgcn_update_dpp(0, (int)v, 0x141, 0xF, 0xF, true));
    v = max(v, (unsigned)__builtin_amdgcn_update_dpp(0, (int)v, 0x140, 0xF, 0xF, true));
    unsigned a = __builtin_amdgcn_readlane(v, 0), b = __builtin_amdgcn_readlane(v, 16);
    unsigned c = __builtin_amdgcn_readlane(v, 32), d = __builtin_amdgcn_readlane(v, 48);
    return max(max(a, b), max(c, d));
}

__device__ __forceinline__ unsigned wave_scan_u32(unsigned v) {
    v += (unsigned)__builtin_amdgcn_update_dpp(0, (int)v, 0x111, 0xF, 0xF, true);
    v += (unsigned)__builtin_amdgcn_update_dpp(0, (int)v, 0x112, 0xF, 0xF, true);
    v += (unsigned)__builtin_amdgcn_update_dpp(0, (int)v, 0x114, 0xF, 0xF, true);
    v += (unsigned)__builtin_amdgcn_update_dpp(0, (int)v, 0x118, 0xF, 0xF, true);
    v += (unsigned)__builtin_amdgcn_update_dpp(0, (int)v, 0x142, 0xA, 0xF, false);
    v += (unsigned)__builtin_amdgcn_update_dpp(0, (int)v, 0x143, 0xC, 0xF, false);
    return v;
}
__device__ __forceinline__ unsigned stream_offsets(unsigned m, unsigned (&pos)[8]) {
    unsigned c = m - ((m >> 1) & 0x55555555u);
    c = (c & 0x33333333u) + ((c >> 2) & 0x33333333u);
    unsigned ex[4], tot[4];
#pragma unroll
    for (int j = 0; j < 4; ++j) {
        const unsigned f = (c >> (4 * j)) & 0x000f000fu, inc = wave_scan_u32(f);
        tot[j] = __builtin_amdgcn_readlane(inc, 63); ex[j] = inc - f;
    }
    unsigned base = 0;
#pragma unroll
    for (int g = 0; g < 8; ++g) {
        pos[g] = base + (g < 4 ? ex[g] & 0xffffu : ex[g - 4] >> 16);
        base += g < 4 ? tot[g] & 0xffffu : tot[g - 4] >> 16;
    }
    return base;
}
__device__ __forceinline__ unsigned tag_bits(unsigned n) { return ((n * 0x00204081u) & 0x01010101u) << 7; }

__device__ __forceinline__ bool give_up(GAS unsigned* tmo, unsigned call, unsigned long long t0, unsigned& polls) {
    if (__builtin_expect((++polls & 31u) != 0u, 1)) return false;
    if (rfl(__hip_atomic_load(tmo, __ATOMIC_RELAXED, __HIP_MEMORY_SCOPE_AGENT)) == call ||
        __builtin_amdgcn_s_memrealtime() - t0 > DEADLINE) {
        __hip_atomic_store(tmo, call, __ATOMIC_RELAXED, __HIP_MEMORY_SCOPE_AGENT);
        return true;
    }
    return false;
}

__device__ __forceinline__ bool wait_flag(GAS unsigned* flag, GAS unsigned* tmo, unsigned call) {
    const unsigned long long t0 = __builtin_amdgcn_s_memrealtime();
    unsigned polls = 0;
    while (rfl(__hip_atomic_load(flag, __ATOMIC_RELAXED, __HIP_MEMORY_SCOPE_SYSTEM)) < call) {
        if (give_up(tmo, call, t0, polls)) return true;
        __builtin_amdgcn_s_sleep(1);
    }
    return false;
}

#define GLDS(rsrc, ldsaddr, voff, soff) \
    asm volatile("s_mov_b32 m0, %0\n\ts_nop 0\n\tbuffer_load_dwordx4 %1, %2, %3 offen lds" \
                 :: "s"(ldsaddr), "v"(voff), "s"(rsrc), "s"(soff) : "memory")
#define GLDS_SYS(rsrc, ldsaddr, voff, soff) \
    asm volatile("s_mov_b32 m0, %0\n\ts_nop 0\n\tbuffer_load_dwordx4 %1, %2, %3 offen sc0 sc1 lds" \
                 :: "s"(ldsaddr), "v"(voff), "s"(rsrc), "s"(soff) : "memory")

struct Frag { s16x8 x[8]; s16x4 w[16]; };

template <int SL, int R>
__device__ __forceinline__ void rd(Frag& f, const unsigned (&ax)[2][2], const unsigned (&aw)[2][4]) {
    constexpr int g = R / 3, e = R % 3, so = (SL & 1) * SLOT, ks = g >> 2, i = g & 3;
    if constexpr (e == 0) f.x[g] = *(LAS s16x8*)(uintptr_t)(ax[SL >> 1][ks] + so + i * 2048);
    else f.w[g * 2 + e - 1] = __builtin_amdgcn_ds_read_tr16_b64_v4i16(
             (LAS s16x4*)(uintptr_t)(aw[SL >> 1][i] + so + ks * 8192 + (e - 1) * 1024));
}
template <int SL, int R>
__device__ __forceinline__ void rd_all(Frag& f, const unsigned (&ax)[2][2], const unsigned (&aw)[2][4]) {
    if constexpr (R < 24) { rd<SL, R>(f, ax, aw); rd_all<SL, R + 1>(f, ax, aw); }
}
template <int G, bool PAD = false>
__device__ __forceinline__ void mfma(f32x4 (&acc)[16], const Frag& cur) {
    constexpr int ks = G >> 4, mi = (G >> 2) & 3, c = (mi & 1) ? 3 - (G & 3) : G & 3;
    const s16x8 wf = __builtin_shufflevector(cur.w[(ks * 4 + c) * 2], cur.w[(ks * 4 + c) * 2 + 1], 0, 1, 2, 3, 4, 5, 6, 7);
    if constexpr (PAD)
        asm volatile("s_nop 1\n\tv_mfma_f32_16x16x32_bf16 %0, %1, %2, %0" : "+v"(acc[mi * 4 + c]) : "v"(wf), "v"(cur.x[ks * 4 + mi]));
    else
        asm volatile("v_mfma_f32_16x16x32_bf16 %0, %1, %2, %0" : "+v"(acc[mi * 4 + c]) : "v"(wf), "v"(cur.x[ks * 4 + mi]));
}
template <int G>
__device__ __forceinline__ void give(f32x4 (&acc)[16], char* pp) {
    constexpr int mi = (G >> 2) & 3, i = mi * 4 + ((mi & 1) ? 3 - (G & 3) : G & 3);
    asm volatile("global_store_dwordx4 %0, %1, off offset:%2 sc0 sc1\n\ts_nop 1"
                 :: "v"(pp + (i >> 2) * 4096), "v"(acc[i]), "n"((i & 3) * 1024) : "memory");
}
template <int SL, int G, bool BAR = true>
__device__ __forceinline__ void kstep(f32x4 (&acc)[16], const Frag& cur, Frag& nxt,
                                      const unsigned (&ax)[2][2], const unsigned (&aw)[2][4]) {
    if constexpr (G < 32) {
        mfma<G>(acc, cur);
        if constexpr (G < 24) rd<SL, G>(nxt, ax, aw);
        if constexpr (G == 31) { SB(); WAIT_LGKM0(); if constexpr (BAR) BARRIER(); }
        SB();
        kstep<SL, G + 1, BAR>(acc, cur, nxt, ax, aw);
    }
}
template <int G, int END, bool PAD>
__device__ __forceinline__ void klast(f32x4 (&acc)[16], const Frag& cur) {
    if constexpr (G < END) { mfma<G, PAD>(acc, cur); SB(); klast<G + 1, END, PAD>(acc, cur); }
}
template <int G>
__device__ __forceinline__ void klast_give(f32x4 (&acc)[16], const Frag& cur, char* pp) {
    if constexpr (G < 32) mfma<G, true>(acc, cur);
    if constexpr (G == 32) asm volatile("s_nop 7" ::: "memory");
    if constexpr (G >= 20) give<G - 4>(acc, pp);
    SB();
    if constexpr (G < 35) klast_give<G + 1>(acc, cur, pp);
}

struct Args {
    const void* x; const void* w;
    void* myws; void* myout;
    void* ws[NR];
    int rank, pad;
};

__global__ void __launch_bounds__(512, 2)
gemm_a2a_kernel(const Args args)
{
    typedef __attribute__((address_space(4))) const Args* ArgP;
    ArgP ap = (ArgP)__builtin_amdgcn_kernarg_segment_ptr();
    unsigned long long xp, wp;
    static_assert(offsetof(Args, x) == 0 && offsetof(Args, w) == 8, "");
    asm volatile(".Lgemm_a2a_top:\n\ts_load_dwordx2 %0, %2, 0x0\n\ts_load_dwordx2 %1, %2, 0x8" : "=&s"(xp), "=&s"(wp) : "s"(ap));
    SB();
    int rank;
#define ARGS_READY() \
    asm volatile("" : "+s"(ap)); \
    rank = ap->rank; myws = (char*)ap->myws; tmo = (GAS unsigned*)(myws + WS_TMO); \
    pflag = (GAS unsigned*)(myws + WS_PFLAG + gt * 512); part = myws + WS_PART + (size_t)gt * 65536
    extern __shared__ __attribute__((aligned(16))) char smem[];
    LAS char* lds = (LAS char*)smem;
    const unsigned ldsb = (unsigned)(uintptr_t)lds;
    const int tid = threadIdx.x, lane = tid & 63;
    const int wid = rfl(tid >> 6);
    const int b = blockIdx.x;
    const int dst = b & 7;
    const int isB = (b >> 3) & 1, tile = b >> 4, tm = tile & 3, tn = tile >> 2;
    const int gt = dst * 16 + tile;
    const int m0 = tm * BM;
    const int nc0 = tn * BN;
    const int nst = isB ? NKT - KA : KA, kbase = isB ? KA : 0;
    char *myws, *part; GAS unsigned *tmo, *pflag;
    LAS unsigned* ctrl = (LAS unsigned*)(lds + CTRL);

    f32x4 y[8];
    unsigned call, bad = 0;

    if (__builtin_expect(wid >= 4, 1)) {
        const int lw = wid - 4;
        const int ph = (dst + 2 * isB) & 3, rot = (tm + tn) & 3;
        const unsigned k0 = kbase + ((0x244400E4u >> (ph * 8 + rot * 2)) & 3);
        const unsigned lx = ldsb + lw * 4096, lwb = ldsb + XBYTES + lw * 4096;
        unsigned vx[4], vw[4];
#pragma unroll
        for (int i = 0; i < 4; ++i) {
            const int k = 16 * lw + 4 * i + (lane >> 4), f = ((k & 3) << 1) | (k & 8);
            vw[i] = (unsigned)(k * (ND * 2) + (dst * NSH + nc0) * 2 + (((lane & 15) ^ f) << 4));
            asm volatile("" : "+v"(vw[i]));
        }
        asm volatile("s_waitcnt lgkmcnt(0)" : "+s"(xp), "+s"(wp));
        __amdgpu_buffer_rsrc_t rx = __builtin_amdgcn_make_buffer_rsrc((void*)xp, 0, MLOC * KD * 2, 0x00020000);
        __amdgpu_buffer_rsrc_t rw = __builtin_amdgcn_make_buffer_rsrc((void*)wp, 0, KD * ND * 2, 0x00020000);
#pragma unroll
        for (int i = 0; i < 4; ++i) GLDS(rw, lwb + i * 1024, vw[i], k0 * (BK * ND * 2));
        SB();
#pragma unroll
        for (int i = 0; i < 4; ++i) {
            const int row = 32 * lw + 8 * i + (lane >> 3);
            vx[i] = (unsigned)((m0 + row) * (KD * 2) + (((lane & 7) ^ ((row >> 1) & 7)) << 4));
            GLDS(rx, lx + i * 1024, vx[i], k0 * (BK * 2));
        }
        SB();
        int ktab;
        {
            const int ls = ph + (isB ? NKT - KA - LASTB : KA - LASTA);
            const int bs = lane < ph ? 0 : lane >= ls ? ls : ph + ((lane - ph) & ~3);
            const int sz = lane < ph ? ph : lane >= ls ? nst - ls : 4;
            ktab = kbase + bs + (lane - bs + rot * ((sz + 3) >> 2)) % sz;
        }
#define ISSUE(t) do { \
            const unsigned kt_ = __builtin_amdgcn_readlane(ktab, t), so_ = ((t) & 3) * SLOT; \
            _Pragma("unroll") for (int i = 0; i < 4; ++i) GLDS(rw, lwb + so_ + i * 1024, vw[i], kt_ * (BK * ND * 2)); \
            _Pragma("unroll") for (int i = 0; i < 4; ++i) GLDS(rx, lx + so_ + i * 1024, vx[i], kt_ * (BK * 2)); } while (0)
        ISSUE(1);
        ARGS_READY();
        WAIT_VM(8); BARRIER();
        ISSUE(2);
        WAIT_VM(8); BARRIER();
        unsigned cv, fl = 0;
        asm volatile("global_load_dword %0, %1, off sc1" : "=v"(cv) : "v"(myws + WS_CNT + b * 4) : "memory");
        char* peerws;
        asm volatile("global_load_dwordx2 %0, %1, off" : "=v"(peerws)
                     : "v"((const char*)ap + offsetof(Args, ws) + (lane & 7) * 8) : "memory");
        ISSUE(3);
        for (int t = 0; t < nst - NS; ++t) {
            ISSUE(t + NS);
            if (t == nst - NS - 2) asm volatile("global_load_dword %0, %1, off sc0 sc1" : "=v"(fl) : "v"(pflag + lw * 32) : "memory");
            WAIT_VM(16); BARRIER();
        }
        WAIT_VM(8);
        asm volatile("" : "+v"(fl), "+v"(cv), "+v"(peerws));
        call = rfl(cv) + 1;
        char* gp = peerws + WS_AMAX + (call & 1) * 8192 + (rank * NTILE + gt) * 8;
        asm volatile("" : "+v"(gp));
        if (!isB) {
            BARRIER(); WAIT_VM(0); BARRIER(); BARRIER();
            if (lw == 0) {
                const char* dws = (const char*)ap->ws[dst];
                const char* tp = lane < 8 ? gp : lane < 16 ? dws + WS_DATA + (rank * 16 + tile) * 16384 + (lane & 3) * 4096
                                                          : dws + WS_MASK + (call & 1) * 524288 + (rank * 16 + tile) * 4096;
                unsigned dummy;
                asm volatile("global_load_dword %0, %1, off sc0 sc1\n\ts_waitcnt vmcnt(0)" : "=&v"(dummy) : "v"(tp) : "memory");
                if (lane == 0) __hip_atomic_store((GAS unsigned*)(myws + WS_CNT) + b, call, __ATOMIC_RELAXED, __HIP_MEMORY_SCOPE_AGENT);
            }
        } else {
        {
            __amdgpu_buffer_rsrc_t rp = __builtin_amdgcn_make_buffer_rsrc((void*)part, 0, 65536, 0x00020000);
            const unsigned vp = lane * 16;
#define POLL_PART() asm volatile("global_load_dword %0, %1, off sc0 sc1" : "=v"(fl) : "v"(pflag + lw * 32) : "memory")
#define FETCH_HALF() do { const unsigned hl_ = ldsb + nh * SLOT + lw * 8192, hs_ = lw * 16384 + nh * 8192; \
                _Pragma("unroll") for (int i = 0; i < 8; ++i) GLDS_SYS(rp, hl_ + i * 1024, vp, hs_ + i * 1024); ++nh; } while (0)
            bool seen = rfl(fl) >= call;
            unsigned nh = 0;
            if (seen) FETCH_HALF(); else POLL_PART();
            BARRIER();
            if (seen) { FETCH_HALF(); WAIT_VM(16); }
            else {
                WAIT_VM(0); asm volatile("" : "+v"(fl)); seen = rfl(fl) >= call;
                if (seen) FETCH_HALF(); else POLL_PART();
            }
            BARRIER();
            if (!seen) { WAIT_VM(0); asm volatile("" : "+v"(fl)); if (rfl(fl) < call) bad = wait_flag(pflag + lw * 32, tmo, call); }
#pragma nounroll
            while (nh < 2) FETCH_HALF();
            WAIT_VM(0); BARRIER();
        }
        BARRIER();
        if (lw == 0) {
            const u32x4 m4 = *(LAS u32x4*)ctrl;
            const u32x2 gr = {max(max(m4.x, m4.y), max(m4.z, m4.w)), call};
            if (lane < NR) asm volatile("global_store_dwordx2 %0, %1, off sc0 sc1\n\ts_nop 1" :: "v"(gp), "v"(gr) : "memory");
            if (lane == 0) __hip_atomic_store((GAS unsigned*)(myws + WS_CNT) + b, call, __ATOMIC_RELAXED, __HIP_MEMORY_SCOPE_AGENT);
        }
#pragma unroll
        for (int i = 0; i < 8; ++i) y[i] = *(LAS f32x4*)(lds + 8 * CSTR + lw * 8192 + i * 1024 + lane * 16);
        }
    } else {
        const int wm = wid >> 1, wn = wid & 1;
        const int fr = lane & 15, fq = lane >> 4, q = (lane & 15) >> 2, p = lane & 3;
        asm volatile("s_waitcnt lgkmcnt(0)" : "+s"(xp), "+s"(wp));
        if (b < 64) {
            unsigned long long pc; unsigned len, rel;
            asm volatile("s_getpc_b64 %0\n.Lgemm_a2a_here:\n\t"
                         "s_mov_b32 %1, .Lgemm_a2a_here-.Lgemm_a2a_top\n\t"
                         "s_mov_b32 %2, .Lgemm_a2a_e1-.Lgemm_a2a_top\n\t"
                         "s_max_u32 %2, %2, .Lgemm_a2a_e2-.Lgemm_a2a_top" : "=s"(pc), "=s"(rel), "=s"(len) :: "scc");
            pc -= rel;
            const unsigned off = (((b >> 3) * 4 + wid) * 8 + (lane & 7)) * 128;
            if (lane < 8 && off < len) {
                unsigned dummy;
                asm volatile("global_load_dword %0, %1, off\n\ts_waitcnt vmcnt(0)" : "=&v"(dummy) : "v"((const char*)pc + off) : "memory");
            }
        }
        ARGS_READY();
        unsigned callv = __hip_atomic_load((GAS unsigned*)(myws + WS_CNT) + b, __ATOMIC_RELAXED, __HIP_MEMORY_SCOPE_AGENT);
        unsigned ax[2][2], aw[2][4];
#pragma unroll
        for (int ks = 0; ks < 2; ++ks) {
            const int row = wm * 64 + fr;
            ax[0][ks] = ldsb + row * 128 + (((ks * 4 + fq) ^ ((row >> 1) & 7)) << 4);
            ax[1][ks] = ax[0][ks] + 2 * SLOT;
            asm volatile("" : "+v"(ax[0][ks]), "+v"(ax[1][ks]));
        }
#pragma unroll
        for (int c = 0; c < 4; ++c) {
            const int k = 8 * fq + q, f = ((k & 3) << 1) | (k & 8);
            aw[0][c] = ldsb + XBYTES + k * 256 + (((wn * 8 + 2 * c + (p >> 1)) ^ f) << 4) + (p & 1) * 8;
            aw[1][c] = aw[0][c] + 2 * SLOT;
            asm volatile("" : "+v"(aw[0][c]), "+v"(aw[1][c]));
        }
        f32x4 acc[16];
#pragma unroll
        for (int i = 0; i < 16; ++i) { acc[i] = (f32x4){0.f, 0.f, 0.f, 0.f}; asm volatile("" : "+v"(acc[i])); }
        Frag f0, f1;
        BARRIER();
        rd_all<0, 0>(f0, ax, aw);
        WAIT_LGKM0(); BARRIER();
        asm volatile("s_nop 1");
        for (int t = 0; t < nst - 4; t += 4) {
            kstep<1, 0>(acc, f0, f1, ax, aw); kstep<2, 0>(acc, f1, f0, ax, aw);
            kstep<3, 0>(acc, f0, f1, ax, aw); kstep<0, 0>(acc, f1, f0, ax, aw);
        }
        call = rfl(callv) + 1;
        SB();
        kstep<1, 0>(acc, f0, f1, ax, aw); kstep<2, 0>(acc, f1, f0, ax, aw);
        kstep<3, 0, false>(acc, f0, f1, ax, aw);
        klast<0, 16, false>(acc, f1);
        if (!isB) {
            BARRIER();
            klast_give<16>(acc, f1, part + wid * 16384 + lane * 16);
            WAIT_VM(0);
            if (lane == 0) __hip_atomic_store(pflag + wid * 32, call, __ATOMIC_RELAXED, __HIP_MEMORY_SCOPE_SYSTEM);
        } else {
        BARRIER();
        f32x4 pa[16];
#pragma unroll
        for (int i = 0; i < 16; ++i) pa[i] = *(LAS f32x4*)(lds + (i < 8 ? 0 : SLOT) + wid * 8192 + (i & 7) * 1024 + lane * 16);
        SB();
        klast<16, 32, true>(acc, f1);
        asm volatile("s_nop 7" ::: "memory");
#pragma unroll
        for (int i = 0; i < 16; ++i) asm volatile("" : "+v"(acc[i]));
#pragma unroll
        for (int i = 8; i < 16; ++i) {
            acc[i] += pa[i];
            *(LAS f32x4*)(lds + 8 * CSTR + wid * 8192 + (i - 8) * 1024 + lane * 16) = acc[i];
        }
        float mx4[4] = {0.f, 0.f, 0.f, 0.f};
#pragma unroll
        for (int i = 0; i < 8; ++i) y[i] = acc[i] + pa[i];
#pragma unroll
        for (int i = 0; i < 8; ++i)
#pragma unroll
            for (int e = 0; e < 4; ++e) mx4[e] = __builtin_fmaxf(__builtin_fmaxf(acc[8 + i][e], y[i][e]), mx4[e]);
        const float mxf = fmaxf(fmaxf(mx4[0], mx4[1]), fmaxf(mx4[2], mx4[3]));
        const unsigned wmx = wave_max_u32(__float_as_uint(mxf) & 0x7fffffffu);
        if (lane == 0) ctrl[wid] = wmx;
        WAIT_LGKM0(); BARRIER();
        asm volatile(".Lgemm_a2a_e2:");
        }
    }
    const unsigned tag = call % 255u + 1u, tg0 = tag_bits(tag & 15u), tg1 = tag_bits(tag >> 4);
    if (isB) {
        const int sid = (wid & 3) * 2 + (wid >> 2), sidx = (rank * 16 + tile) * 8 + sid;
        char* dws = (char*)ap->ws[dst];
        unsigned mq[4];
#pragma unroll
        for (int c = 0; c < 4; ++c) mq[c] = __float_as_uint(y[2 * c + 1][3]) >> 31;
#pragma unroll
        for (int e = 6; e >= 0; --e)
#pragma unroll
            for (int c = 0; c < 4; ++c) mq[c] = __builtin_amdgcn_alignbit(mq[c], __float_as_uint(y[2 * c + (e >> 2)][e & 3]), 31);
        const unsigned m = ~(mq[0] | (mq[1] << 8) | (mq[2] << 16) | (mq[3] << 24));
        {
            const u32x2 mg = {m, call};
            asm volatile("global_store_dwordx2 %0, %1, off sc0 sc1\n\ts_nop 1"
                         :: "v"(dws + WS_MASK + (call & 1) * 524288 + (sidx * 64 + lane) * 8), "v"(mg) : "memory");
        }
        unsigned wa[8];
        const unsigned total = stream_offsets(m, wa);
        const unsigned cbase = ldsb + sid * CSTR, trash = ldsb + 12 * CSTR + wid * 512 + lane * 4;
#pragma unroll
        for (int g = 0; g < 8; ++g) wa[g] = cbase + wa[g] * 4;
#pragma unroll
        for (int e = 0; e < 4; ++e) {
            unsigned sg[8], ad[8];
#pragma unroll
            for (int i = 0; i < 8; ++i) asm("v_ashrrev_i32 %0, 31, %1" : "=v"(sg[i]) : "v"(y[i][e]));
#pragma unroll
            for (int i = 0; i < 8; ++i) asm("v_bfi_b32 %0, %1, %2, %3" : "=v"(ad[i]) : "v"(sg[i]), "v"(trash), "v"(wa[i]));
#pragma unroll
            for (int i = 0; i < 8; ++i) {
                *(LAS float*)(uintptr_t)(ad[i] + 4 * e) = y[i][e];
                if (e < 3) asm("v_lshl_add_u32 %0, %1, 2, %0" : "+v"(wa[i]) : "v"(sg[i]));
            }
        }
        f32x4 v[2][4];
#pragma unroll
        for (int h = 0; h < 2; ++h)
#pragma unroll
            for (int i = 0; i < 4; ++i) v[h][i] = *(LAS f32x4*)(uintptr_t)(cbase + h * 4096 + lane * 64 + ((i * 16) ^ ((lane << 2) & 48)));
        char* dptr = dws + WS_DATA + sidx * 2048 + lane * 16;
        asm volatile("" : "+v"(dptr));
        if (wid >= 4) {
            const int lw = wid & 3;
            const char* g = myws + WS_AMAX + (call & 1) * 8192 + lw * 2048 + lane * 16;
            unsigned mx = 0;
            const unsigned long long t0 = __builtin_amdgcn_s_memrealtime();
            unsigned polls = 0;
            for (;;) {
                u32x4 a, c;
                asm volatile("global_load_dwordx4 %0, %2, off sc0 sc1\n\t"
                             "global_load_dwordx4 %1, %2, off offset:1024 sc0 sc1\n\t"
                             "s_waitcnt vmcnt(0)"
                             : "=&v"(a), "=&v"(c) : "v"(g) : "memory");
                const bool ok = a.y == call && a.w == call && c.y == call && c.w == call;
                mx = max(max(a.x, a.z), max(c.x, c.z));
                if (__all(ok)) break;
                if (give_up(tmo, call, t0, polls)) { bad = 1; break; }
                __builtin_amdgcn_s_sleep(1);
            }
            mx = wave_max_u32(mx);
            if (lane == 0) ctrl[8 + lw] = mx;
        }
        if (lane == 0) ctrl[12 + wid] = bad;
        WAIT_LGKM0(); BARRIER();
        if (tid == 0) __hip_atomic_store((GAS unsigned*)(myws + WS_GO + gt * 128), call, __ATOMIC_RELAXED, __HIP_MEMORY_SCOPE_AGENT);
        const u32x4 g4 = *(LAS u32x4*)(ctrl + 8), b4 = *(LAS u32x4*)(ctrl + 12) | *(LAS u32x4*)(ctrl + 16);
        const float amax = __uint_as_float(max(max(g4.x, g4.y), max(g4.z, g4.w)));
        const bool timed_out = (b4.x | b4.y | b4.z | b4.w) != 0;
        float inv = 448.0f * __builtin_amdgcn_rcpf(amax);
        if (__builtin_expect(!(amax > 1e-30f && amax < 1e30f), 0)) inv = amax > 0.f ? 1.0f / (amax / 448.0f) : 0.f;
#pragma unroll
        for (int h = 0; h < 2; ++h) {
            if (h * 1024u < total) {
                u32x4 o;
#pragma unroll
                for (int i = 0; i < 4; ++i) {
                    int r = __builtin_amdgcn_cvt_pk_fp8_f32(fminf(v[h][i][0] * inv, 448.f), fminf(v[h][i][1] * inv, 448.f), 0, false);
                    r = __builtin_amdgcn_cvt_pk_fp8_f32(fminf(v[h][i][2] * inv, 448.f), fminf(v[h][i][3] * inv, 448.f), r, true);
                    o[i] = ((timed_out ? 0x7f7f7f7fu : (unsigned)r) & 0x7f7f7f7fu) | ((i & 1) ? tg1 : tg0);
                }
                if (h * 1024u + lane * 16u < total)
                    asm volatile("global_store_dwordx4 %0, %1, off offset:%2 sc0 sc1\n\ts_nop 1" :: "v"(dptr), "v"(o), "n"(h * 1024) : "memory");
            }
        }
    } else {
        unsigned m;
        {
            const char* mp = myws + WS_MASK + (call & 1) * 524288 + ((gt * 8 + wid) * 64 + lane) * 8;
            const unsigned long long t0 = __builtin_amdgcn_s_memrealtime();
            unsigned polls = 0;
            for (;;) {
                u32x2 g;
                asm volatile("global_load_dwordx2 %0, %1, off sc0 sc1\n\ts_waitcnt vmcnt(0)" : "=&v"(g) : "v"(mp) : "memory");
                m = g.x;
                if (__all(g.y == call)) break;
                if (give_up(tmo, call, t0, polls)) { bad = true; m = 0; break; }
                __builtin_amdgcn_s_sleep(8);
            }
        }
        unsigned pos[8];
        const unsigned tot = stream_offsets(m, pos);
        const unsigned sbase = ldsb + wid * 8192;
#pragma unroll
        for (int g = 0; g < 8; ++g) *(LAS unsigned*)(uintptr_t)(sbase + 4096 + g * 256 + lane * 4) = pos[g] * 16 + ((m >> (4 * g)) & 15u);
        unsigned ra[8], rb[8], sel[8];
#pragma unroll
        for (int j = 0; j < 8; ++j)
            sel[j] = *(LAS unsigned*)(uintptr_t)(sbase + 4096 + ((j >> 2) * 4 + (lane & 3)) * 256 + ((j & 3) * 16 + (lane >> 2)) * 4);
#pragma unroll
        for (int j = 0; j < 8; ++j) {
            const unsigned e = sel[j], d0 = e >> 6, d1 = d0 + 1;
            const unsigned sp = ((e & 15u) * 0x00204081u) & 0x01010101u, mk = (sp << 8) - sp, s2 = sp + (sp << 8);
            ra[j] = sbase + (d0 ^ ((d0 >> 4) & 3u)) * 4;
            rb[j] = sbase + (d1 ^ ((d1 >> 4) & 3u)) * 4;
            sel[j] = ((((s2 + (sp << 16)) << 8) + __builtin_amdgcn_perm(0u, (e >> 4) & 3u, 0u)) & mk) | (0x0c0c0c0cu & ~mk);
        }
#pragma unroll
        for (int j = 0; j < 8; ++j) asm volatile("" : "+v"(ra[j]), "+v"(rb[j]), "+v"(sel[j]));
        char* optr = (char*)ap->myout + (size_t)(dst * MLOC + m0 + (wid >> 2) * 64 + (wid & 1) * 32 + (lane >> 2)) * NSH
                     + nc0 + ((wid >> 1) & 1) * 64 + (lane & 3) * 16;
        char* sp = myws + WS_DATA + (gt * 8 + wid) * 2048 + lane * 16;
        asm volatile("" : "+v"(optr), "+v"(sp));
        if (!bad) bad = wait_flag((GAS unsigned*)(myws + WS_GO + gt * 128), tmo, call);
        const bool n0 = lane * 16u < tot, n1 = 1024u + lane * 16u < tot;
        u32x4 d0 = {0, 0, 0, 0}, d1 = {0, 0, 0, 0};
        if (!bad) {
            const unsigned long long t0 = __builtin_amdgcn_s_memrealtime();
            unsigned polls = 0;
            unsigned long long p0 = __ballot(n0), p1 = __ballot(n1);
            do {
                unsigned tw;
                asm volatile("s_mov_b64 exec, %4\n\tglobal_load_dwordx4 %0, %3, off sc0 sc1\n\t"
                             "s_mov_b64 exec, %5\n\tglobal_load_dwordx4 %1, %3, off offset:1024 sc0 sc1\n\t"
                             "s_mov_b64 exec, -1\n\tglobal_load_dword %2, %6, off sc1\n\ts_waitcnt vmcnt(0)"
                             : "+v"(d0), "+v"(d1), "=&v"(tw) : "v"(sp), "s"(p0), "s"(p1), "v"(tmo) : "memory");
                p0 &= __ballot((((d0.x & 0x80808080u) ^ tg0) | ((d0.y & 0x80808080u) ^ tg1) | ((d0.z & 0x80808080u) ^ tg0) | ((d0.w & 0x80808080u) ^ tg1)) != 0);
                p1 &= __ballot((((d1.x & 0x80808080u) ^ tg0) | ((d1.y & 0x80808080u) ^ tg1) | ((d1.z & 0x80808080u) ^ tg0) | ((d1.w & 0x80808080u) ^ tg1)) != 0);
                if (rfl(tw) == call || ((p0 | p1) && give_up(tmo, call, t0, polls))) { bad = true; break; }
            } while (p0 | p1);
        }
        *(LAS u32x4*)(uintptr_t)(sbase + lane * 16) = d0 & 0x7f7f7f7fu;
        *(LAS u32x4*)(uintptr_t)(sbase + 1024 + lane * 16) = d1 & 0x7f7f7f7fu;
        const unsigned poison = bad ? 0x7f7f7f7fu : 0u;
        unsigned lo[8], hi[8];
#pragma unroll
        for (int g = 0; g < 8; ++g) { lo[g] = *(LAS unsigned*)(uintptr_t)ra[g]; hi[g] = *(LAS unsigned*)(uintptr_t)rb[g]; }
#pragma unroll
        for (int h = 0; h < 2; ++h) {
            u32x4 o;
#pragma unroll
            for (int i = 0; i < 4; ++i) o[i] = __builtin_amdgcn_perm(hi[h * 4 + i], lo[h * 4 + i], sel[h * 4 + i]) | poison;
            asm volatile("global_store_dwordx4 %0, %1, off sc0 sc1\n\ts_nop 1" :: "v"(optr + h * 16 * NSH), "v"(o) : "memory");
        }
        const u32x4 z = {0, 0, 0, 0};
        if (n0) asm volatile("global_store_dwordx4 %0, %1, off sc0 sc1\n\ts_nop 1" :: "v"(sp), "v"(z) : "memory");
        if (n1) asm volatile("global_store_dwordx4 %0, %1, off offset:1024 sc0 sc1\n\ts_nop 1" :: "v"(sp), "v"(z) : "memory");
        WAIT_VM(0);
    }
    asm volatile(".Lgemm_a2a_e1:");
}

}

extern "C" void kernel_launch(int rank, int nranks, ncclComm_t comm,
                              hipStream_t stream,
                              void* const* d_in, const int* in_sizes, int n_in,
                              void* d_out, int out_size,
                              const int* in_dtypes, int out_dtype,
                              void* const* d_out_all,
                              void* const* d_ws_all, size_t ws_bytes) {
    if (nranks != NR || n_in < 2 || in_sizes[0] != MLOC * KD || in_sizes[1] != KD * ND ||
        out_size != ND * NSH || ws_bytes < WS_END || (unsigned)rank >= (unsigned)NR) {
        fprintf(stderr, "kernel_launch: unexpected problem shape\n");
        return;
    }
    HIP_CHECK(hipFuncSetAttribute((const void*)gemm_a2a_kernel, hipFuncAttributeMaxDynamicSharedMemorySize, LDS_BYTES));
    Args a;
    a.x = d_in[0]; a.w = d_in[1]; a.myws = d_ws_all[rank]; a.myout = d_out; a.rank = rank; a.pad = 0;
    for (int p = 0; p < NR; ++p) a.ws[p] = d_ws_all[p];
    hipLaunchKernelGGL(gemm_a2a_kernel, dim3(NWG), dim3(512), LDS_BYTES, stream, a);
    HIP_CHECK(hipGetLastError());
}
```
